```python
import numpy as np

kernel_name = "gpu_a2a_gemm_m4096_k4096_n16384_w8a8_dqrelu"


ABSMAX_RTOL = 0.001

COMPUTE_DTYPE = "int8"
EPILOGUE = "dequant_relu"
IN_QUANT = "int8"
OUT_QUANT = None
IN_DTYPES = {"x": "int8", "w_mat": "int8",
             "scale_x": "float32", "scale_w": "float32"}
WORLD_SIZE = 8

def setup_inputs():

    x = np.random.randint(-127, 128, size=(4096, 4096)).astype(np.int8)
    w_mat = np.random.randint(-127, 128, size=(4096, 16384)).astype(np.int8)
    scale_x = (0.02 * (1.0 + np.random.rand(1))).astype(np.float32)
    scale_w = (0.02 * (1.0 + np.random.rand(1))).astype(np.float32)
    return {"x": x, "w_mat": w_mat, "scale_x": scale_x, "scale_w": scale_w}

def reference(x, w_mat, scale_x, scale_w):
    acc = x.astype(np.float64) @ w_mat.astype(np.float64)
    y = (acc.astype(np.float32) * (scale_x * scale_w)).astype(np.float32)
    return np.maximum(y, 0.0).astype(np.float32)
```

```cpp
#include <hip/hip_runtime.h>
#include <rccl/rccl.h>
#include <cstddef>
#include <cstdio>

#define COMPUTE_ASM \
  "v_mov_b32 v128, 0\n" \
  "v_mov_b32 v129, 0\n" \
  "v_mov_b32 v130, 0\n" \
  "v_mov_b32 v131, 0\n" \
  "v_mov_b32 v132, 0\n" \
  "v_mov_b32 v133, 0\n" \
  "v_mov_b32 v134, 0\n" \
  "v_mov_b32 v135, 0\n" \
  "v_mov_b32 v136, 0\n" \
  "v_mov_b32 v137, 0\n" \
  "v_mov_b32 v138, 0\n" \
  "v_mov_b32 v139, 0\n" \
  "v_mov_b32 v140, 0\n" \
  "v_mov_b32 v141, 0\n" \
  "v_mov_b32 v142, 0\n" \
  "v_mov_b32 v143, 0\n" \
  "v_mov_b32 v144, 0\n" \
  "v_mov_b32 v145, 0\n" \
  "v_mov_b32 v146, 0\n" \
  "v_mov_b32 v147, 0\n" \
  "v_mov_b32 v148, 0\n" \
  "v_mov_b32 v149, 0\n" \
  "v_mov_b32 v150, 0\n" \
  "v_mov_b32 v151, 0\n" \
  "v_mov_b32 v152, 0\n" \
  "v_mov_b32 v153, 0\n" \
  "v_mov_b32 v154, 0\n" \
  "v_mov_b32 v155, 0\n" \
  "v_mov_b32 v156, 0\n" \
  "v_mov_b32 v157, 0\n" \
  "v_mov_b32 v158, 0\n" \
  "v_mov_b32 v159, 0\n" \
  "v_mov_b32 v160, 0\n" \
  "v_mov_b32 v161, 0\n" \
  "v_mov_b32 v162, 0\n" \
  "v_mov_b32 v163, 0\n" \
  "v_mov_b32 v164, 0\n" \
  "v_mov_b32 v165, 0\n" \
  "v_mov_b32 v166, 0\n" \
  "v_mov_b32 v167, 0\n" \
  "v_mov_b32 v168, 0\n" \
  "v_mov_b32 v169, 0\n" \
  "v_mov_b32 v170, 0\n" \
  "v_mov_b32 v171, 0\n" \
  "v_mov_b32 v172, 0\n" \
  "v_mov_b32 v173, 0\n" \
  "v_mov_b32 v174, 0\n" \
  "v_mov_b32 v175, 0\n" \
  "v_mov_b32 v176, 0\n" \
  "v_mov_b32 v177, 0\n" \
  "v_mov_b32 v178, 0\n" \
  "v_mov_b32 v179, 0\n" \
  "v_mov_b32 v180, 0\n" \
  "v_mov_b32 v181, 0\n" \
  "v_mov_b32 v182, 0\n" \
  "v_mov_b32 v183, 0\n" \
  "v_mov_b32 v184, 0\n" \
  "v_mov_b32 v185, 0\n" \
  "v_mov_b32 v186, 0\n" \
  "v_mov_b32 v187, 0\n" \
  "v_mov_b32 v188, 0\n" \
  "v_mov_b32 v189, 0\n" \
  "v_mov_b32 v190, 0\n" \
  "v_mov_b32 v191, 0\n" \
  "v_mov_b32 v192, 0\n" \
  "v_mov_b32 v193, 0\n" \
  "v_mov_b32 v194, 0\n" \
  "v_mov_b32 v195, 0\n" \
  "v_mov_b32 v196, 0\n" \
  "v_mov_b32 v197, 0\n" \
  "v_mov_b32 v198, 0\n" \
  "v_mov_b32 v199, 0\n" \
  "v_mov_b32 v200, 0\n" \
  "v_mov_b32 v201, 0\n" \
  "v_mov_b32 v202, 0\n" \
  "v_mov_b32 v203, 0\n" \
  "v_mov_b32 v204, 0\n" \
  "v_mov_b32 v205, 0\n" \
  "v_mov_b32 v206, 0\n" \
  "v_mov_b32 v207, 0\n" \
  "v_mov_b32 v208, 0\n" \
  "v_mov_b32 v209, 0\n" \
  "v_mov_b32 v210, 0\n" \
  "v_mov_b32 v211, 0\n" \
  "v_mov_b32 v212, 0\n" \
  "v_mov_b32 v213, 0\n" \
  "v_mov_b32 v214, 0\n" \
  "v_mov_b32 v215, 0\n" \
  "v_mov_b32 v216, 0\n" \
  "v_mov_b32 v217, 0\n" \
  "v_mov_b32 v218, 0\n" \
  "v_mov_b32 v219, 0\n" \
  "v_mov_b32 v220, 0\n" \
  "v_mov_b32 v221, 0\n" \
  "v_mov_b32 v222, 0\n" \
  "v_mov_b32 v223, 0\n" \
  "v_mov_b32 v224, 0\n" \
  "v_mov_b32 v225, 0\n" \
  "v_mov_b32 v226, 0\n" \
  "v_mov_b32 v227, 0\n" \
  "v_mov_b32 v228, 0\n" \
  "v_mov_b32 v229, 0\n" \
  "v_mov_b32 v230, 0\n" \
  "v_mov_b32 v231, 0\n" \
  "v_mov_b32 v232, 0\n" \
  "v_mov_b32 v233, 0\n" \
  "v_mov_b32 v234, 0\n" \
  "v_mov_b32 v235, 0\n" \
  "v_mov_b32 v236, 0\n" \
  "v_mov_b32 v237, 0\n" \
  "v_mov_b32 v238, 0\n" \
  "v_mov_b32 v239, 0\n" \
  "v_mov_b32 v240, 0\n" \
  "v_mov_b32 v241, 0\n" \
  "v_mov_b32 v242, 0\n" \
  "v_mov_b32 v243, 0\n" \
  "v_mov_b32 v244, 0\n" \
  "v_mov_b32 v245, 0\n" \
  "v_mov_b32 v246, 0\n" \
  "v_mov_b32 v247, 0\n" \
  "v_mov_b32 v248, 0\n" \
  "v_mov_b32 v249, 0\n" \
  "v_mov_b32 v250, 0\n" \
  "v_mov_b32 v251, 0\n" \
  "v_mov_b32 v252, 0\n" \
  "v_mov_b32 v253, 0\n" \
  "v_mov_b32 v254, 0\n" \
  "v_mov_b32 v255, 0\n" \
  "s_mov_b32 %[sA], 0\n" \
  "v_add_u32 %[xc0], %[sA], %[xa0]\n" \
  "v_xor_b32 %[xc1], 64, %[xc0]\n" \
  "s_barrier\n" \
  "ds_read_b128 v[16:19], %[xc0] offset:0\n" \
  "ds_read_b128 v[20:23], %[xc1] offset:0\n" \
  "ds_read_b128 v[24:27], %[xc0] offset:16384\n" \
  "ds_read_b128 v[28:31], %[xc1] offset:16384\n" \
  "ds_read_b64_tr_b8 v[64:65], %[wr0] offset:0\n" \
  "ds_read_b64_tr_b8 v[66:67], %[wr0] offset:1024\n" \
  "ds_read_b64_tr_b8 v[80:81], %[wr0] offset:8192\n" \
  "ds_read_b64_tr_b8 v[82:83], %[wr0] offset:9216\n" \
  "ds_read_b64_tr_b8 v[96:97], %[wr0] offset:16384\n" \
  "ds_read_b64_tr_b8 v[98:99], %[wr0] offset:17408\n" \
  "ds_read_b64_tr_b8 v[112:113], %[wr0] offset:24576\n" \
  "ds_read_b64_tr_b8 v[114:115], %[wr0] offset:25600\n" \
  "ds_read_b64_tr_b8 v[68:69], %[wr1] offset:0\n" \
  "ds_read_b64_tr_b8 v[70:71], %[wr1] offset:1024\n" \
  "ds_read_b64_tr_b8 v[84:85], %[wr1] offset:8192\n" \
  "ds_read_b64_tr_b8 v[86:87], %[wr1] offset:9216\n" \
  "ds_read_b64_tr_b8 v[100:101], %[wr1] offset:16384\n" \
  "ds_read_b64_tr_b8 v[102:103], %[wr1] offset:17408\n" \
  "ds_read_b64_tr_b8 v[116:117], %[wr1] offset:24576\n" \
  "ds_read_b64_tr_b8 v[118:119], %[wr1] offset:25600\n" \
  "ds_read_b64_tr_b8 v[72:73], %[wr2] offset:0\n" \
  "ds_read_b64_tr_b8 v[74:75], %[wr2] offset:1024\n" \
  "ds_read_b64_tr_b8 v[88:89], %[wr2] offset:8192\n" \
  "ds_read_b64_tr_b8 v[90:91], %[wr2] offset:9216\n" \
  "ds_read_b64_tr_b8 v[104:105], %[wr2] offset:16384\n" \
  "ds_read_b64_tr_b8 v[106:107], %[wr2] offset:17408\n" \
  "ds_read_b64_tr_b8 v[120:121], %[wr2] offset:24576\n" \
  "ds_read_b64_tr_b8 v[122:123], %[wr2] offset:25600\n" \
  "ds_read_b64_tr_b8 v[76:77], %[wr3] offset:0\n" \
  "ds_read_b64_tr_b8 v[78:79], %[wr3] offset:1024\n" \
  "ds_read_b64_tr_b8 v[92:93], %[wr3] offset:8192\n" \
  "ds_read_b64_tr_b8 v[94:95], %[wr3] offset:9216\n" \
  "s_mov_b32 %[sI], 7\n" \
  "Lloop_%=:\n" \
  "s_waitcnt lgkmcnt(9)\n" \
  "v_mfma_i32_16x16x64_i8 v[128:131], v[64:67], v[16:19], v[128:131]\n" \
  "ds_read_b64_tr_b8 v[108:109], %[wr3] offset:16384\n" \
  "v_mfma_i32_16x16x64_i8 v[128:131], v[80:83], v[20:23], v[128:131]\n" \
  "ds_read_b64_tr_b8 v[110:111], %[wr3] offset:17408\n" \
  "v_mfma_i32_16x16x64_i8 v[128:131], v[96:99], v[24:27], v[128:131]\n" \
  "ds_read_b64_tr_b8 v[124:125], %[wr3] offset:24576\n" \
  "v_mfma_i32_16x16x64_i8 v[128:131], v[112:115], v[28:31], v[128:131]\n" \
  "ds_read_b64_tr_b8 v[126:127], %[wr3] offset:25600\n" \
  "v_mfma_i32_16x16x64_i8 v[132:135], v[68:71], v[16:19], v[132:135]\n" \
  "ds_read_b128 v[32:35], %[xc0] offset:2048\n" \
  "v_mfma_i32_16x16x64_i8 v[132:135], v[84:87], v[20:23], v[132:135]\n" \
  "ds_read_b128 v[36:39], %[xc1] offset:2048\n" \
  "v_mfma_i32_16x16x64_i8 v[132:135], v[100:103], v[24:27], v[132:135]\n" \
  "ds_read_b128 v[40:43], %[xc0] offset:18432\n" \
  "v_mfma_i32_16x16x64_i8 v[132:135], v[116:119], v[28:31], v[132:135]\n" \
  "ds_read_b128 v[44:47], %[xc1] offset:18432\n" \
  "v_mfma_i32_16x16x64_i8 v[136:139], v[72:75], v[16:19], v[136:139]\n" \
  "ds_read_b128 v[48:51], %[xc0] offset:4096\n" \
  "s_waitcnt lgkmcnt(9)\n" \
  "v_mfma_i32_16x16x64_i8 v[136:139], v[88:91], v[20:23], v[136:139]\n" \
  "ds_read_b128 v[52:55], %[xc1] offset:4096\n" \
  "v_mfma_i32_16x16x64_i8 v[136:139], v[104:107], v[24:27], v[136:139]\n" \
  "ds_read_b128 v[56:59], %[xc0] offset:20480\n" \
  "v_mfma_i32_16x16x64_i8 v[136:139], v[120:123], v[28:31], v[136:139]\n" \
  "ds_read_b128 v[60:63], %[xc1] offset:20480\n" \
  "v_mfma_i32_16x16x64_i8 v[140:143], v[76:79], v[16:19], v[140:143]\n" \
  "v_mfma_i32_16x16x64_i8 v[140:143], v[92:95], v[20:23], v[140:143]\n" \
  "s_waitcnt lgkmcnt(7)\n" \
  "v_mfma_i32_16x16x64_i8 v[140:143], v[108:111], v[24:27], v[140:143]\n" \
  "v_mfma_i32_16x16x64_i8 v[140:143], v[124:127], v[28:31], v[140:143]\n" \
  "v_mfma_i32_16x16x64_i8 v[144:147], v[64:67], v[32:35], v[144:147]\n" \
  "s_waitcnt lgkmcnt(4)\n" \
  "v_mfma_i32_16x16x64_i8 v[144:147], v[80:83], v[36:39], v[144:147]\n" \
  "v_mfma_i32_16x16x64_i8 v[144:147], v[96:99], v[40:43], v[144:147]\n" \
  "v_mfma_i32_16x16x64_i8 v[144:147], v[112:115], v[44:47], v[144:147]\n" \
  "v_mfma_i32_16x16x64_i8 v[148:151], v[116:119], v[44:47], v[148:151]\n" \
  "ds_read_b128 v[16:19], %[xc0] offset:6144\n" \
  "v_mfma_i32_16x16x64_i8 v[148:151], v[100:103], v[40:43], v[148:151]\n" \
  "ds_read_b128 v[20:23], %[xc1] offset:6144\n" \
  "v_mfma_i32_16x16x64_i8 v[148:151], v[84:87], v[36:39], v[148:151]\n" \
  "ds_read_b128 v[24:27], %[xc0] offset:22528\n" \
  "v_mfma_i32_16x16x64_i8 v[148:151], v[68:71], v[32:35], v[148:151]\n" \
  "ds_read_b128 v[28:31], %[xc1] offset:22528\n" \
  "v_mfma_i32_16x16x64_i8 v[152:155], v[72:75], v[32:35], v[152:155]\n" \
  "v_mfma_i32_16x16x64_i8 v[152:155], v[88:91], v[36:39], v[152:155]\n" \
  "v_mfma_i32_16x16x64_i8 v[152:155], v[104:107], v[40:43], v[152:155]\n" \
  "s_add_u32 %[sA], %[sA], 0x8000\n" \
  "s_cmp_eq_u32 %[sA], 0x18000\n" \
  "v_mfma_i32_16x16x64_i8 v[152:155], v[120:123], v[44:47], v[152:155]\n" \
  "s_cselect_b32 %[sA], 0, %[sA]\n" \
  "v_mfma_i32_16x16x64_i8 v[156:159], v[124:127], v[44:47], v[156:159]\n" \
  "v_add_u32 %[xc0], %[sA], %[xa0]\n" \
  "v_xor_b32 %[xc1], 64, %[xc0]\n" \
  "v_mfma_i32_16x16x64_i8 v[156:159], v[108:111], v[40:43], v[156:159]\n" \
  "v_mfma_i32_16x16x64_i8 v[156:159], v[92:95], v[36:39], v[156:159]\n" \
  "v_mfma_i32_16x16x64_i8 v[156:159], v[76:79], v[32:35], v[156:159]\n" \
  "s_waitcnt lgkmcnt(1)\n" \
  "v_mfma_i32_16x16x64_i8 v[160:163], v[64:67], v[48:51], v[160:163]\n" \
  "v_mfma_i32_16x16x64_i8 v[160:163], v[80:83], v[52:55], v[160:163]\n" \
  "v_mfma_i32_16x16x64_i8 v[160:163], v[96:99], v[56:59], v[160:163]\n" \
  "v_mfma_i32_16x16x64_i8 v[160:163], v[112:115], v[60:63], v[160:163]\n" \
  "v_mfma_i32_16x16x64_i8 v[164:167], v[116:119], v[60:63], v[164:167]\n" \
  "v_mfma_i32_16x16x64_i8 v[164:167], v[100:103], v[56:59], v[164:167]\n" \
  "v_mfma_i32_16x16x64_i8 v[164:167], v[84:87], v[52:55], v[164:167]\n" \
  "v_mfma_i32_16x16x64_i8 v[164:167], v[68:71], v[48:51], v[164:167]\n" \
  "v_mfma_i32_16x16x64_i8 v[168:171], v[72:75], v[48:51], v[168:171]\n" \
  "v_mfma_i32_16x16x64_i8 v[168:171], v[88:91], v[52:55], v[168:171]\n" \
  "v_mfma_i32_16x16x64_i8 v[168:171], v[104:107], v[56:59], v[168:171]\n" \
  "v_mfma_i32_16x16x64_i8 v[168:171], v[120:123], v[60:63], v[168:171]\n" \
  "s_waitcnt lgkmcnt(0)\n" \
  "s_barrier\n" \
  "v_mfma_i32_16x16x64_i8 v[172:175], v[124:127], v[60:63], v[172:175]\n" \
  "ds_read_b128 v[32:35], %[xc0] offset:0\n" \
  "v_mfma_i32_16x16x64_i8 v[172:175], v[108:111], v[56:59], v[172:175]\n" \
  "ds_read_b128 v[36:39], %[xc1] offset:0\n" \
  "v_mfma_i32_16x16x64_i8 v[172:175], v[92:95], v[52:55], v[172:175]\n" \
  "ds_read_b128 v[40:43], %[xc0] offset:16384\n" \
  "v_mfma_i32_16x16x64_i8 v[172:175], v[76:79], v[48:51], v[172:175]\n" \
  "ds_read_b128 v[44:47], %[xc1] offset:16384\n" \
  "v_mfma_i32_16x16x64_i8 v[176:179], v[64:67], v[16:19], v[176:179]\n" \
  "v_mfma_i32_16x16x64_i8 v[176:179], v[80:83], v[20:23], v[176:179]\n" \
  "v_mfma_i32_16x16x64_i8 v[176:179], v[96:99], v[24:27], v[176:179]\n" \
  "v_mfma_i32_16x16x64_i8 v[176:179], v[112:115], v[28:31], v[176:179]\n" \
  "v_mfma_i32_16x16x64_i8 v[180:183], v[116:119], v[28:31], v[180:183]\n" \
  "ds_read_b128 v[48:51], %[xc0] offset:6144\n" \
  "v_mfma_i32_16x16x64_i8 v[180:183], v[100:103], v[24:27], v[180:183]\n" \
  "ds_read_b128 v[52:55], %[xc1] offset:6144\n" \
  "v_mfma_i32_16x16x64_i8 v[180:183], v[84:87], v[20:23], v[180:183]\n" \
  "ds_read_b128 v[56:59], %[xc0] offset:22528\n" \
  "v_mfma_i32_16x16x64_i8 v[180:183], v[68:71], v[16:19], v[180:183]\n" \
  "ds_read_b128 v[60:63], %[xc1] offset:22528\n" \
  "v_mfma_i32_16x16x64_i8 v[184:187], v[72:75], v[16:19], v[184:187]\n" \
  "v_mfma_i32_16x16x64_i8 v[184:187], v[88:91], v[20:23], v[184:187]\n" \
  "v_mfma_i32_16x16x64_i8 v[184:187], v[104:107], v[24:27], v[184:187]\n" \
  "v_mfma_i32_16x16x64_i8 v[184:187], v[120:123], v[28:31], v[184:187]\n" \
  "v_mfma_i32_16x16x64_i8 v[188:191], v[124:127], v[28:31], v[188:191]\n" \
  "v_mfma_i32_16x16x64_i8 v[188:191], v[108:111], v[24:27], v[188:191]\n" \
  "v_mfma_i32_16x16x64_i8 v[188:191], v[92:95], v[20:23], v[188:191]\n" \
  "v_mfma_i32_16x16x64_i8 v[188:191], v[76:79], v[16:19], v[188:191]\n" \
  "s_waitcnt lgkmcnt(1)\n" \
  "v_mfma_i32_16x16x64_i8 v[192:195], v[64:67], v[32:35], v[192:195]\n" \
  "ds_read_b128 v[16:19], %[xc0] offset:2048\n" \
  "v_mfma_i32_16x16x64_i8 v[192:195], v[80:83], v[36:39], v[192:195]\n" \
  "ds_read_b128 v[20:23], %[xc1] offset:2048\n" \
  "v_mfma_i32_16x16x64_i8 v[192:195], v[96:99], v[40:43], v[192:195]\n" \
  "ds_read_b128 v[24:27], %[xc0] offset:18432\n" \
  "v_mfma_i32_16x16x64_i8 v[192:195], v[112:115], v[44:47], v[192:195]\n" \
  "ds_read_b128 v[28:31], %[xc1] offset:18432\n" \
  "v_mfma_i32_16x16x64_i8 v[196:199], v[116:119], v[44:47], v[196:199]\n" \
  "v_mfma_i32_16x16x64_i8 v[196:199], v[100:103], v[40:43], v[196:199]\n" \
  "v_mfma_i32_16x16x64_i8 v[196:199], v[84:87], v[36:39], v[196:199]\n" \
  "v_mfma_i32_16x16x64_i8 v[196:199], v[68:71], v[32:35], v[196:199]\n" \
  "v_mfma_i32_16x16x64_i8 v[200:203], v[72:75], v[32:35], v[200:203]\n" \
  "v_mfma_i32_16x16x64_i8 v[200:203], v[88:91], v[36:39], v[200:203]\n" \
  "v_mfma_i32_16x16x64_i8 v[200:203], v[104:107], v[40:43], v[200:203]\n" \
  "v_mfma_i32_16x16x64_i8 v[200:203], v[120:123], v[44:47], v[200:203]\n" \
  "v_mfma_i32_16x16x64_i8 v[204:207], v[124:127], v[44:47], v[204:207]\n" \
  "v_mfma_i32_16x16x64_i8 v[204:207], v[108:111], v[40:43], v[204:207]\n" \
  "v_mfma_i32_16x16x64_i8 v[204:207], v[92:95], v[36:39], v[204:207]\n" \
  "v_mfma_i32_16x16x64_i8 v[204:207], v[76:79], v[32:35], v[204:207]\n" \
  "s_waitcnt lgkmcnt(0)\n" \
  "v_mfma_i32_16x16x64_i8 v[208:211], v[64:67], v[16:19], v[208:211]\n" \
  "ds_read_b128 v[32:35], %[xc0] offset:4096\n" \
  "v_mfma_i32_16x16x64_i8 v[208:211], v[80:83], v[20:23], v[208:211]\n" \
  "ds_read_b128 v[36:39], %[xc1] offset:4096\n" \
  "v_mfma_i32_16x16x64_i8 v[208:211], v[96:99], v[24:27], v[208:211]\n" \
  "ds_read_b128 v[40:43], %[xc0] offset:20480\n" \
  "v_mfma_i32_16x16x64_i8 v[208:211], v[112:115], v[28:31], v[208:211]\n" \
  "ds_read_b128 v[44:47], %[xc1] offset:20480\n" \
  "v_mfma_i32_16x16x64_i8 v[212:215], v[116:119], v[28:31], v[212:215]\n" \
  "v_mfma_i32_16x16x64_i8 v[212:215], v[100:103], v[24:27], v[212:215]\n" \
  "v_mfma_i32_16x16x64_i8 v[212:215], v[84:87], v[20:23], v[212:215]\n" \
  "s_add_u32 %[sA], %[sA], 0x8000\n" \
  "s_cmp_eq_u32 %[sA], 0x18000\n" \
  "v_mfma_i32_16x16x64_i8 v[212:215], v[68:71], v[16:19], v[212:215]\n" \
  "s_cselect_b32 %[sA], 0, %[sA]\n" \
  "v_mfma_i32_16x16x64_i8 v[216:219], v[72:75], v[16:19], v[216:219]\n" \
  "v_add_u32 %[xc0], %[sA], %[xa0]\n" \
  "v_xor_b32 %[xc1], 64, %[xc0]\n" \
  "v_mfma_i32_16x16x64_i8 v[216:219], v[88:91], v[20:23], v[216:219]\n" \
  "v_mfma_i32_16x16x64_i8 v[216:219], v[104:107], v[24:27], v[216:219]\n" \
  "v_mfma_i32_16x16x64_i8 v[216:219], v[120:123], v[28:31], v[216:219]\n" \
  "v_mfma_i32_16x16x64_i8 v[220:223], v[124:127], v[28:31], v[220:223]\n" \
  "v_mfma_i32_16x16x64_i8 v[220:223], v[108:111], v[24:27], v[220:223]\n" \
  "v_mfma_i32_16x16x64_i8 v[220:223], v[92:95], v[20:23], v[220:223]\n" \
  "v_mfma_i32_16x16x64_i8 v[220:223], v[76:79], v[16:19], v[220:223]\n" \
  "s_waitcnt lgkmcnt(0)\n" \
  "s_barrier\n" \
  "v_mfma_i32_16x16x64_i8 v[224:227], v[64:67], v[32:35], v[224:227]\n" \
  "ds_read_b128 v[16:19], %[xc0] offset:0\n" \
  "v_mfma_i32_16x16x64_i8 v[224:227], v[80:83], v[36:39], v[224:227]\n" \
  "ds_read_b128 v[20:23], %[xc1] offset:0\n" \
  "v_mfma_i32_16x16x64_i8 v[224:227], v[96:99], v[40:43], v[224:227]\n" \
  "ds_read_b128 v[24:27], %[xc0] offset:16384\n" \
  "v_mfma_i32_16x16x64_i8 v[224:227], v[112:115], v[44:47], v[224:227]\n" \
  "ds_read_b128 v[28:31], %[xc1] offset:16384\n" \
  "v_mfma_i32_16x16x64_i8 v[240:243], v[64:67], v[48:51], v[240:243]\n" \
  "ds_read_b64_tr_b8 v[64:65], %[wr0] offset:32768\n" \
  "v_mfma_i32_16x16x64_i8 v[240:243], v[80:83], v[52:55], v[240:243]\n" \
  "ds_read_b64_tr_b8 v[66:67], %[wr0] offset:33792\n" \
  "v_mfma_i32_16x16x64_i8 v[240:243], v[96:99], v[56:59], v[240:243]\n" \
  "ds_read_b64_tr_b8 v[80:81], %[wr0] offset:40960\n" \
  "v_mfma_i32_16x16x64_i8 v[240:243], v[112:115], v[60:63], v[240:243]\n" \
  "ds_read_b64_tr_b8 v[82:83], %[wr0] offset:41984\n" \
  "v_mfma_i32_16x16x64_i8 v[228:231], v[68:71], v[32:35], v[228:231]\n" \
  "ds_read_b64_tr_b8 v[96:97], %[wr0] offset:49152\n" \
  "v_mfma_i32_16x16x64_i8 v[228:231], v[84:87], v[36:39], v[228:231]\n" \
  "ds_read_b64_tr_b8 v[98:99], %[wr0] offset:50176\n" \
  "v_mfma_i32_16x16x64_i8 v[228:231], v[100:103], v[40:43], v[228:231]\n" \
  "ds_read_b64_tr_b8 v[112:113], %[wr0] offset:57344\n" \
  "v_mfma_i32_16x16x64_i8 v[228:231], v[116:119], v[44:47], v[228:231]\n" \
  "ds_read_b64_tr_b8 v[114:115], %[wr0] offset:58368\n" \
  "v_mfma_i32_16x16x64_i8 v[244:247], v[68:71], v[48:51], v[244:247]\n" \
  "ds_read_b64_tr_b8 v[68:69], %[wr1] offset:32768\n" \
  "v_mfma_i32_16x16x64_i8 v[244:247], v[84:87], v[52:55], v[244:247]\n" \
  "ds_read_b64_tr_b8 v[70:71], %[wr1] offset:33792\n" \
  "v_mfma_i32_16x16x64_i8 v[244:247], v[100:103], v[56:59], v[244:247]\n" \
  "ds_read_b64_tr_b8 v[84:85], %[wr1] offset:40960\n" \
  "v_mfma_i32_16x16x64_i8 v[244:247], v[116:119], v[60:63], v[244:247]\n" \
  "ds_read_b64_tr_b8 v[86:87], %[wr1] offset:41984\n" \
  "v_mfma_i32_16x16x64_i8 v[232:235], v[72:75], v[32:35], v[232:235]\n" \
  "ds_read_b64_tr_b8 v[100:101], %[wr1] offset:49152\n" \
  "v_mfma_i32_16x16x64_i8 v[232:235], v[88:91], v[36:39], v[232:235]\n" \
  "ds_read_b64_tr_b8 v[102:103], %[wr1] offset:50176\n" \
  "v_mfma_i32_16x16x64_i8 v[232:235], v[104:107], v[40:43], v[232:235]\n" \
  "ds_read_b64_tr_b8 v[116:117], %[wr1] offset:57344\n" \
  "v_mfma_i32_16x16x64_i8 v[232:235], v[120:123], v[44:47], v[232:235]\n" \
  "ds_read_b64_tr_b8 v[118:119], %[wr1] offset:58368\n" \
  "v_mfma_i32_16x16x64_i8 v[248:251], v[72:75], v[48:51], v[248:251]\n" \
  "ds_read_b64_tr_b8 v[72:73], %[wr2] offset:32768\n" \
  "v_mfma_i32_16x16x64_i8 v[248:251], v[88:91], v[52:55], v[248:251]\n" \
  "ds_read_b64_tr_b8 v[74:75], %[wr2] offset:33792\n" \
  "v_mfma_i32_16x16x64_i8 v[248:251], v[104:107], v[56:59], v[248:251]\n" \
  "ds_read_b64_tr_b8 v[88:89], %[wr2] offset:40960\n" \
  "v_mfma_i32_16x16x64_i8 v[248:251], v[120:123], v[60:63], v[248:251]\n" \
  "ds_read_b64_tr_b8 v[90:91], %[wr2] offset:41984\n" \
  "v_mfma_i32_16x16x64_i8 v[236:239], v[76:79], v[32:35], v[236:239]\n" \
  "ds_read_b64_tr_b8 v[104:105], %[wr2] offset:49152\n" \
  "v_mfma_i32_16x16x64_i8 v[236:239], v[92:95], v[36:39], v[236:239]\n" \
  "ds_read_b64_tr_b8 v[106:107], %[wr2] offset:50176\n" \
  "v_mfma_i32_16x16x64_i8 v[236:239], v[108:111], v[40:43], v[236:239]\n" \
  "ds_read_b64_tr_b8 v[120:121], %[wr2] offset:57344\n" \
  "v_mfma_i32_16x16x64_i8 v[236:239], v[124:127], v[44:47], v[236:239]\n" \
  "ds_read_b64_tr_b8 v[122:123], %[wr2] offset:58368\n" \
  "v_mfma_i32_16x16x64_i8 v[252:255], v[76:79], v[48:51], v[252:255]\n" \
  "ds_read_b64_tr_b8 v[76:77], %[wr3] offset:32768\n" \
  "v_mfma_i32_16x16x64_i8 v[252:255], v[92:95], v[52:55], v[252:255]\n" \
  "ds_read_b64_tr_b8 v[78:79], %[wr3] offset:33792\n" \
  "v_mfma_i32_16x16x64_i8 v[252:255], v[108:111], v[56:59], v[252:255]\n" \
  "ds_read_b64_tr_b8 v[92:93], %[wr3] offset:40960\n" \
  "v_mfma_i32_16x16x64_i8 v[252:255], v[124:127], v[60:63], v[252:255]\n" \
  "ds_read_b64_tr_b8 v[94:95], %[wr3] offset:41984\n" \
  "s_waitcnt lgkmcnt(9)\n" \
  "v_mfma_i32_16x16x64_i8 v[128:131], v[64:67], v[16:19], v[128:131]\n" \
  "ds_read_b64_tr_b8 v[108:109], %[wr3] offset:49152\n" \
  "v_mfma_i32_16x16x64_i8 v[128:131], v[80:83], v[20:23], v[128:131]\n" \
  "ds_read_b64_tr_b8 v[110:111], %[wr3] offset:50176\n" \
  "v_mfma_i32_16x16x64_i8 v[128:131], v[96:99], v[24:27], v[128:131]\n" \
  "ds_read_b64_tr_b8 v[124:125], %[wr3] offset:57344\n" \
  "v_mfma_i32_16x16x64_i8 v[128:131], v[112:115], v[28:31], v[128:131]\n" \
  "ds_read_b64_tr_b8 v[126:127], %[wr3] offset:58368\n" \
  "v_mfma_i32_16x16x64_i8 v[132:135], v[68:71], v[16:19], v[132:135]\n" \
  "ds_read_b128 v[32:35], %[xc0] offset:2048\n" \
  "v_mfma_i32_16x16x64_i8 v[132:135], v[84:87], v[20:23], v[132:135]\n" \
  "ds_read_b128 v[36:39], %[xc1] offset:2048\n" \
  "v_mfma_i32_16x16x64_i8 v[132:135], v[100:103], v[24:27], v[132:135]\n" \
  "ds_read_b128 v[40:43], %[xc0] offset:18432\n" \
  "v_mfma_i32_16x16x64_i8 v[132:135], v[116:119], v[28:31], v[132:135]\n" \
  "ds_read_b128 v[44:47], %[xc1] offset:18432\n" \
  "v_mfma_i32_16x16x64_i8 v[136:139], v[72:75], v[16:19], v[136:139]\n" \
  "ds_read_b128 v[48:51], %[xc0] offset:4096\n" \
  "s_waitcnt lgkmcnt(9)\n" \
  "v_mfma_i32_16x16x64_i8 v[136:139], v[88:91], v[20:23], v[136:139]\n" \
  "ds_read_b128 v[52:55], %[xc1] offset:4096\n" \
  "v_mfma_i32_16x16x64_i8 v[136:139], v[104:107], v[24:27], v[136:139]\n" \
  "ds_read_b128 v[56:59], %[xc0] offset:20480\n" \
  "v_mfma_i32_16x16x64_i8 v[136:139], v[120:123], v[28:31], v[136:139]\n" \
  "ds_read_b128 v[60:63], %[xc1] offset:20480\n" \
  "v_mfma_i32_16x16x64_i8 v[140:143], v[76:79], v[16:19], v[140:143]\n" \
  "v_mfma_i32_16x16x64_i8 v[140:143], v[92:95], v[20:23], v[140:143]\n" \
  "s_waitcnt lgkmcnt(7)\n" \
  "v_mfma_i32_16x16x64_i8 v[140:143], v[108:111], v[24:27], v[140:143]\n" \
  "v_mfma_i32_16x16x64_i8 v[140:143], v[124:127], v[28:31], v[140:143]\n" \
  "v_mfma_i32_16x16x64_i8 v[144:147], v[64:67], v[32:35], v[144:147]\n" \
  "s_waitcnt lgkmcnt(4)\n" \
  "v_mfma_i32_16x16x64_i8 v[144:147], v[80:83], v[36:39], v[144:147]\n" \
  "v_mfma_i32_16x16x64_i8 v[144:147], v[96:99], v[40:43], v[144:147]\n" \
  "v_mfma_i32_16x16x64_i8 v[144:147], v[112:115], v[44:47], v[144:147]\n" \
  "v_mfma_i32_16x16x64_i8 v[148:151], v[116:119], v[44:47], v[148:151]\n" \
  "ds_read_b128 v[16:19], %[xc0] offset:6144\n" \
  "v_mfma_i32_16x16x64_i8 v[148:151], v[100:103], v[40:43], v[148:151]\n" \
  "ds_read_b128 v[20:23], %[xc1] offset:6144\n" \
  "v_mfma_i32_16x16x64_i8 v[148:151], v[84:87], v[36:39], v[148:151]\n" \
  "ds_read_b128 v[24:27], %[xc0] offset:22528\n" \
  "v_mfma_i32_16x16x64_i8 v[148:151], v[68:71], v[32:35], v[148:151]\n" \
  "ds_read_b128 v[28:31], %[xc1] offset:22528\n" \
  "v_mfma_i32_16x16x64_i8 v[152:155], v[72:75], v[32:35], v[152:155]\n" \
  "v_mfma_i32_16x16x64_i8 v[152:155], v[88:91], v[36:39], v[152:155]\n" \
  "v_mfma_i32_16x16x64_i8 v[152:155], v[104:107], v[40:43], v[152:155]\n" \
  "s_add_u32 %[sA], %[sA], 0x8000\n" \
  "s_cmp_eq_u32 %[sA], 0x18000\n" \
  "v_mfma_i32_16x16x64_i8 v[152:155], v[120:123], v[44:47], v[152:155]\n" \
  "s_cselect_b32 %[sA], 0, %[sA]\n" \
  "v_mfma_i32_16x16x64_i8 v[156:159], v[124:127], v[44:47], v[156:159]\n" \
  "v_add_u32 %[xc0], %[sA], %[xa0]\n" \
  "v_xor_b32 %[xc1], 64, %[xc0]\n" \
  "v_mfma_i32_16x16x64_i8 v[156:159], v[108:111], v[40:43], v[156:159]\n" \
  "v_mfma_i32_16x16x64_i8 v[156:159], v[92:95], v[36:39], v[156:159]\n" \
  "v_mfma_i32_16x16x64_i8 v[156:159], v[76:79], v[32:35], v[156:159]\n" \
  "s_waitcnt lgkmcnt(1)\n" \
  "v_mfma_i32_16x16x64_i8 v[160:163], v[64:67], v[48:51], v[160:163]\n" \
  "v_mfma_i32_16x16x64_i8 v[160:163], v[80:83], v[52:55], v[160:163]\n" \
  "v_mfma_i32_16x16x64_i8 v[160:163], v[96:99], v[56:59], v[160:163]\n" \
  "v_mfma_i32_16x16x64_i8 v[160:163], v[112:115], v[60:63], v[160:163]\n" \
  "v_mfma_i32_16x16x64_i8 v[164:167], v[116:119], v[60:63], v[164:167]\n" \
  "v_mfma_i32_16x16x64_i8 v[164:167], v[100:103], v[56:59], v[164:167]\n" \
  "v_mfma_i32_16x16x64_i8 v[164:167], v[84:87], v[52:55], v[164:167]\n" \
  "v_mfma_i32_16x16x64_i8 v[164:167], v[68:71], v[48:51], v[164:167]\n" \
  "v_mfma_i32_16x16x64_i8 v[168:171], v[72:75], v[48:51], v[168:171]\n" \
  "v_mfma_i32_16x16x64_i8 v[168:171], v[88:91], v[52:55], v[168:171]\n" \
  "v_mfma_i32_16x16x64_i8 v[168:171], v[104:107], v[56:59], v[168:171]\n" \
  "v_mfma_i32_16x16x64_i8 v[168:171], v[120:123], v[60:63], v[168:171]\n" \
  "s_waitcnt lgkmcnt(0)\n" \
  "s_barrier\n" \
  "v_mfma_i32_16x16x64_i8 v[172:175], v[124:127], v[60:63], v[172:175]\n" \
  "ds_read_b128 v[32:35], %[xc0] offset:0\n" \
  "v_mfma_i32_16x16x64_i8 v[172:175], v[108:111], v[56:59], v[172:175]\n" \
  "ds_read_b128 v[36:39], %[xc1] offset:0\n" \
  "v_mfma_i32_16x16x64_i8 v[172:175], v[92:95], v[52:55], v[172:175]\n" \
  "ds_read_b128 v[40:43], %[xc0] offset:16384\n" \
  "v_mfma_i32_16x16x64_i8 v[172:175], v[76:79], v[48:51], v[172:175]\n" \
  "ds_read_b128 v[44:47], %[xc1] offset:16384\n" \
  "v_mfma_i32_16x16x64_i8 v[176:179], v[64:67], v[16:19], v[176:179]\n" \
  "v_mfma_i32_16x16x64_i8 v[176:179], v[80:83], v[20:23], v[176:179]\n" \
  "v_mfma_i32_16x16x64_i8 v[176:179], v[96:99], v[24:27], v[176:179]\n" \
  "v_mfma_i32_16x16x64_i8 v[176:179], v[112:115], v[28:31], v[176:179]\n" \
  "v_mfma_i32_16x16x64_i8 v[180:183], v[116:119], v[28:31], v[180:183]\n" \
  "ds_read_b128 v[48:51], %[xc0] offset:6144\n" \
  "v_mfma_i32_16x16x64_i8 v[180:183], v[100:103], v[24:27], v[180:183]\n" \
  "ds_read_b128 v[52:55], %[xc1] offset:6144\n" \
  "v_mfma_i32_16x16x64_i8 v[180:183], v[84:87], v[20:23], v[180:183]\n" \
  "ds_read_b128 v[56:59], %[xc0] offset:22528\n" \
  "v_mfma_i32_16x16x64_i8 v[180:183], v[68:71], v[16:19], v[180:183]\n" \
  "ds_read_b128 v[60:63], %[xc1] offset:22528\n" \
  "v_mfma_i32_16x16x64_i8 v[184:187], v[72:75], v[16:19], v[184:187]\n" \
  "v_mfma_i32_16x16x64_i8 v[184:187], v[88:91], v[20:23], v[184:187]\n" \
  "v_mfma_i32_16x16x64_i8 v[184:187], v[104:107], v[24:27], v[184:187]\n" \
  "v_mfma_i32_16x16x64_i8 v[184:187], v[120:123], v[28:31], v[184:187]\n" \
  "v_mfma_i32_16x16x64_i8 v[188:191], v[124:127], v[28:31], v[188:191]\n" \
  "v_mfma_i32_16x16x64_i8 v[188:191], v[108:111], v[24:27], v[188:191]\n" \
  "v_mfma_i32_16x16x64_i8 v[188:191], v[92:95], v[20:23], v[188:191]\n" \
  "v_mfma_i32_16x16x64_i8 v[188:191], v[76:79], v[16:19], v[188:191]\n" \
  "s_waitcnt lgkmcnt(1)\n" \
  "v_mfma_i32_16x16x64_i8 v[192:195], v[64:67], v[32:35], v[192:195]\n" \
  "ds_read_b128 v[16:19], %[xc0] offset:2048\n" \
  "v_mfma_i32_16x16x64_i8 v[192:195], v[80:83], v[36:39], v[192:195]\n" \
  "ds_read_b128 v[20:23], %[xc1] offset:2048\n" \
  "v_mfma_i32_16x16x64_i8 v[192:195], v[96:99], v[40:43], v[192:195]\n" \
  "ds_read_b128 v[24:27], %[xc0] offset:18432\n" \
  "v_mfma_i32_16x16x64_i8 v[192:195], v[112:115], v[44:47], v[192:195]\n" \
  "ds_read_b128 v[28:31], %[xc1] offset:18432\n" \
  "v_mfma_i32_16x16x64_i8 v[196:199], v[116:119], v[44:47], v[196:199]\n" \
  "v_mfma_i32_16x16x64_i8 v[196:199], v[100:103], v[40:43], v[196:199]\n" \
  "v_mfma_i32_16x16x64_i8 v[196:199], v[84:87], v[36:39], v[196:199]\n" \
  "v_mfma_i32_16x16x64_i8 v[196:199], v[68:71], v[32:35], v[196:199]\n" \
  "v_mfma_i32_16x16x64_i8 v[200:203], v[72:75], v[32:35], v[200:203]\n" \
  "v_mfma_i32_16x16x64_i8 v[200:203], v[88:91], v[36:39], v[200:203]\n" \
  "v_mfma_i32_16x16x64_i8 v[200:203], v[104:107], v[40:43], v[200:203]\n" \
  "v_mfma_i32_16x16x64_i8 v[200:203], v[120:123], v[44:47], v[200:203]\n" \
  "v_mfma_i32_16x16x64_i8 v[204:207], v[124:127], v[44:47], v[204:207]\n" \
  "v_mfma_i32_16x16x64_i8 v[204:207], v[108:111], v[40:43], v[204:207]\n" \
  "v_mfma_i32_16x16x64_i8 v[204:207], v[92:95], v[36:39], v[204:207]\n" \
  "v_mfma_i32_16x16x64_i8 v[204:207], v[76:79], v[32:35], v[204:207]\n" \
  "s_waitcnt lgkmcnt(0)\n" \
  "v_mfma_i32_16x16x64_i8 v[208:211], v[64:67], v[16:19], v[208:211]\n" \
  "ds_read_b128 v[32:35], %[xc0] offset:4096\n" \
  "v_mfma_i32_16x16x64_i8 v[208:211], v[80:83], v[20:23], v[208:211]\n" \
  "ds_read_b128 v[36:39], %[xc1] offset:4096\n" \
  "v_mfma_i32_16x16x64_i8 v[208:211], v[96:99], v[24:27], v[208:211]\n" \
  "ds_read_b128 v[40:43], %[xc0] offset:20480\n" \
  "v_mfma_i32_16x16x64_i8 v[208:211], v[112:115], v[28:31], v[208:211]\n" \
  "ds_read_b128 v[44:47], %[xc1] offset:20480\n" \
  "v_mfma_i32_16x16x64_i8 v[212:215], v[116:119], v[28:31], v[212:215]\n" \
  "v_mfma_i32_16x16x64_i8 v[212:215], v[100:103], v[24:27], v[212:215]\n" \
  "v_mfma_i32_16x16x64_i8 v[212:215], v[84:87], v[20:23], v[212:215]\n" \
  "s_add_u32 %[sA], %[sA], 0x8000\n" \
  "s_cmp_eq_u32 %[sA], 0x18000\n" \
  "v_mfma_i32_16x16x64_i8 v[212:215], v[68:71], v[16:19], v[212:215]\n" \
  "s_cselect_b32 %[sA], 0, %[sA]\n" \
  "v_mfma_i32_16x16x64_i8 v[216:219], v[72:75], v[16:19], v[216:219]\n" \
  "v_add_u32 %[xc0], %[sA], %[xa0]\n" \
  "v_xor_b32 %[xc1], 64, %[xc0]\n" \
  "v_mfma_i32_16x16x64_i8 v[216:219], v[88:91], v[20:23], v[216:219]\n" \
  "v_mfma_i32_16x16x64_i8 v[216:219], v[104:107], v[24:27], v[216:219]\n" \
  "v_mfma_i32_16x16x64_i8 v[216:219], v[120:123], v[28:31], v[216:219]\n" \
  "v_mfma_i32_16x16x64_i8 v[220:223], v[124:127], v[28:31], v[220:223]\n" \
  "v_mfma_i32_16x16x64_i8 v[220:223], v[108:111], v[24:27], v[220:223]\n" \
  "v_mfma_i32_16x16x64_i8 v[220:223], v[92:95], v[20:23], v[220:223]\n" \
  "v_mfma_i32_16x16x64_i8 v[220:223], v[76:79], v[16:19], v[220:223]\n" \
  "s_waitcnt lgkmcnt(0)\n" \
  "s_barrier\n" \
  "v_mfma_i32_16x16x64_i8 v[224:227], v[64:67], v[32:35], v[224:227]\n" \
  "ds_read_b128 v[16:19], %[xc0] offset:0\n" \
  "v_mfma_i32_16x16x64_i8 v[224:227], v[80:83], v[36:39], v[224:227]\n" \
  "ds_read_b128 v[20:23], %[xc1] offset:0\n" \
  "v_mfma_i32_16x16x64_i8 v[224:227], v[96:99], v[40:43], v[224:227]\n" \
  "ds_read_b128 v[24:27], %[xc0] offset:16384\n" \
  "v_mfma_i32_16x16x64_i8 v[224:227], v[112:115], v[44:47], v[224:227]\n" \
  "ds_read_b128 v[28:31], %[xc1] offset:16384\n" \
  "v_mfma_i32_16x16x64_i8 v[240:243], v[64:67], v[48:51], v[240:243]\n" \
  "ds_read_b64_tr_b8 v[64:65], %[wr0] offset:0\n" \
  "v_mfma_i32_16x16x64_i8 v[240:243], v[80:83], v[52:55], v[240:243]\n" \
  "ds_read_b64_tr_b8 v[66:67], %[wr0] offset:1024\n" \
  "v_mfma_i32_16x16x64_i8 v[240:243], v[96:99], v[56:59], v[240:243]\n" \
  "ds_read_b64_tr_b8 v[80:81], %[wr0] offset:8192\n" \
  "v_mfma_i32_16x16x64_i8 v[240:243], v[112:115], v[60:63], v[240:243]\n" \
  "ds_read_b64_tr_b8 v[82:83], %[wr0] offset:9216\n" \
  "v_mfma_i32_16x16x64_i8 v[228:231], v[68:71], v[32:35], v[228:231]\n" \
  "ds_read_b64_tr_b8 v[96:97], %[wr0] offset:16384\n" \
  "v_mfma_i32_16x16x64_i8 v[228:231], v[84:87], v[36:39], v[228:231]\n" \
  "ds_read_b64_tr_b8 v[98:99], %[wr0] offset:17408\n" \
  "v_mfma_i32_16x16x64_i8 v[228:231], v[100:103], v[40:43], v[228:231]\n" \
  "ds_read_b64_tr_b8 v[112:113], %[wr0] offset:24576\n" \
  "v_mfma_i32_16x16x64_i8 v[228:231], v[116:119], v[44:47], v[228:231]\n" \
  "ds_read_b64_tr_b8 v[114:115], %[wr0] offset:25600\n" \
  "v_mfma_i32_16x16x64_i8 v[244:247], v[68:71], v[48:51], v[244:247]\n" \
  "ds_read_b64_tr_b8 v[68:69], %[wr1] offset:0\n" \
  "v_mfma_i32_16x16x64_i8 v[244:247], v[84:87], v[52:55], v[244:247]\n" \
  "ds_read_b64_tr_b8 v[70:71], %[wr1] offset:1024\n" \
  "v_mfma_i32_16x16x64_i8 v[244:247], v[100:103], v[56:59], v[244:247]\n" \
  "ds_read_b64_tr_b8 v[84:85], %[wr1] offset:8192\n" \
  "v_mfma_i32_16x16x64_i8 v[244:247], v[116:119], v[60:63], v[244:247]\n" \
  "ds_read_b64_tr_b8 v[86:87], %[wr1] offset:9216\n" \
  "v_mfma_i32_16x16x64_i8 v[232:235], v[72:75], v[32:35], v[232:235]\n" \
  "ds_read_b64_tr_b8 v[100:101], %[wr1] offset:16384\n" \
  "v_mfma_i32_16x16x64_i8 v[232:235], v[88:91], v[36:39], v[232:235]\n" \
  "ds_read_b64_tr_b8 v[102:103], %[wr1] offset:17408\n" \
  "v_mfma_i32_16x16x64_i8 v[232:235], v[104:107], v[40:43], v[232:235]\n" \
  "ds_read_b64_tr_b8 v[116:117], %[wr1] offset:24576\n" \
  "v_mfma_i32_16x16x64_i8 v[232:235], v[120:123], v[44:47], v[232:235]\n" \
  "ds_read_b64_tr_b8 v[118:119], %[wr1] offset:25600\n" \
  "v_mfma_i32_16x16x64_i8 v[248:251], v[72:75], v[48:51], v[248:251]\n" \
  "ds_read_b64_tr_b8 v[72:73], %[wr2] offset:0\n" \
  "v_mfma_i32_16x16x64_i8 v[248:251], v[88:91], v[52:55], v[248:251]\n" \
  "ds_read_b64_tr_b8 v[74:75], %[wr2] offset:1024\n" \
  "v_mfma_i32_16x16x64_i8 v[248:251], v[104:107], v[56:59], v[248:251]\n" \
  "ds_read_b64_tr_b8 v[88:89], %[wr2] offset:8192\n" \
  "v_mfma_i32_16x16x64_i8 v[248:251], v[120:123], v[60:63], v[248:251]\n" \
  "ds_read_b64_tr_b8 v[90:91], %[wr2] offset:9216\n" \
  "v_mfma_i32_16x16x64_i8 v[236:239], v[76:79], v[32:35], v[236:239]\n" \
  "ds_read_b64_tr_b8 v[104:105], %[wr2] offset:16384\n" \
  "v_mfma_i32_16x16x64_i8 v[236:239], v[92:95], v[36:39], v[236:239]\n" \
  "ds_read_b64_tr_b8 v[106:107], %[wr2] offset:17408\n" \
  "v_mfma_i32_16x16x64_i8 v[236:239], v[108:111], v[40:43], v[236:239]\n" \
  "ds_read_b64_tr_b8 v[120:121], %[wr2] offset:24576\n" \
  "v_mfma_i32_16x16x64_i8 v[236:239], v[124:127], v[44:47], v[236:239]\n" \
  "ds_read_b64_tr_b8 v[122:123], %[wr2] offset:25600\n" \
  "v_mfma_i32_16x16x64_i8 v[252:255], v[76:79], v[48:51], v[252:255]\n" \
  "ds_read_b64_tr_b8 v[76:77], %[wr3] offset:0\n" \
  "v_mfma_i32_16x16x64_i8 v[252:255], v[92:95], v[52:55], v[252:255]\n" \
  "ds_read_b64_tr_b8 v[78:79], %[wr3] offset:1024\n" \
  "v_mfma_i32_16x16x64_i8 v[252:255], v[108:111], v[56:59], v[252:255]\n" \
  "ds_read_b64_tr_b8 v[92:93], %[wr3] offset:8192\n" \
  "v_mfma_i32_16x16x64_i8 v[252:255], v[124:127], v[60:63], v[252:255]\n" \
  "ds_read_b64_tr_b8 v[94:95], %[wr3] offset:9216\n" \
  "s_sub_u32 %[sI], %[sI], 1\n" \
  "s_cmp_lg_u32 %[sI], 0\n" \
  "s_cbranch_scc1 Lloop_%=\n" \
  "s_waitcnt lgkmcnt(9)\n" \
  "v_mfma_i32_16x16x64_i8 v[128:131], v[64:67], v[16:19], v[128:131]\n" \
  "ds_read_b64_tr_b8 v[108:109], %[wr3] offset:16384\n" \
  "v_mfma_i32_16x16x64_i8 v[128:131], v[80:83], v[20:23], v[128:131]\n" \
  "ds_read_b64_tr_b8 v[110:111], %[wr3] offset:17408\n" \
  "v_mfma_i32_16x16x64_i8 v[128:131], v[96:99], v[24:27], v[128:131]\n" \
  "ds_read_b64_tr_b8 v[124:125], %[wr3] offset:24576\n" \
  "v_mfma_i32_16x16x64_i8 v[128:131], v[112:115], v[28:31], v[128:131]\n" \
  "ds_read_b64_tr_b8 v[126:127], %[wr3] offset:25600\n" \
  "v_mfma_i32_16x16x64_i8 v[132:135], v[68:71], v[16:19], v[132:135]\n" \
  "ds_read_b128 v[32:35], %[xc0] offset:2048\n" \
  "v_mfma_i32_16x16x64_i8 v[132:135], v[84:87], v[20:23], v[132:135]\n" \
  "ds_read_b128 v[36:39], %[xc1] offset:2048\n" \
  "v_mfma_i32_16x16x64_i8 v[132:135], v[100:103], v[24:27], v[132:135]\n" \
  "ds_read_b128 v[40:43], %[xc0] offset:18432\n" \
  "v_mfma_i32_16x16x64_i8 v[132:135], v[116:119], v[28:31], v[132:135]\n" \
  "ds_read_b128 v[44:47], %[xc1] offset:18432\n" \
  "v_mfma_i32_16x16x64_i8 v[136:139], v[72:75], v[16:19], v[136:139]\n" \
  "ds_read_b128 v[48:51], %[xc0] offset:4096\n" \
  "s_waitcnt lgkmcnt(9)\n" \
  "v_mfma_i32_16x16x64_i8 v[136:139], v[88:91], v[20:23], v[136:139]\n" \
  "ds_read_b128 v[52:55], %[xc1] offset:4096\n" \
  "v_mfma_i32_16x16x64_i8 v[136:139], v[104:107], v[24:27], v[136:139]\n" \
  "ds_read_b128 v[56:59], %[xc0] offset:20480\n" \
  "v_mfma_i32_16x16x64_i8 v[136:139], v[120:123], v[28:31], v[136:139]\n" \
  "ds_read_b128 v[60:63], %[xc1] offset:20480\n" \
  "v_mfma_i32_16x16x64_i8 v[140:143], v[76:79], v[16:19], v[140:143]\n" \
  "v_mfma_i32_16x16x64_i8 v[140:143], v[92:95], v[20:23], v[140:143]\n" \
  "s_waitcnt lgkmcnt(7)\n" \
  "v_mfma_i32_16x16x64_i8 v[140:143], v[108:111], v[24:27], v[140:143]\n" \
  "v_mfma_i32_16x16x64_i8 v[140:143], v[124:127], v[28:31], v[140:143]\n" \
  "v_mfma_i32_16x16x64_i8 v[144:147], v[64:67], v[32:35], v[144:147]\n" \
  "ds_read_b128 v[16:19], %[xc0] offset:6144\n" \
  "s_waitcnt lgkmcnt(5)\n" \
  "v_mfma_i32_16x16x64_i8 v[144:147], v[80:83], v[36:39], v[144:147]\n" \
  "ds_read_b128 v[20:23], %[xc1] offset:6144\n" \
  "v_mfma_i32_16x16x64_i8 v[144:147], v[96:99], v[40:43], v[144:147]\n" \
  "ds_read_b128 v[24:27], %[xc0] offset:22528\n" \
  "v_mfma_i32_16x16x64_i8 v[144:147], v[112:115], v[44:47], v[144:147]\n" \
  "ds_read_b128 v[28:31], %[xc1] offset:22528\n" \
  "v_mfma_i32_16x16x64_i8 v[148:151], v[116:119], v[44:47], v[148:151]\n" \
  "v_mfma_i32_16x16x64_i8 v[148:151], v[100:103], v[40:43], v[148:151]\n" \
  "v_mfma_i32_16x16x64_i8 v[148:151], v[84:87], v[36:39], v[148:151]\n" \
  "s_add_u32 %[sA], %[sA], 0x8000\n" \
  "s_cmp_eq_u32 %[sA], 0x18000\n" \
  "v_mfma_i32_16x16x64_i8 v[148:151], v[68:71], v[32:35], v[148:151]\n" \
  "s_cselect_b32 %[sA], 0, %[sA]\n" \
  "v_mfma_i32_16x16x64_i8 v[152:155], v[72:75], v[32:35], v[152:155]\n" \
  "v_add_u32 %[xc0], %[sA], %[xa0]\n" \
  "v_xor_b32 %[xc1], 64, %[xc0]\n" \
  "v_mfma_i32_16x16x64_i8 v[152:155], v[88:91], v[36:39], v[152:155]\n" \
  "v_mfma_i32_16x16x64_i8 v[152:155], v[104:107], v[40:43], v[152:155]\n" \
  "v_mfma_i32_16x16x64_i8 v[152:155], v[120:123], v[44:47], v[152:155]\n" \
  "v_mfma_i32_16x16x64_i8 v[156:159], v[124:127], v[44:47], v[156:159]\n" \
  "v_mfma_i32_16x16x64_i8 v[156:159], v[108:111], v[40:43], v[156:159]\n" \
  "v_mfma_i32_16x16x64_i8 v[156:159], v[92:95], v[36:39], v[156:159]\n" \
  "v_mfma_i32_16x16x64_i8 v[156:159], v[76:79], v[32:35], v[156:159]\n" \
  "s_waitcnt lgkmcnt(0)\n" \
  "s_barrier\n" \
  "v_mfma_i32_16x16x64_i8 v[160:163], v[64:67], v[48:51], v[160:163]\n" \
  "ds_read_b128 v[32:35], %[xc0] offset:0\n" \
  "buffer_load_dword %[vtmp], off, %[rt], 0 sc1\n" \
  "v_mfma_i32_16x16x64_i8 v[160:163], v[80:83], v[52:55], v[160:163]\n" \
  "ds_read_b128 v[36:39], %[xc1] offset:0\n" \
  "v_mfma_i32_16x16x64_i8 v[160:163], v[96:99], v[56:59], v[160:163]\n" \
  "ds_read_b128 v[40:43], %[xc0] offset:16384\n" \
  "v_mfma_i32_16x16x64_i8 v[160:163], v[112:115], v[60:63], v[160:163]\n" \
  "ds_read_b128 v[44:47], %[xc1] offset:16384\n" \
  "v_mfma_i32_16x16x64_i8 v[176:179], v[64:67], v[16:19], v[176:179]\n" \
  "ds_read_b64_tr_b8 v[64:65], %[wr0] offset:32768\n" \
  "v_mfma_i32_16x16x64_i8 v[176:179], v[80:83], v[20:23], v[176:179]\n" \
  "ds_read_b64_tr_b8 v[66:67], %[wr0] offset:33792\n" \
  "v_mfma_i32_16x16x64_i8 v[176:179], v[96:99], v[24:27], v[176:179]\n" \
  "ds_read_b64_tr_b8 v[80:81], %[wr0] offset:40960\n" \
  "v_mfma_i32_16x16x64_i8 v[176:179], v[112:115], v[28:31], v[176:179]\n" \
  "ds_read_b64_tr_b8 v[82:83], %[wr0] offset:41984\n" \
  "v_mfma_i32_16x16x64_i8 v[164:167], v[68:71], v[48:51], v[164:167]\n" \
  "ds_read_b64_tr_b8 v[96:97], %[wr0] offset:49152\n" \
  "v_mfma_i32_16x16x64_i8 v[164:167], v[84:87], v[52:55], v[164:167]\n" \
  "ds_read_b64_tr_b8 v[98:99], %[wr0] offset:50176\n" \
  "v_mfma_i32_16x16x64_i8 v[164:167], v[100:103], v[56:59], v[164:167]\n" \
  "ds_read_b64_tr_b8 v[112:113], %[wr0] offset:57344\n" \
  "v_mfma_i32_16x16x64_i8 v[164:167], v[116:119], v[60:63], v[164:167]\n" \
  "ds_read_b64_tr_b8 v[114:115], %[wr0] offset:58368\n" \
  "v_mfma_i32_16x16x64_i8 v[180:183], v[68:71], v[16:19], v[180:183]\n" \
  "ds_read_b64_tr_b8 v[68:69], %[wr1] offset:32768\n" \
  "v_mfma_i32_16x16x64_i8 v[180:183], v[84:87], v[20:23], v[180:183]\n" \
  "ds_read_b64_tr_b8 v[70:71], %[wr1] offset:33792\n" \
  "v_mfma_i32_16x16x64_i8 v[180:183], v[100:103], v[24:27], v[180:183]\n" \
  "ds_read_b64_tr_b8 v[84:85], %[wr1] offset:40960\n" \
  "v_mfma_i32_16x16x64_i8 v[180:183], v[116:119], v[28:31], v[180:183]\n" \
  "ds_read_b64_tr_b8 v[86:87], %[wr1] offset:41984\n" \
  "v_mfma_i32_16x16x64_i8 v[168:171], v[72:75], v[48:51], v[168:171]\n" \
  "ds_read_b64_tr_b8 v[100:101], %[wr1] offset:49152\n" \
  "v_mfma_i32_16x16x64_i8 v[168:171], v[88:91], v[52:55], v[168:171]\n" \
  "ds_read_b64_tr_b8 v[102:103], %[wr1] offset:50176\n" \
  "v_mfma_i32_16x16x64_i8 v[168:171], v[104:107], v[56:59], v[168:171]\n" \
  "ds_read_b64_tr_b8 v[116:117], %[wr1] offset:57344\n" \
  "v_mfma_i32_16x16x64_i8 v[168:171], v[120:123], v[60:63], v[168:171]\n" \
  "ds_read_b64_tr_b8 v[118:119], %[wr1] offset:58368\n" \
  "v_mfma_i32_16x16x64_i8 v[184:187], v[72:75], v[16:19], v[184:187]\n" \
  "ds_read_b64_tr_b8 v[72:73], %[wr2] offset:32768\n" \
  "v_mfma_i32_16x16x64_i8 v[184:187], v[88:91], v[20:23], v[184:187]\n" \
  "ds_read_b64_tr_b8 v[74:75], %[wr2] offset:33792\n" \
  "v_mfma_i32_16x16x64_i8 v[184:187], v[104:107], v[24:27], v[184:187]\n" \
  "ds_read_b64_tr_b8 v[88:89], %[wr2] offset:40960\n" \
  "v_mfma_i32_16x16x64_i8 v[184:187], v[120:123], v[28:31], v[184:187]\n" \
  "ds_read_b64_tr_b8 v[90:91], %[wr2] offset:41984\n" \
  "v_mfma_i32_16x16x64_i8 v[172:175], v[76:79], v[48:51], v[172:175]\n" \
  "ds_read_b64_tr_b8 v[104:105], %[wr2] offset:49152\n" \
  "v_mfma_i32_16x16x64_i8 v[172:175], v[92:95], v[52:55], v[172:175]\n" \
  "ds_read_b64_tr_b8 v[106:107], %[wr2] offset:50176\n" \
  "v_mfma_i32_16x16x64_i8 v[172:175], v[108:111], v[56:59], v[172:175]\n" \
  "ds_read_b64_tr_b8 v[120:121], %[wr2] offset:57344\n" \
  "v_mfma_i32_16x16x64_i8 v[172:175], v[124:127], v[60:63], v[172:175]\n" \
  "ds_read_b64_tr_b8 v[122:123], %[wr2] offset:58368\n" \
  "v_mfma_i32_16x16x64_i8 v[188:191], v[76:79], v[16:19], v[188:191]\n" \
  "ds_read_b64_tr_b8 v[76:77], %[wr3] offset:32768\n" \
  "v_mfma_i32_16x16x64_i8 v[188:191], v[92:95], v[20:23], v[188:191]\n" \
  "ds_read_b64_tr_b8 v[78:79], %[wr3] offset:33792\n" \
  "v_mfma_i32_16x16x64_i8 v[188:191], v[108:111], v[24:27], v[188:191]\n" \
  "ds_read_b64_tr_b8 v[92:93], %[wr3] offset:40960\n" \
  "v_mfma_i32_16x16x64_i8 v[188:191], v[124:127], v[28:31], v[188:191]\n" \
  "ds_read_b64_tr_b8 v[94:95], %[wr3] offset:41984\n" \
  "s_waitcnt lgkmcnt(9)\n" \
  "v_mfma_i32_16x16x64_i8 v[128:131], v[64:67], v[32:35], v[128:131]\n" \
  "ds_read_b64_tr_b8 v[108:109], %[wr3] offset:49152\n" \
  "v_mfma_i32_16x16x64_i8 v[128:131], v[80:83], v[36:39], v[128:131]\n" \
  "ds_read_b64_tr_b8 v[110:111], %[wr3] offset:50176\n" \
  "v_mfma_i32_16x16x64_i8 v[128:131], v[96:99], v[40:43], v[128:131]\n" \
  "ds_read_b64_tr_b8 v[124:125], %[wr3] offset:57344\n" \
  "v_mfma_i32_16x16x64_i8 v[128:131], v[112:115], v[44:47], v[128:131]\n" \
  "ds_read_b64_tr_b8 v[126:127], %[wr3] offset:58368\n" \
  "v_mfma_i32_16x16x64_i8 v[132:135], v[68:71], v[32:35], v[132:135]\n" \
  "ds_read_b128 v[48:51], %[xc0] offset:2048\n" \
  "v_mfma_i32_16x16x64_i8 v[132:135], v[84:87], v[36:39], v[132:135]\n" \
  "ds_read_b128 v[52:55], %[xc1] offset:2048\n" \
  "v_mfma_i32_16x16x64_i8 v[132:135], v[100:103], v[40:43], v[132:135]\n" \
  "ds_read_b128 v[56:59], %[xc0] offset:18432\n" \
  "v_mfma_i32_16x16x64_i8 v[132:135], v[116:119], v[44:47], v[132:135]\n" \
  "ds_read_b128 v[60:63], %[xc1] offset:18432\n" \
  "v_mfma_i32_16x16x64_i8 v[136:139], v[72:75], v[32:35], v[136:139]\n" \
  "ds_read_b128 v[16:19], %[xc0] offset:4096\n" \
  "s_waitcnt lgkmcnt(9)\n" \
  "v_mfma_i32_16x16x64_i8 v[136:139], v[88:91], v[36:39], v[136:139]\n" \
  "ds_read_b128 v[20:23], %[xc1] offset:4096\n" \
  "v_mfma_i32_16x16x64_i8 v[136:139], v[104:107], v[40:43], v[136:139]\n" \
  "ds_read_b128 v[24:27], %[xc0] offset:20480\n" \
  "v_mfma_i32_16x16x64_i8 v[136:139], v[120:123], v[44:47], v[136:139]\n" \
  "ds_read_b128 v[28:31], %[xc1] offset:20480\n" \
  "v_mfma_i32_16x16x64_i8 v[140:143], v[76:79], v[32:35], v[140:143]\n" \
  "s_waitcnt vmcnt(0)\n" \
  "v_readfirstlane_b32 %[sZ], %[vtmp]\n" \
  "v_mfma_i32_16x16x64_i8 v[140:143], v[92:95], v[36:39], v[140:143]\n" \
  "s_waitcnt lgkmcnt(7)\n" \
  "v_mfma_i32_16x16x64_i8 v[140:143], v[108:111], v[40:43], v[140:143]\n" \
  "s_cmp_eq_u32 %[sZ], %[sCall]\n" \
  "s_cselect_b32 %[sZ], 0x7fc00000, 0\n" \
  "s_or_b32 %[sS], %[sS], %[sZ]\n" \
  "v_mov_b32 %[vZ], %[sZ]\n" \
  "v_mfma_i32_16x16x64_i8 v[140:143], v[124:127], v[44:47], v[140:143]\n" \
  "v_mfma_i32_16x16x64_i8 v[144:147], v[64:67], v[48:51], v[144:147]\n" \
  "ds_read_b128 v[32:35], %[xc0] offset:6144\n" \
  "v_cvt_f32_i32 v128, v128\n" \
  "v_cvt_f32_i32 v132, v132\n" \
  "s_waitcnt lgkmcnt(5)\n" \
  "v_mfma_i32_16x16x64_i8 v[144:147], v[80:83], v[52:55], v[144:147]\n" \
  "ds_read_b128 v[36:39], %[xc1] offset:6144\n" \
  "v_cvt_f32_i32 v129, v129\n" \
  "v_cvt_f32_i32 v133, v133\n" \
  "v_mfma_i32_16x16x64_i8 v[144:147], v[96:99], v[56:59], v[144:147]\n" \
  "ds_read_b128 v[40:43], %[xc0] offset:22528\n" \
  "v_mul_f32 %[t0], %[sS], v128\n" \
  "v_mul_f32 %[t1], %[sS], v132\n" \
  "v_mfma_i32_16x16x64_i8 v[144:147], v[112:115], v[60:63], v[144:147]\n" \
  "ds_read_b128 v[44:47], %[xc1] offset:22528\n" \
  "v_mul_f32 %[t2], %[sS], v129\n" \
  "v_mul_f32 %[t3], %[sS], v133\n" \
  "v_mfma_i32_16x16x64_i8 v[148:151], v[116:119], v[60:63], v[148:151]\n" \
  "v_max_f32 v128, %[t0], %[vZ]\n" \
  "v_max_f32 v132, %[t1], %[vZ]\n" \
  "v_max_f32 v129, %[t2], %[vZ]\n" \
  "v_max_f32 v133, %[t3], %[vZ]\n" \
  "v_mfma_i32_16x16x64_i8 v[148:151], v[100:103], v[56:59], v[148:151]\n" \
  "v_max_f32_dpp v128, %[t1], %[vZ] row_ror:8 row_mask:0xf bank_mask:0xc\n" \
  "v_max_f32_dpp v132, %[t0], %[vZ] row_ror:8 row_mask:0xf bank_mask:0x3\n" \
  "v_max_f32_dpp v129, %[t3], %[vZ] row_ror:8 row_mask:0xf bank_mask:0xc\n" \
  "v_max_f32_dpp v133, %[t2], %[vZ] row_ror:8 row_mask:0xf bank_mask:0x3\n" \
  "v_mfma_i32_16x16x64_i8 v[148:151], v[84:87], v[52:55], v[148:151]\n" \
  "s_add_u32 %[sA], %[sA], 0x8000\n" \
  "s_cmp_eq_u32 %[sA], 0x18000\n" \
  "v_cvt_f32_i32 v130, v130\n" \
  "v_cvt_f32_i32 v134, v134\n" \
  "v_mfma_i32_16x16x64_i8 v[148:151], v[68:71], v[48:51], v[148:151]\n" \
  "s_cselect_b32 %[sA], 0, %[sA]\n" \
  "v_cvt_f32_i32 v131, v131\n" \
  "v_cvt_f32_i32 v135, v135\n" \
  "v_mul_f32 %[t0], %[sS], v130\n" \
  "v_mfma_i32_16x16x64_i8 v[152:155], v[72:75], v[48:51], v[152:155]\n" \
  "v_add_u32 %[xc0], %[sA], %[xa0]\n" \
  "v_xor_b32 %[xc1], 64, %[xc0]\n" \
  "v_mul_f32 %[t1], %[sS], v134\n" \
  "v_mul_f32 %[t2], %[sS], v131\n" \
  "v_mfma_i32_16x16x64_i8 v[152:155], v[88:91], v[52:55], v[152:155]\n" \
  "v_mul_f32 %[t3], %[sS], v135\n" \
  "v_max_f32 v130, %[t0], %[vZ]\n" \
  "v_max_f32 v134, %[t1], %[vZ]\n" \
  "v_max_f32 v131, %[t2], %[vZ]\n" \
  "v_mfma_i32_16x16x64_i8 v[152:155], v[104:107], v[56:59], v[152:155]\n" \
  "v_max_f32 v135, %[t3], %[vZ]\n" \
  "v_max_f32_dpp v130, %[t1], %[vZ] row_ror:8 row_mask:0xf bank_mask:0xc\n" \
  "v_max_f32_dpp v134, %[t0], %[vZ] row_ror:8 row_mask:0xf bank_mask:0x3\n" \
  "v_max_f32_dpp v131, %[t3], %[vZ] row_ror:8 row_mask:0xf bank_mask:0xc\n" \
  "v_mfma_i32_16x16x64_i8 v[152:155], v[120:123], v[60:63], v[152:155]\n" \
  "v_max_f32_dpp v135, %[t2], %[vZ] row_ror:8 row_mask:0xf bank_mask:0x3\n" \
  "v_cvt_f32_i32 v136, v136\n" \
  "v_cvt_f32_i32 v140, v140\n" \
  "v_cvt_f32_i32 v137, v137\n" \
  "v_mfma_i32_16x16x64_i8 v[156:159], v[124:127], v[60:63], v[156:159]\n" \
  "v_cvt_f32_i32 v141, v141\n" \
  "v_mul_f32 %[t0], %[sS], v136\n" \
  "v_mul_f32 %[t1], %[sS], v140\n" \
  "v_mul_f32 %[t2], %[sS], v137\n" \
  "v_mfma_i32_16x16x64_i8 v[156:159], v[108:111], v[56:59], v[156:159]\n" \
  "v_mul_f32 %[t3], %[sS], v141\n" \
  "v_max_f32 v136, %[t0], %[vZ]\n" \
  "v_max_f32 v140, %[t1], %[vZ]\n" \
  "v_max_f32 v137, %[t2], %[vZ]\n" \
  "v_mfma_i32_16x16x64_i8 v[156:159], v[92:95], v[52:55], v[156:159]\n" \
  "v_max_f32 v141, %[t3], %[vZ]\n" \
  "v_max_f32_dpp v136, %[t1], %[vZ] row_ror:8 row_mask:0xf bank_mask:0xc\n" \
  "v_max_f32_dpp v140, %[t0], %[vZ] row_ror:8 row_mask:0xf bank_mask:0x3\n" \
  "v_max_f32_dpp v137, %[t3], %[vZ] row_ror:8 row_mask:0xf bank_mask:0xc\n" \
  "v_mfma_i32_16x16x64_i8 v[156:159], v[76:79], v[48:51], v[156:159]\n" \
  "s_waitcnt lgkmcnt(0)\n" \
  "s_barrier\n" \
  "v_max_f32_dpp v141, %[t2], %[vZ] row_ror:8 row_mask:0xf bank_mask:0x3\n" \
  "v_cvt_f32_i32 v138, v138\n" \
  "v_mfma_i32_16x16x64_i8 v[160:163], v[64:67], v[16:19], v[160:163]\n" \
  "ds_read_b128 v[48:51], %[xc0] offset:0\n" \
  "v_cvt_f32_i32 v142, v142\n" \
  "v_cvt_f32_i32 v139, v139\n" \
  "v_mfma_i32_16x16x64_i8 v[160:163], v[80:83], v[20:23], v[160:163]\n" \
  "ds_read_b128 v[52:55], %[xc1] offset:0\n" \
  "v_cvt_f32_i32 v143, v143\n" \
  "v_mul_f32 %[t0], %[sS], v138\n" \
  "v_mfma_i32_16x16x64_i8 v[160:163], v[96:99], v[24:27], v[160:163]\n" \
  "ds_read_b128 v[56:59], %[xc0] offset:16384\n" \
  "v_mul_f32 %[t1], %[sS], v142\n" \
  "v_mul_f32 %[t2], %[sS], v139\n" \
  "v_mfma_i32_16x16x64_i8 v[160:163], v[112:115], v[28:31], v[160:163]\n" \
  "ds_read_b128 v[60:63], %[xc1] offset:16384\n" \
  "v_mul_f32 %[t3], %[sS], v143\n" \
  "v_max_f32 v138, %[t0], %[vZ]\n" \
  "v_mfma_i32_16x16x64_i8 v[164:167], v[116:119], v[28:31], v[164:167]\n" \
  "v_max_f32 v142, %[t1], %[vZ]\n" \
  "v_max_f32 v139, %[t2], %[vZ]\n" \
  "v_max_f32 v143, %[t3], %[vZ]\n" \
  "v_max_f32_dpp v138, %[t1], %[vZ] row_ror:8 row_mask:0xf bank_mask:0xc\n" \
  "v_mfma_i32_16x16x64_i8 v[164:167], v[100:103], v[24:27], v[164:167]\n" \
  "v_max_f32_dpp v142, %[t0], %[vZ] row_ror:8 row_mask:0xf bank_mask:0x3\n" \
  "v_max_f32_dpp v139, %[t3], %[vZ] row_ror:8 row_mask:0xf bank_mask:0xc\n" \
  "v_max_f32_dpp v143, %[t2], %[vZ] row_ror:8 row_mask:0xf bank_mask:0x3\n" \
  "s_mov_b32 %[sO], 0x0\n" \
  "buffer_store_dwordx4 v[128:131], %[cv], %[rc], %[sO] offen offset:0 sc1\n" \
  "v_mfma_i32_16x16x64_i8 v[164:167], v[84:87], v[20:23], v[164:167]\n" \
  "v_cvt_f32_i32 v144, v144\n" \
  "v_cvt_f32_i32 v148, v148\n" \
  "v_cvt_f32_i32 v145, v145\n" \
  "v_cvt_f32_i32 v149, v149\n" \
  "v_mfma_i32_16x16x64_i8 v[164:167], v[68:71], v[16:19], v[164:167]\n" \
  "v_mul_f32 %[t0], %[sS], v144\n" \
  "v_mul_f32 %[t1], %[sS], v148\n" \
  "v_mul_f32 %[t2], %[sS], v145\n" \
  "v_mul_f32 %[t3], %[sS], v149\n" \
  "v_mfma_i32_16x16x64_i8 v[168:171], v[72:75], v[16:19], v[168:171]\n" \
  "v_max_f32 v144, %[t0], %[vZ]\n" \
  "v_max_f32 v148, %[t1], %[vZ]\n" \
  "v_max_f32 v145, %[t2], %[vZ]\n" \
  "v_max_f32 v149, %[t3], %[vZ]\n" \
  "v_mfma_i32_16x16x64_i8 v[168:171], v[88:91], v[20:23], v[168:171]\n" \
  "v_max_f32_dpp v144, %[t1], %[vZ] row_ror:8 row_mask:0xf bank_mask:0xc\n" \
  "v_max_f32_dpp v148, %[t0], %[vZ] row_ror:8 row_mask:0xf bank_mask:0x3\n" \
  "v_max_f32_dpp v145, %[t3], %[vZ] row_ror:8 row_mask:0xf bank_mask:0xc\n" \
  "v_max_f32_dpp v149, %[t2], %[vZ] row_ror:8 row_mask:0xf bank_mask:0x3\n" \
  "v_mfma_i32_16x16x64_i8 v[168:171], v[104:107], v[24:27], v[168:171]\n" \
  "v_cvt_f32_i32 v146, v146\n" \
  "v_cvt_f32_i32 v150, v150\n" \
  "v_cvt_f32_i32 v147, v147\n" \
  "v_cvt_f32_i32 v151, v151\n" \
  "v_mfma_i32_16x16x64_i8 v[168:171], v[120:123], v[28:31], v[168:171]\n" \
  "v_mul_f32 %[t0], %[sS], v146\n" \
  "v_mul_f32 %[t1], %[sS], v150\n" \
  "v_mul_f32 %[t2], %[sS], v147\n" \
  "v_mul_f32 %[t3], %[sS], v151\n" \
  "v_mfma_i32_16x16x64_i8 v[172:175], v[124:127], v[28:31], v[172:175]\n" \
  "v_max_f32 v146, %[t0], %[vZ]\n" \
  "v_max_f32 v150, %[t1], %[vZ]\n" \
  "v_max_f32 v147, %[t2], %[vZ]\n" \
  "v_max_f32 v151, %[t3], %[vZ]\n" \
  "v_mfma_i32_16x16x64_i8 v[172:175], v[108:111], v[24:27], v[172:175]\n" \
  "v_max_f32_dpp v146, %[t1], %[vZ] row_ror:8 row_mask:0xf bank_mask:0xc\n" \
  "v_max_f32_dpp v150, %[t0], %[vZ] row_ror:8 row_mask:0xf bank_mask:0x3\n" \
  "v_max_f32_dpp v147, %[t3], %[vZ] row_ror:8 row_mask:0xf bank_mask:0xc\n" \
  "v_max_f32_dpp v151, %[t2], %[vZ] row_ror:8 row_mask:0xf bank_mask:0x3\n" \
  "v_mfma_i32_16x16x64_i8 v[172:175], v[92:95], v[20:23], v[172:175]\n" \
  "v_cvt_f32_i32 v152, v152\n" \
  "v_cvt_f32_i32 v156, v156\n" \
  "v_cvt_f32_i32 v153, v153\n" \
  "v_cvt_f32_i32 v157, v157\n" \
  "v_mfma_i32_16x16x64_i8 v[172:175], v[76:79], v[16:19], v[172:175]\n" \
  "v_mul_f32 %[t0], %[sS], v152\n" \
  "v_mul_f32 %[t1], %[sS], v156\n" \
  "v_mul_f32 %[t2], %[sS], v153\n" \
  "v_mul_f32 %[t3], %[sS], v157\n" \
  "v_mfma_i32_16x16x64_i8 v[176:179], v[64:67], v[32:35], v[176:179]\n" \
  "v_max_f32 v152, %[t0], %[vZ]\n" \
  "v_max_f32 v156, %[t1], %[vZ]\n" \
  "v_max_f32 v153, %[t2], %[vZ]\n" \
  "v_max_f32 v157, %[t3], %[vZ]\n" \
  "v_mfma_i32_16x16x64_i8 v[176:179], v[80:83], v[36:39], v[176:179]\n" \
  "s_mov_b32 %[sO], 0x0\n" \
  "buffer_store_dwordx4 v[136:139], %[cv], %[rc], %[sO] offen offset:128 sc1\n" \
  "s_mov_b32 %[sO], 0x80000\n" \
  "buffer_store_dwordx4 v[132:135], %[cv], %[rc], %[sO] offen offset:0 sc1\n" \
  "v_mfma_i32_16x16x64_i8 v[176:179], v[96:99], v[40:43], v[176:179]\n" \
  "v_max_f32_dpp v152, %[t1], %[vZ] row_ror:8 row_mask:0xf bank_mask:0xc\n" \
  "v_max_f32_dpp v156, %[t0], %[vZ] row_ror:8 row_mask:0xf bank_mask:0x3\n" \
  "v_max_f32_dpp v153, %[t3], %[vZ] row_ror:8 row_mask:0xf bank_mask:0xc\n" \
  "v_max_f32_dpp v157, %[t2], %[vZ] row_ror:8 row_mask:0xf bank_mask:0x3\n" \
  "v_mfma_i32_16x16x64_i8 v[176:179], v[112:115], v[44:47], v[176:179]\n" \
  "v_cvt_f32_i32 v154, v154\n" \
  "v_cvt_f32_i32 v158, v158\n" \
  "v_cvt_f32_i32 v155, v155\n" \
  "v_cvt_f32_i32 v159, v159\n" \
  "v_mfma_i32_16x16x64_i8 v[180:183], v[116:119], v[44:47], v[180:183]\n" \
  "v_mul_f32 %[t0], %[sS], v154\n" \
  "v_mul_f32 %[t1], %[sS], v158\n" \
  "v_mul_f32 %[t2], %[sS], v155\n" \
  "v_mul_f32 %[t3], %[sS], v159\n" \
  "v_mfma_i32_16x16x64_i8 v[180:183], v[100:103], v[40:43], v[180:183]\n" \
  "v_max_f32 v154, %[t0], %[vZ]\n" \
  "v_max_f32 v158, %[t1], %[vZ]\n" \
  "v_max_f32 v155, %[t2], %[vZ]\n" \
  "v_max_f32 v159, %[t3], %[vZ]\n" \
  "v_mfma_i32_16x16x64_i8 v[180:183], v[84:87], v[36:39], v[180:183]\n" \
  "v_max_f32_dpp v154, %[t1], %[vZ] row_ror:8 row_mask:0xf bank_mask:0xc\n" \
  "v_max_f32_dpp v158, %[t0], %[vZ] row_ror:8 row_mask:0xf bank_mask:0x3\n" \
  "v_max_f32_dpp v155, %[t3], %[vZ] row_ror:8 row_mask:0xf bank_mask:0xc\n" \
  "v_max_f32_dpp v159, %[t2], %[vZ] row_ror:8 row_mask:0xf bank_mask:0x3\n" \
  "v_mfma_i32_16x16x64_i8 v[180:183], v[68:71], v[32:35], v[180:183]\n" \
  "v_cvt_f32_i32 v160, v160\n" \
  "v_cvt_f32_i32 v164, v164\n" \
  "v_cvt_f32_i32 v161, v161\n" \
  "v_cvt_f32_i32 v165, v165\n" \
  "v_mfma_i32_16x16x64_i8 v[184:187], v[72:75], v[32:35], v[184:187]\n" \
  "v_mul_f32 %[t0], %[sS], v160\n" \
  "v_mul_f32 %[t1], %[sS], v164\n" \
  "v_mul_f32 %[t2], %[sS], v161\n" \
  "v_mul_f32 %[t3], %[sS], v165\n" \
  "v_mfma_i32_16x16x64_i8 v[184:187], v[88:91], v[36:39], v[184:187]\n" \
  "v_max_f32 v160, %[t0], %[vZ]\n" \
  "v_max_f32 v164, %[t1], %[vZ]\n" \
  "v_max_f32 v161, %[t2], %[vZ]\n" \
  "v_max_f32 v165, %[t3], %[vZ]\n" \
  "v_mfma_i32_16x16x64_i8 v[184:187], v[104:107], v[40:43], v[184:187]\n" \
  "v_max_f32_dpp v160, %[t1], %[vZ] row_ror:8 row_mask:0xf bank_mask:0xc\n" \
  "v_max_f32_dpp v164, %[t0], %[vZ] row_ror:8 row_mask:0xf bank_mask:0x3\n" \
  "v_max_f32_dpp v161, %[t3], %[vZ] row_ror:8 row_mask:0xf bank_mask:0xc\n" \
  "v_max_f32_dpp v165, %[t2], %[vZ] row_ror:8 row_mask:0xf bank_mask:0x3\n" \
  "v_mfma_i32_16x16x64_i8 v[184:187], v[120:123], v[44:47], v[184:187]\n" \
  "v_cvt_f32_i32 v162, v162\n" \
  "v_cvt_f32_i32 v166, v166\n" \
  "v_cvt_f32_i32 v163, v163\n" \
  "v_cvt_f32_i32 v167, v167\n" \
  "v_mfma_i32_16x16x64_i8 v[188:191], v[124:127], v[44:47], v[188:191]\n" \
  "v_mul_f32 %[t0], %[sS], v162\n" \
  "v_mul_f32 %[t1], %[sS], v166\n" \
  "v_mul_f32 %[t2], %[sS], v163\n" \
  "v_mul_f32 %[t3], %[sS], v167\n" \
  "v_mfma_i32_16x16x64_i8 v[188:191], v[108:111], v[40:43], v[188:191]\n" \
  "s_mov_b32 %[sO], 0x80000\n" \
  "buffer_store_dwordx4 v[140:143], %[cv], %[rc], %[sO] offen offset:128 sc1\n" \
  "s_mov_b32 %[sO], 0x100000\n" \
  "buffer_store_dwordx4 v[144:147], %[cv], %[rc], %[sO] offen offset:0 sc1\n" \
  "v_mfma_i32_16x16x64_i8 v[188:191], v[92:95], v[36:39], v[188:191]\n" \
  "v_max_f32 v162, %[t0], %[vZ]\n" \
  "v_max_f32 v166, %[t1], %[vZ]\n" \
  "v_max_f32 v163, %[t2], %[vZ]\n" \
  "v_max_f32 v167, %[t3], %[vZ]\n" \
  "v_mfma_i32_16x16x64_i8 v[188:191], v[76:79], v[32:35], v[188:191]\n" \
  "v_max_f32_dpp v162, %[t1], %[vZ] row_ror:8 row_mask:0xf bank_mask:0xc\n" \
  "v_max_f32_dpp v166, %[t0], %[vZ] row_ror:8 row_mask:0xf bank_mask:0x3\n" \
  "v_max_f32_dpp v163, %[t3], %[vZ] row_ror:8 row_mask:0xf bank_mask:0xc\n" \
  "v_max_f32_dpp v167, %[t2], %[vZ] row_ror:8 row_mask:0xf bank_mask:0x3\n" \
  "s_waitcnt lgkmcnt(0)\n" \
  "v_mfma_i32_16x16x64_i8 v[192:195], v[64:67], v[48:51], v[192:195]\n" \
  "v_cvt_f32_i32 v168, v168\n" \
  "v_cvt_f32_i32 v172, v172\n" \
  "v_cvt_f32_i32 v169, v169\n" \
  "v_cvt_f32_i32 v173, v173\n" \
  "v_mfma_i32_16x16x64_i8 v[192:195], v[80:83], v[52:55], v[192:195]\n" \
  "v_mul_f32 %[t0], %[sS], v168\n" \
  "v_mul_f32 %[t1], %[sS], v172\n" \
  "v_mul_f32 %[t2], %[sS], v169\n" \
  "v_mul_f32 %[t3], %[sS], v173\n" \
  "v_mfma_i32_16x16x64_i8 v[192:195], v[96:99], v[56:59], v[192:195]\n" \
  "v_max_f32 v168, %[t0], %[vZ]\n" \
  "v_max_f32 v172, %[t1], %[vZ]\n" \
  "v_max_f32 v169, %[t2], %[vZ]\n" \
  "v_max_f32 v173, %[t3], %[vZ]\n" \
  "v_mfma_i32_16x16x64_i8 v[192:195], v[112:115], v[60:63], v[192:195]\n" \
  "v_max_f32_dpp v168, %[t1], %[vZ] row_ror:8 row_mask:0xf bank_mask:0xc\n" \
  "v_max_f32_dpp v172, %[t0], %[vZ] row_ror:8 row_mask:0xf bank_mask:0x3\n" \
  "v_max_f32_dpp v169, %[t3], %[vZ] row_ror:8 row_mask:0xf bank_mask:0xc\n" \
  "v_max_f32_dpp v173, %[t2], %[vZ] row_ror:8 row_mask:0xf bank_mask:0x3\n" \
  "v_mfma_i32_16x16x64_i8 v[196:199], v[68:71], v[48:51], v[196:199]\n" \
  "ds_read_b128 v[16:19], %[xc0] offset:2048\n" \
  "v_cvt_f32_i32 v170, v170\n" \
  "v_cvt_f32_i32 v174, v174\n" \
  "v_mfma_i32_16x16x64_i8 v[196:199], v[84:87], v[52:55], v[196:199]\n" \
  "ds_read_b128 v[20:23], %[xc1] offset:2048\n" \
  "v_cvt_f32_i32 v171, v171\n" \
  "v_cvt_f32_i32 v175, v175\n" \
  "v_mfma_i32_16x16x64_i8 v[196:199], v[100:103], v[56:59], v[196:199]\n" \
  "ds_read_b128 v[24:27], %[xc0] offset:18432\n" \
  "v_mul_f32 %[t0], %[sS], v170\n" \
  "v_mul_f32 %[t1], %[sS], v174\n" \
  "v_mfma_i32_16x16x64_i8 v[196:199], v[116:119], v[60:63], v[196:199]\n" \
  "ds_read_b128 v[28:31], %[xc1] offset:18432\n" \
  "v_mul_f32 %[t2], %[sS], v171\n" \
  "v_mul_f32 %[t3], %[sS], v175\n" \
  "v_mfma_i32_16x16x64_i8 v[200:203], v[72:75], v[48:51], v[200:203]\n" \
  "ds_read_b128 v[32:35], %[xc0] offset:4096\n" \
  "v_max_f32 v170, %[t0], %[vZ]\n" \
  "v_max_f32 v174, %[t1], %[vZ]\n" \
  "v_mfma_i32_16x16x64_i8 v[200:203], v[88:91], v[52:55], v[200:203]\n" \
  "ds_read_b128 v[36:39], %[xc1] offset:4096\n" \
  "s_mov_b32 %[sO], 0x100000\n" \
  "buffer_store_dwordx4 v[152:155], %[cv], %[rc], %[sO] offen offset:128 sc1\n" \
  "s_mov_b32 %[sO], 0x180000\n" \
  "buffer_store_dwordx4 v[148:151], %[cv], %[rc], %[sO] offen offset:0 sc1\n" \
  "v_mfma_i32_16x16x64_i8 v[200:203], v[104:107], v[56:59], v[200:203]\n" \
  "ds_read_b128 v[40:43], %[xc0] offset:20480\n" \
  "v_max_f32 v171, %[t2], %[vZ]\n" \
  "v_max_f32 v175, %[t3], %[vZ]\n" \
  "v_mfma_i32_16x16x64_i8 v[200:203], v[120:123], v[60:63], v[200:203]\n" \
  "ds_read_b128 v[44:47], %[xc1] offset:20480\n" \
  "v_max_f32_dpp v170, %[t1], %[vZ] row_ror:8 row_mask:0xf bank_mask:0xc\n" \
  "v_max_f32_dpp v174, %[t0], %[vZ] row_ror:8 row_mask:0xf bank_mask:0x3\n" \
  "v_mfma_i32_16x16x64_i8 v[204:207], v[76:79], v[48:51], v[204:207]\n" \
  "v_max_f32_dpp v171, %[t3], %[vZ] row_ror:8 row_mask:0xf bank_mask:0xc\n" \
  "v_max_f32_dpp v175, %[t2], %[vZ] row_ror:8 row_mask:0xf bank_mask:0x3\n" \
  "v_cvt_f32_i32 v176, v176\n" \
  "v_cvt_f32_i32 v180, v180\n" \
  "v_mfma_i32_16x16x64_i8 v[204:207], v[92:95], v[52:55], v[204:207]\n" \
  "v_cvt_f32_i32 v177, v177\n" \
  "v_cvt_f32_i32 v181, v181\n" \
  "v_mul_f32 %[t0], %[sS], v176\n" \
  "v_mul_f32 %[t1], %[sS], v180\n" \
  "v_mfma_i32_16x16x64_i8 v[204:207], v[108:111], v[56:59], v[204:207]\n" \
  "v_mul_f32 %[t2], %[sS], v177\n" \
  "v_mul_f32 %[t3], %[sS], v181\n" \
  "v_max_f32 v176, %[t0], %[vZ]\n" \
  "v_max_f32 v180, %[t1], %[vZ]\n" \
  "v_mfma_i32_16x16x64_i8 v[204:207], v[124:127], v[60:63], v[204:207]\n" \
  "v_max_f32 v177, %[t2], %[vZ]\n" \
  "v_max_f32 v181, %[t3], %[vZ]\n" \
  "v_max_f32_dpp v176, %[t1], %[vZ] row_ror:8 row_mask:0xf bank_mask:0xc\n" \
  "v_max_f32_dpp v180, %[t0], %[vZ] row_ror:8 row_mask:0xf bank_mask:0x3\n" \
  "s_waitcnt lgkmcnt(5)\n" \
  "v_mfma_i32_16x16x64_i8 v[208:211], v[64:67], v[16:19], v[208:211]\n" \
  "ds_read_b128 v[48:51], %[xc0] offset:6144\n" \
  "v_max_f32_dpp v177, %[t3], %[vZ] row_ror:8 row_mask:0xf bank_mask:0xc\n" \
  "v_max_f32_dpp v181, %[t2], %[vZ] row_ror:8 row_mask:0xf bank_mask:0x3\n" \
  "v_mfma_i32_16x16x64_i8 v[208:211], v[80:83], v[20:23], v[208:211]\n" \
  "ds_read_b128 v[52:55], %[xc1] offset:6144\n" \
  "v_cvt_f32_i32 v178, v178\n" \
  "v_cvt_f32_i32 v182, v182\n" \
  "v_mfma_i32_16x16x64_i8 v[208:211], v[96:99], v[24:27], v[208:211]\n" \
  "ds_read_b128 v[56:59], %[xc0] offset:22528\n" \
  "v_cvt_f32_i32 v179, v179\n" \
  "v_cvt_f32_i32 v183, v183\n" \
  "s_waitcnt lgkmcnt(5)\n" \
  "v_mfma_i32_16x16x64_i8 v[208:211], v[112:115], v[28:31], v[208:211]\n" \
  "ds_read_b128 v[60:63], %[xc1] offset:22528\n" \
  "v_mul_f32 %[t0], %[sS], v178\n" \
  "v_mul_f32 %[t1], %[sS], v182\n" \
  "v_mfma_i32_16x16x64_i8 v[212:215], v[116:119], v[28:31], v[212:215]\n" \
  "v_mul_f32 %[t2], %[sS], v179\n" \
  "v_mul_f32 %[t3], %[sS], v183\n" \
  "v_max_f32 v178, %[t0], %[vZ]\n" \
  "v_max_f32 v182, %[t1], %[vZ]\n" \
  "v_mfma_i32_16x16x64_i8 v[212:215], v[100:103], v[24:27], v[212:215]\n" \
  "s_mov_b32 %[sO], 0x180000\n" \
  "buffer_store_dwordx4 v[156:159], %[cv], %[rc], %[sO] offen offset:128 sc1\n" \
  "s_mov_b32 %[sO], 0x200000\n" \
  "buffer_store_dwordx4 v[160:163], %[cv], %[rc], %[sO] offen offset:0 sc1\n" \
  "v_mfma_i32_16x16x64_i8 v[212:215], v[84:87], v[20:23], v[212:215]\n" \
  "s_add_u32 %[sA], %[sA], 0x8000\n" \
  "s_cmp_eq_u32 %[sA], 0x18000\n" \
  "v_max_f32 v179, %[t2], %[vZ]\n" \
  "v_max_f32 v183, %[t3], %[vZ]\n" \
  "v_mfma_i32_16x16x64_i8 v[212:215], v[68:71], v[16:19], v[212:215]\n" \
  "s_cselect_b32 %[sA], 0, %[sA]\n" \
  "v_max_f32_dpp v178, %[t1], %[vZ] row_ror:8 row_mask:0xf bank_mask:0xc\n" \
  "v_max_f32_dpp v182, %[t0], %[vZ] row_ror:8 row_mask:0xf bank_mask:0x3\n" \
  "v_max_f32_dpp v179, %[t3], %[vZ] row_ror:8 row_mask:0xf bank_mask:0xc\n" \
  "v_mfma_i32_16x16x64_i8 v[216:219], v[72:75], v[16:19], v[216:219]\n" \
  "v_add_u32 %[xc0], %[sA], %[xa0]\n" \
  "v_xor_b32 %[xc1], 64, %[xc0]\n" \
  "v_max_f32_dpp v183, %[t2], %[vZ] row_ror:8 row_mask:0xf bank_mask:0x3\n" \
  "v_cvt_f32_i32 v184, v184\n" \
  "v_mfma_i32_16x16x64_i8 v[216:219], v[88:91], v[20:23], v[216:219]\n" \
  "v_cvt_f32_i32 v188, v188\n" \
  "v_cvt_f32_i32 v185, v185\n" \
  "v_cvt_f32_i32 v189, v189\n" \
  "v_mul_f32 %[t0], %[sS], v184\n" \
  "v_mfma_i32_16x16x64_i8 v[216:219], v[104:107], v[24:27], v[216:219]\n" \
  "v_mul_f32 %[t1], %[sS], v188\n" \
  "v_mul_f32 %[t2], %[sS], v185\n" \
  "v_mul_f32 %[t3], %[sS], v189\n" \
  "v_max_f32 v184, %[t0], %[vZ]\n" \
  "v_mfma_i32_16x16x64_i8 v[216:219], v[120:123], v[28:31], v[216:219]\n" \
  "v_max_f32 v188, %[t1], %[vZ]\n" \
  "v_max_f32 v185, %[t2], %[vZ]\n" \
  "v_max_f32 v189, %[t3], %[vZ]\n" \
  "v_max_f32_dpp v184, %[t1], %[vZ] row_ror:8 row_mask:0xf bank_mask:0xc\n" \
  "v_mfma_i32_16x16x64_i8 v[220:223], v[124:127], v[28:31], v[220:223]\n" \
  "v_max_f32_dpp v188, %[t0], %[vZ] row_ror:8 row_mask:0xf bank_mask:0x3\n" \
  "v_max_f32_dpp v185, %[t3], %[vZ] row_ror:8 row_mask:0xf bank_mask:0xc\n" \
  "v_max_f32_dpp v189, %[t2], %[vZ] row_ror:8 row_mask:0xf bank_mask:0x3\n" \
  "v_cvt_f32_i32 v186, v186\n" \
  "v_mfma_i32_16x16x64_i8 v[220:223], v[108:111], v[24:27], v[220:223]\n" \
  "v_cvt_f32_i32 v190, v190\n" \
  "v_cvt_f32_i32 v187, v187\n" \
  "v_cvt_f32_i32 v191, v191\n" \
  "v_mul_f32 %[t0], %[sS], v186\n" \
  "v_mfma_i32_16x16x64_i8 v[220:223], v[92:95], v[20:23], v[220:223]\n" \
  "v_mul_f32 %[t1], %[sS], v190\n" \
  "v_mul_f32 %[t2], %[sS], v187\n" \
  "v_mul_f32 %[t3], %[sS], v191\n" \
  "v_max_f32 v186, %[t0], %[vZ]\n" \
  "v_mfma_i32_16x16x64_i8 v[220:223], v[76:79], v[16:19], v[220:223]\n" \
  "s_waitcnt lgkmcnt(0)\n" \
  "s_barrier\n" \
  "v_max_f32 v190, %[t1], %[vZ]\n" \
  "v_max_f32 v187, %[t2], %[vZ]\n" \
  "v_mfma_i32_16x16x64_i8 v[224:227], v[64:67], v[32:35], v[224:227]\n" \
  "ds_read_b128 v[16:19], %[xc0] offset:0\n" \
  "v_max_f32 v191, %[t3], %[vZ]\n" \
  "v_max_f32_dpp v186, %[t1], %[vZ] row_ror:8 row_mask:0xf bank_mask:0xc\n" \
  "v_mfma_i32_16x16x64_i8 v[224:227], v[80:83], v[36:39], v[224:227]\n" \
  "ds_read_b128 v[20:23], %[xc1] offset:0\n" \
  "s_mov_b32 %[sO], 0x200000\n" \
  "buffer_store_dwordx4 v[168:171], %[cv], %[rc], %[sO] offen offset:128 sc1\n" \
  "s_mov_b32 %[sO], 0x280000\n" \
  "buffer_store_dwordx4 v[164:167], %[cv], %[rc], %[sO] offen offset:0 sc1\n" \
  "v_mfma_i32_16x16x64_i8 v[224:227], v[96:99], v[40:43], v[224:227]\n" \
  "ds_read_b128 v[24:27], %[xc0] offset:16384\n" \
  "v_max_f32_dpp v190, %[t0], %[vZ] row_ror:8 row_mask:0xf bank_mask:0x3\n" \
  "v_max_f32_dpp v187, %[t3], %[vZ] row_ror:8 row_mask:0xf bank_mask:0xc\n" \
  "v_mfma_i32_16x16x64_i8 v[224:227], v[112:115], v[44:47], v[224:227]\n" \
  "ds_read_b128 v[28:31], %[xc1] offset:16384\n" \
  "v_max_f32_dpp v191, %[t2], %[vZ] row_ror:8 row_mask:0xf bank_mask:0x3\n" \
  "v_mfma_i32_16x16x64_i8 v[240:243], v[64:67], v[48:51], v[240:243]\n" \
  "ds_read_b64_tr_b8 v[64:65], %[wr0] offset:0\n" \
  "v_mfma_i32_16x16x64_i8 v[240:243], v[80:83], v[52:55], v[240:243]\n" \
  "ds_read_b64_tr_b8 v[66:67], %[wr0] offset:1024\n" \
  "v_mfma_i32_16x16x64_i8 v[240:243], v[96:99], v[56:59], v[240:243]\n" \
  "ds_read_b64_tr_b8 v[80:81], %[wr0] offset:8192\n" \
  "v_mfma_i32_16x16x64_i8 v[240:243], v[112:115], v[60:63], v[240:243]\n" \
  "ds_read_b64_tr_b8 v[82:83], %[wr0] offset:9216\n" \
  "v_mfma_i32_16x16x64_i8 v[228:231], v[68:71], v[32:35], v[228:231]\n" \
  "ds_read_b64_tr_b8 v[96:97], %[wr0] offset:16384\n" \
  "v_mfma_i32_16x16x64_i8 v[228:231], v[84:87], v[36:39], v[228:231]\n" \
  "ds_read_b64_tr_b8 v[98:99], %[wr0] offset:17408\n" \
  "v_mfma_i32_16x16x64_i8 v[228:231], v[100:103], v[40:43], v[228:231]\n" \
  "ds_read_b64_tr_b8 v[112:113], %[wr0] offset:24576\n" \
  "v_mfma_i32_16x16x64_i8 v[228:231], v[116:119], v[44:47], v[228:231]\n" \
  "ds_read_b64_tr_b8 v[114:115], %[wr0] offset:25600\n" \
  "v_mfma_i32_16x16x64_i8 v[244:247], v[68:71], v[48:51], v[244:247]\n" \
  "ds_read_b64_tr_b8 v[68:69], %[wr1] offset:0\n" \
  "v_mfma_i32_16x16x64_i8 v[244:247], v[84:87], v[52:55], v[244:247]\n" \
  "ds_read_b64_tr_b8 v[70:71], %[wr1] offset:1024\n" \
  "s_mov_b32 %[sO], 0x280000\n" \
  "buffer_store_dwordx4 v[172:175], %[cv], %[rc], %[sO] offen offset:128 sc1\n" \
  "s_mov_b32 %[sO], 0x300000\n" \
  "buffer_store_dwordx4 v[176:179], %[cv], %[rc], %[sO] offen offset:0 sc1\n" \
  "v_mfma_i32_16x16x64_i8 v[244:247], v[100:103], v[56:59], v[244:247]\n" \
  "ds_read_b64_tr_b8 v[84:85], %[wr1] offset:8192\n" \
  "v_mfma_i32_16x16x64_i8 v[244:247], v[116:119], v[60:63], v[244:247]\n" \
  "ds_read_b64_tr_b8 v[86:87], %[wr1] offset:9216\n" \
  "v_mfma_i32_16x16x64_i8 v[232:235], v[72:75], v[32:35], v[232:235]\n" \
  "ds_read_b64_tr_b8 v[100:101], %[wr1] offset:16384\n" \
  "v_mfma_i32_16x16x64_i8 v[232:235], v[88:91], v[36:39], v[232:235]\n" \
  "ds_read_b64_tr_b8 v[102:103], %[wr1] offset:17408\n" \
  "v_mfma_i32_16x16x64_i8 v[232:235], v[104:107], v[40:43], v[232:235]\n" \
  "ds_read_b64_tr_b8 v[116:117], %[wr1] offset:24576\n" \
  "v_mfma_i32_16x16x64_i8 v[232:235], v[120:123], v[44:47], v[232:235]\n" \
  "ds_read_b64_tr_b8 v[118:119], %[wr1] offset:25600\n" \
  "v_mfma_i32_16x16x64_i8 v[248:251], v[72:75], v[48:51], v[248:251]\n" \
  "ds_read_b64_tr_b8 v[72:73], %[wr2] offset:0\n" \
  "v_mfma_i32_16x16x64_i8 v[248:251], v[88:91], v[52:55], v[248:251]\n" \
  "ds_read_b64_tr_b8 v[74:75], %[wr2] offset:1024\n" \
  "v_mfma_i32_16x16x64_i8 v[248:251], v[104:107], v[56:59], v[248:251]\n" \
  "ds_read_b64_tr_b8 v[88:89], %[wr2] offset:8192\n" \
  "v_mfma_i32_16x16x64_i8 v[248:251], v[120:123], v[60:63], v[248:251]\n" \
  "ds_read_b64_tr_b8 v[90:91], %[wr2] offset:9216\n" \
  "v_mfma_i32_16x16x64_i8 v[236:239], v[76:79], v[32:35], v[236:239]\n" \
  "ds_read_b64_tr_b8 v[104:105], %[wr2] offset:16384\n" \
  "v_mfma_i32_16x16x64_i8 v[236:239], v[92:95], v[36:39], v[236:239]\n" \
  "ds_read_b64_tr_b8 v[106:107], %[wr2] offset:17408\n" \
  "s_mov_b32 %[sO], 0x300000\n" \
  "buffer_store_dwordx4 v[184:187], %[cv], %[rc], %[sO] offen offset:128 sc1\n" \
  "s_mov_b32 %[sO], 0x380000\n" \
  "buffer_store_dwordx4 v[180:183], %[cv], %[rc], %[sO] offen offset:0 sc1\n" \
  "v_mfma_i32_16x16x64_i8 v[236:239], v[108:111], v[40:43], v[236:239]\n" \
  "ds_read_b64_tr_b8 v[120:121], %[wr2] offset:24576\n" \
  "v_mfma_i32_16x16x64_i8 v[236:239], v[124:127], v[44:47], v[236:239]\n" \
  "ds_read_b64_tr_b8 v[122:123], %[wr2] offset:25600\n" \
  "v_mfma_i32_16x16x64_i8 v[252:255], v[76:79], v[48:51], v[252:255]\n" \
  "ds_read_b64_tr_b8 v[76:77], %[wr3] offset:0\n" \
  "v_mfma_i32_16x16x64_i8 v[252:255], v[92:95], v[52:55], v[252:255]\n" \
  "ds_read_b64_tr_b8 v[78:79], %[wr3] offset:1024\n" \
  "v_mfma_i32_16x16x64_i8 v[252:255], v[108:111], v[56:59], v[252:255]\n" \
  "ds_read_b64_tr_b8 v[92:93], %[wr3] offset:8192\n" \
  "v_mfma_i32_16x16x64_i8 v[252:255], v[124:127], v[60:63], v[252:255]\n" \
  "ds_read_b64_tr_b8 v[94:95], %[wr3] offset:9216\n" \
  "s_waitcnt lgkmcnt(9)\n" \
  "v_mfma_i32_16x16x64_i8 v[192:195], v[64:67], v[16:19], v[192:195]\n" \
  "ds_read_b64_tr_b8 v[108:109], %[wr3] offset:16384\n" \
  "v_mfma_i32_16x16x64_i8 v[192:195], v[80:83], v[20:23], v[192:195]\n" \
  "ds_read_b64_tr_b8 v[110:111], %[wr3] offset:17408\n" \
  "v_mfma_i32_16x16x64_i8 v[192:195], v[96:99], v[24:27], v[192:195]\n" \
  "ds_read_b64_tr_b8 v[124:125], %[wr3] offset:24576\n" \
  "v_mfma_i32_16x16x64_i8 v[192:195], v[112:115], v[28:31], v[192:195]\n" \
  "ds_read_b64_tr_b8 v[126:127], %[wr3] offset:25600\n" \
  "v_mfma_i32_16x16x64_i8 v[196:199], v[68:71], v[16:19], v[196:199]\n" \
  "ds_read_b128 v[32:35], %[xc0] offset:2048\n" \
  "v_mfma_i32_16x16x64_i8 v[196:199], v[84:87], v[20:23], v[196:199]\n" \
  "ds_read_b128 v[36:39], %[xc1] offset:2048\n" \
  "s_mov_b32 %[sO], 0x380000\n" \
  "buffer_store_dwordx4 v[188:191], %[cv], %[rc], %[sO] offen offset:128 sc1\n" \
  "v_mfma_i32_16x16x64_i8 v[196:199], v[100:103], v[24:27], v[196:199]\n" \
  "ds_read_b128 v[40:43], %[xc0] offset:18432\n" \
  "v_mfma_i32_16x16x64_i8 v[196:199], v[116:119], v[28:31], v[196:199]\n" \
  "ds_read_b128 v[44:47], %[xc1] offset:18432\n" \
  "v_mfma_i32_16x16x64_i8 v[200:203], v[72:75], v[16:19], v[200:203]\n" \
  "ds_read_b128 v[48:51], %[xc0] offset:4096\n" \
  "s_waitcnt lgkmcnt(9)\n" \
  "v_mfma_i32_16x16x64_i8 v[200:203], v[88:91], v[20:23], v[200:203]\n" \
  "ds_read_b128 v[52:55], %[xc1] offset:4096\n" \
  "v_mfma_i32_16x16x64_i8 v[200:203], v[104:107], v[24:27], v[200:203]\n" \
  "ds_read_b128 v[56:59], %[xc0] offset:20480\n" \
  "v_mfma_i32_16x16x64_i8 v[200:203], v[120:123], v[28:31], v[200:203]\n" \
  "ds_read_b128 v[60:63], %[xc1] offset:20480\n" \
  "v_mfma_i32_16x16x64_i8 v[204:207], v[76:79], v[16:19], v[204:207]\n" \
  "v_mfma_i32_16x16x64_i8 v[204:207], v[92:95], v[20:23], v[204:207]\n" \
  "s_waitcnt lgkmcnt(7)\n" \
  "v_mfma_i32_16x16x64_i8 v[204:207], v[108:111], v[24:27], v[204:207]\n" \
  "v_mfma_i32_16x16x64_i8 v[204:207], v[124:127], v[28:31], v[204:207]\n" \
  "v_mfma_i32_16x16x64_i8 v[208:211], v[64:67], v[32:35], v[208:211]\n" \
  "ds_read_b128 v[16:19], %[xc0] offset:6144\n" \
  "v_cvt_f32_i32 v192, v192\n" \
  "v_cvt_f32_i32 v196, v196\n" \
  "s_waitcnt lgkmcnt(5)\n" \
  "v_mfma_i32_16x16x64_i8 v[208:211], v[80:83], v[36:39], v[208:211]\n" \
  "ds_read_b128 v[20:23], %[xc1] offset:6144\n" \
  "v_cvt_f32_i32 v193, v193\n" \
  "v_cvt_f32_i32 v197, v197\n" \
  "v_mfma_i32_16x16x64_i8 v[208:211], v[96:99], v[40:43], v[208:211]\n" \
  "ds_read_b128 v[24:27], %[xc0] offset:22528\n" \
  "v_mul_f32 %[t0], %[sS], v192\n" \
  "v_mul_f32 %[t1], %[sS], v196\n" \
  "v_mfma_i32_16x16x64_i8 v[208:211], v[112:115], v[44:47], v[208:211]\n" \
  "ds_read_b128 v[28:31], %[xc1] offset:22528\n" \
  "v_mul_f32 %[t2], %[sS], v193\n" \
  "v_mul_f32 %[t3], %[sS], v197\n" \
  "v_mfma_i32_16x16x64_i8 v[212:215], v[116:119], v[44:47], v[212:215]\n" \
  "v_max_f32 v192, %[t0], %[vZ]\n" \
  "v_max_f32 v196, %[t1], %[vZ]\n" \
  "v_max_f32 v193, %[t2], %[vZ]\n" \
  "v_max_f32 v197, %[t3], %[vZ]\n" \
  "v_mfma_i32_16x16x64_i8 v[212:215], v[100:103], v[40:43], v[212:215]\n" \
  "v_max_f32_dpp v192, %[t1], %[vZ] row_ror:8 row_mask:0xf bank_mask:0xc\n" \
  "v_max_f32_dpp v196, %[t0], %[vZ] row_ror:8 row_mask:0xf bank_mask:0x3\n" \
  "v_max_f32_dpp v193, %[t3], %[vZ] row_ror:8 row_mask:0xf bank_mask:0xc\n" \
  "v_max_f32_dpp v197, %[t2], %[vZ] row_ror:8 row_mask:0xf bank_mask:0x3\n" \
  "v_mfma_i32_16x16x64_i8 v[212:215], v[84:87], v[36:39], v[212:215]\n" \
  "v_cvt_f32_i32 v194, v194\n" \
  "v_cvt_f32_i32 v198, v198\n" \
  "v_cvt_f32_i32 v195, v195\n" \
  "v_cvt_f32_i32 v199, v199\n" \
  "v_mfma_i32_16x16x64_i8 v[212:215], v[68:71], v[32:35], v[212:215]\n" \
  "v_mul_f32 %[t0], %[sS], v194\n" \
  "v_mul_f32 %[t1], %[sS], v198\n" \
  "v_mul_f32 %[t2], %[sS], v195\n" \
  "v_mul_f32 %[t3], %[sS], v199\n" \
  "v_mfma_i32_16x16x64_i8 v[216:219], v[72:75], v[32:35], v[216:219]\n" \
  "v_max_f32 v194, %[t0], %[vZ]\n" \
  "v_max_f32 v198, %[t1], %[vZ]\n" \
  "v_max_f32 v195, %[t2], %[vZ]\n" \
  "v_max_f32 v199, %[t3], %[vZ]\n" \
  "v_mfma_i32_16x16x64_i8 v[216:219], v[88:91], v[36:39], v[216:219]\n" \
  "v_max_f32_dpp v194, %[t1], %[vZ] row_ror:8 row_mask:0xf bank_mask:0xc\n" \
  "v_max_f32_dpp v198, %[t0], %[vZ] row_ror:8 row_mask:0xf bank_mask:0x3\n" \
  "v_max_f32_dpp v195, %[t3], %[vZ] row_ror:8 row_mask:0xf bank_mask:0xc\n" \
  "v_max_f32_dpp v199, %[t2], %[vZ] row_ror:8 row_mask:0xf bank_mask:0x3\n" \
  "v_mfma_i32_16x16x64_i8 v[216:219], v[104:107], v[40:43], v[216:219]\n" \
  "v_cvt_f32_i32 v200, v200\n" \
  "v_cvt_f32_i32 v204, v204\n" \
  "v_cvt_f32_i32 v201, v201\n" \
  "v_cvt_f32_i32 v205, v205\n" \
  "v_mfma_i32_16x16x64_i8 v[216:219], v[120:123], v[44:47], v[216:219]\n" \
  "v_mul_f32 %[t0], %[sS], v200\n" \
  "v_mul_f32 %[t1], %[sS], v204\n" \
  "v_mul_f32 %[t2], %[sS], v201\n" \
  "v_mul_f32 %[t3], %[sS], v205\n" \
  "v_mfma_i32_16x16x64_i8 v[220:223], v[124:127], v[44:47], v[220:223]\n" \
  "v_max_f32 v200, %[t0], %[vZ]\n" \
  "v_max_f32 v204, %[t1], %[vZ]\n" \
  "v_max_f32 v201, %[t2], %[vZ]\n" \
  "v_max_f32 v205, %[t3], %[vZ]\n" \
  "v_mfma_i32_16x16x64_i8 v[220:223], v[108:111], v[40:43], v[220:223]\n" \
  "v_max_f32_dpp v200, %[t1], %[vZ] row_ror:8 row_mask:0xf bank_mask:0xc\n" \
  "v_max_f32_dpp v204, %[t0], %[vZ] row_ror:8 row_mask:0xf bank_mask:0x3\n" \
  "v_max_f32_dpp v201, %[t3], %[vZ] row_ror:8 row_mask:0xf bank_mask:0xc\n" \
  "v_max_f32_dpp v205, %[t2], %[vZ] row_ror:8 row_mask:0xf bank_mask:0x3\n" \
  "v_mfma_i32_16x16x64_i8 v[220:223], v[92:95], v[36:39], v[220:223]\n" \
  "v_cvt_f32_i32 v202, v202\n" \
  "v_cvt_f32_i32 v206, v206\n" \
  "v_cvt_f32_i32 v203, v203\n" \
  "v_cvt_f32_i32 v207, v207\n" \
  "v_mfma_i32_16x16x64_i8 v[220:223], v[76:79], v[32:35], v[220:223]\n" \
  "v_mul_f32 %[t0], %[sS], v202\n" \
  "v_mul_f32 %[t1], %[sS], v206\n" \
  "v_mul_f32 %[t2], %[sS], v203\n" \
  "v_mul_f32 %[t3], %[sS], v207\n" \
  "s_waitcnt lgkmcnt(0)\n" \
  "v_mfma_i32_16x16x64_i8 v[224:227], v[64:67], v[48:51], v[224:227]\n" \
  "v_max_f32 v202, %[t0], %[vZ]\n" \
  "v_max_f32 v206, %[t1], %[vZ]\n" \
  "v_max_f32 v203, %[t2], %[vZ]\n" \
  "v_max_f32 v207, %[t3], %[vZ]\n" \
  "v_mfma_i32_16x16x64_i8 v[224:227], v[80:83], v[52:55], v[224:227]\n" \
  "v_max_f32_dpp v202, %[t1], %[vZ] row_ror:8 row_mask:0xf bank_mask:0xc\n" \
  "v_max_f32_dpp v206, %[t0], %[vZ] row_ror:8 row_mask:0xf bank_mask:0x3\n" \
  "v_max_f32_dpp v203, %[t3], %[vZ] row_ror:8 row_mask:0xf bank_mask:0xc\n" \
  "v_max_f32_dpp v207, %[t2], %[vZ] row_ror:8 row_mask:0xf bank_mask:0x3\n" \
  "s_mov_b32 %[sO], 0x800000\n" \
  "buffer_store_dwordx4 v[192:195], %[cv], %[rc], %[sO] offen offset:0 sc1\n" \
  "v_mfma_i32_16x16x64_i8 v[224:227], v[96:99], v[56:59], v[224:227]\n" \
  "v_cvt_f32_i32 v208, v208\n" \
  "v_cvt_f32_i32 v212, v212\n" \
  "v_cvt_f32_i32 v209, v209\n" \
  "v_cvt_f32_i32 v213, v213\n" \
  "v_mfma_i32_16x16x64_i8 v[224:227], v[112:115], v[60:63], v[224:227]\n" \
  "v_mul_f32 %[t0], %[sS], v208\n" \
  "v_mul_f32 %[t1], %[sS], v212\n" \
  "v_mul_f32 %[t2], %[sS], v209\n" \
  "v_mul_f32 %[t3], %[sS], v213\n" \
  "v_mfma_i32_16x16x64_i8 v[228:231], v[116:119], v[60:63], v[228:231]\n" \
  "v_max_f32 v208, %[t0], %[vZ]\n" \
  "v_max_f32 v212, %[t1], %[vZ]\n" \
  "v_max_f32 v209, %[t2], %[vZ]\n" \
  "v_max_f32 v213, %[t3], %[vZ]\n" \
  "v_mfma_i32_16x16x64_i8 v[228:231], v[100:103], v[56:59], v[228:231]\n" \
  "v_max_f32_dpp v208, %[t1], %[vZ] row_ror:8 row_mask:0xf bank_mask:0xc\n" \
  "v_max_f32_dpp v212, %[t0], %[vZ] row_ror:8 row_mask:0xf bank_mask:0x3\n" \
  "v_max_f32_dpp v209, %[t3], %[vZ] row_ror:8 row_mask:0xf bank_mask:0xc\n" \
  "v_max_f32_dpp v213, %[t2], %[vZ] row_ror:8 row_mask:0xf bank_mask:0x3\n" \
  "v_mfma_i32_16x16x64_i8 v[228:231], v[84:87], v[52:55], v[228:231]\n" \
  "v_cvt_f32_i32 v210, v210\n" \
  "v_cvt_f32_i32 v214, v214\n" \
  "v_cvt_f32_i32 v211, v211\n" \
  "v_cvt_f32_i32 v215, v215\n" \
  "v_mfma_i32_16x16x64_i8 v[228:231], v[68:71], v[48:51], v[228:231]\n" \
  "v_mul_f32 %[t0], %[sS], v210\n" \
  "v_mul_f32 %[t1], %[sS], v214\n" \
  "v_mul_f32 %[t2], %[sS], v211\n" \
  "v_mul_f32 %[t3], %[sS], v215\n" \
  "v_mfma_i32_16x16x64_i8 v[232:235], v[72:75], v[48:51], v[232:235]\n" \
  "v_max_f32 v210, %[t0], %[vZ]\n" \
  "v_max_f32 v214, %[t1], %[vZ]\n" \
  "v_max_f32 v211, %[t2], %[vZ]\n" \
  "v_max_f32 v215, %[t3], %[vZ]\n" \
  "v_mfma_i32_16x16x64_i8 v[232:235], v[88:91], v[52:55], v[232:235]\n" \
  "v_max_f32_dpp v210, %[t1], %[vZ] row_ror:8 row_mask:0xf bank_mask:0xc\n" \
  "v_max_f32_dpp v214, %[t0], %[vZ] row_ror:8 row_mask:0xf bank_mask:0x3\n" \
  "v_max_f32_dpp v211, %[t3], %[vZ] row_ror:8 row_mask:0xf bank_mask:0xc\n" \
  "v_max_f32_dpp v215, %[t2], %[vZ] row_ror:8 row_mask:0xf bank_mask:0x3\n" \
  "v_mfma_i32_16x16x64_i8 v[232:235], v[104:107], v[56:59], v[232:235]\n" \
  "v_cvt_f32_i32 v216, v216\n" \
  "v_cvt_f32_i32 v220, v220\n" \
  "v_cvt_f32_i32 v217, v217\n" \
  "v_cvt_f32_i32 v221, v221\n" \
  "v_mfma_i32_16x16x64_i8 v[232:235], v[120:123], v[60:63], v[232:235]\n" \
  "v_mul_f32 %[t0], %[sS], v216\n" \
  "v_mul_f32 %[t1], %[sS], v220\n" \
  "v_mul_f32 %[t2], %[sS], v217\n" \
  "v_mul_f32 %[t3], %[sS], v221\n" \
  "v_mfma_i32_16x16x64_i8 v[236:239], v[124:127], v[60:63], v[236:239]\n" \
  "v_max_f32 v216, %[t0], %[vZ]\n" \
  "v_max_f32 v220, %[t1], %[vZ]\n" \
  "v_max_f32 v217, %[t2], %[vZ]\n" \
  "v_max_f32 v221, %[t3], %[vZ]\n" \
  "v_mfma_i32_16x16x64_i8 v[236:239], v[108:111], v[56:59], v[236:239]\n" \
  "s_mov_b32 %[sO], 0x800000\n" \
  "buffer_store_dwordx4 v[200:203], %[cv], %[rc], %[sO] offen offset:128 sc1\n" \
  "s_mov_b32 %[sO], 0x880000\n" \
  "buffer_store_dwordx4 v[196:199], %[cv], %[rc], %[sO] offen offset:0 sc1\n" \
  "v_mfma_i32_16x16x64_i8 v[236:239], v[92:95], v[52:55], v[236:239]\n" \
  "v_max_f32_dpp v216, %[t1], %[vZ] row_ror:8 row_mask:0xf bank_mask:0xc\n" \
  "v_max_f32_dpp v220, %[t0], %[vZ] row_ror:8 row_mask:0xf bank_mask:0x3\n" \
  "v_max_f32_dpp v217, %[t3], %[vZ] row_ror:8 row_mask:0xf bank_mask:0xc\n" \
  "v_max_f32_dpp v221, %[t2], %[vZ] row_ror:8 row_mask:0xf bank_mask:0x3\n" \
  "v_mfma_i32_16x16x64_i8 v[236:239], v[76:79], v[48:51], v[236:239]\n" \
  "v_cvt_f32_i32 v218, v218\n" \
  "v_cvt_f32_i32 v222, v222\n" \
  "v_cvt_f32_i32 v219, v219\n" \
  "v_cvt_f32_i32 v223, v223\n" \
  "v_mfma_i32_16x16x64_i8 v[240:243], v[64:67], v[16:19], v[240:243]\n" \
  "v_mul_f32 %[t0], %[sS], v218\n" \
  "v_mul_f32 %[t1], %[sS], v222\n" \
  "v_mul_f32 %[t2], %[sS], v219\n" \
  "v_mul_f32 %[t3], %[sS], v223\n" \
  "v_mfma_i32_16x16x64_i8 v[240:243], v[80:83], v[20:23], v[240:243]\n" \
  "v_max_f32 v218, %[t0], %[vZ]\n" \
  "v_max_f32 v222, %[t1], %[vZ]\n" \
  "v_max_f32 v219, %[t2], %[vZ]\n" \
  "v_max_f32 v223, %[t3], %[vZ]\n" \
  "v_mfma_i32_16x16x64_i8 v[240:243], v[96:99], v[24:27], v[240:243]\n" \
  "v_max_f32_dpp v218, %[t1], %[vZ] row_ror:8 row_mask:0xf bank_mask:0xc\n" \
  "v_max_f32_dpp v222, %[t0], %[vZ] row_ror:8 row_mask:0xf bank_mask:0x3\n" \
  "v_max_f32_dpp v219, %[t3], %[vZ] row_ror:8 row_mask:0xf bank_mask:0xc\n" \
  "v_max_f32_dpp v223, %[t2], %[vZ] row_ror:8 row_mask:0xf bank_mask:0x3\n" \
  "v_mfma_i32_16x16x64_i8 v[240:243], v[112:115], v[28:31], v[240:243]\n" \
  "v_cvt_f32_i32 v224, v224\n" \
  "v_cvt_f32_i32 v228, v228\n" \
  "v_cvt_f32_i32 v225, v225\n" \
  "v_cvt_f32_i32 v229, v229\n" \
  "v_mfma_i32_16x16x64_i8 v[244:247], v[116:119], v[28:31], v[244:247]\n" \
  "v_mul_f32 %[t0], %[sS], v224\n" \
  "v_mul_f32 %[t1], %[sS], v228\n" \
  "v_mul_f32 %[t2], %[sS], v225\n" \
  "v_mul_f32 %[t3], %[sS], v229\n" \
  "v_mfma_i32_16x16x64_i8 v[244:247], v[100:103], v[24:27], v[244:247]\n" \
  "v_max_f32 v224, %[t0], %[vZ]\n" \
  "v_max_f32 v228, %[t1], %[vZ]\n" \
  "v_max_f32 v225, %[t2], %[vZ]\n" \
  "v_max_f32 v229, %[t3], %[vZ]\n" \
  "v_mfma_i32_16x16x64_i8 v[244:247], v[84:87], v[20:23], v[244:247]\n" \
  "v_max_f32_dpp v224, %[t1], %[vZ] row_ror:8 row_mask:0xf bank_mask:0xc\n" \
  "v_max_f32_dpp v228, %[t0], %[vZ] row_ror:8 row_mask:0xf bank_mask:0x3\n" \
  "v_max_f32_dpp v225, %[t3], %[vZ] row_ror:8 row_mask:0xf bank_mask:0xc\n" \
  "v_max_f32_dpp v229, %[t2], %[vZ] row_ror:8 row_mask:0xf bank_mask:0x3\n" \
  "v_mfma_i32_16x16x64_i8 v[244:247], v[68:71], v[16:19], v[244:247]\n" \
  "v_cvt_f32_i32 v226, v226\n" \
  "v_cvt_f32_i32 v230, v230\n" \
  "v_cvt_f32_i32 v227, v227\n" \
  "v_cvt_f32_i32 v231, v231\n" \
  "v_mfma_i32_16x16x64_i8 v[248:251], v[72:75], v[16:19], v[248:251]\n" \
  "v_mul_f32 %[t0], %[sS], v226\n" \
  "v_mul_f32 %[t1], %[sS], v230\n" \
  "v_mul_f32 %[t2], %[sS], v227\n" \
  "v_mul_f32 %[t3], %[sS], v231\n" \
  "v_mfma_i32_16x16x64_i8 v[248:251], v[88:91], v[20:23], v[248:251]\n" \
  "s_mov_b32 %[sO], 0x880000\n" \
  "buffer_store_dwordx4 v[204:207], %[cv], %[rc], %[sO] offen offset:128 sc1\n" \
  "s_mov_b32 %[sO], 0x900000\n" \
  "buffer_store_dwordx4 v[208:211], %[cv], %[rc], %[sO] offen offset:0 sc1\n" \
  "v_mfma_i32_16x16x64_i8 v[248:251], v[104:107], v[24:27], v[248:251]\n" \
  "v_max_f32 v226, %[t0], %[vZ]\n" \
  "v_max_f32 v230, %[t1], %[vZ]\n" \
  "v_max_f32 v227, %[t2], %[vZ]\n" \
  "v_max_f32 v231, %[t3], %[vZ]\n" \
  "v_mfma_i32_16x16x64_i8 v[248:251], v[120:123], v[28:31], v[248:251]\n" \
  "v_max_f32_dpp v226, %[t1], %[vZ] row_ror:8 row_mask:0xf bank_mask:0xc\n" \
  "v_max_f32_dpp v230, %[t0], %[vZ] row_ror:8 row_mask:0xf bank_mask:0x3\n" \
  "v_max_f32_dpp v227, %[t3], %[vZ] row_ror:8 row_mask:0xf bank_mask:0xc\n" \
  "v_max_f32_dpp v231, %[t2], %[vZ] row_ror:8 row_mask:0xf bank_mask:0x3\n" \
  "v_mfma_i32_16x16x64_i8 v[252:255], v[124:127], v[28:31], v[252:255]\n" \
  "v_cvt_f32_i32 v232, v232\n" \
  "v_cvt_f32_i32 v236, v236\n" \
  "v_cvt_f32_i32 v233, v233\n" \
  "v_cvt_f32_i32 v237, v237\n" \
  "v_mfma_i32_16x16x64_i8 v[252:255], v[108:111], v[24:27], v[252:255]\n" \
  "v_mul_f32 %[t0], %[sS], v232\n" \
  "v_mul_f32 %[t1], %[sS], v236\n" \
  "v_mul_f32 %[t2], %[sS], v233\n" \
  "v_mul_f32 %[t3], %[sS], v237\n" \
  "v_mfma_i32_16x16x64_i8 v[252:255], v[92:95], v[20:23], v[252:255]\n" \
  "v_max_f32 v232, %[t0], %[vZ]\n" \
  "v_max_f32 v236, %[t1], %[vZ]\n" \
  "v_max_f32 v233, %[t2], %[vZ]\n" \
  "v_max_f32 v237, %[t3], %[vZ]\n" \
  "v_mfma_i32_16x16x64_i8 v[252:255], v[76:79], v[16:19], v[252:255]\n" \
  "v_max_f32_dpp v232, %[t1], %[vZ] row_ror:8 row_mask:0xf bank_mask:0xc\n" \
  "v_max_f32_dpp v236, %[t0], %[vZ] row_ror:8 row_mask:0xf bank_mask:0x3\n" \
  "v_max_f32_dpp v233, %[t3], %[vZ] row_ror:8 row_mask:0xf bank_mask:0xc\n" \
  "v_max_f32_dpp v237, %[t2], %[vZ] row_ror:8 row_mask:0xf bank_mask:0x3\n" \
  "s_nop 7\n" \
  "s_mov_b32 %[sO], 0x900000\n" \
  "buffer_store_dwordx4 v[216:219], %[cv], %[rc], %[sO] offen offset:128 sc1\n" \
  "s_mov_b32 %[sO], 0x980000\n" \
  "buffer_store_dwordx4 v[212:215], %[cv], %[rc], %[sO] offen offset:0 sc1\n" \
  "s_mov_b32 %[sO], 0x980000\n" \
  "buffer_store_dwordx4 v[220:223], %[cv], %[rc], %[sO] offen offset:128 sc1\n" \
  "v_cvt_f32_i32 v234, v234\n" \
  "v_cvt_f32_i32 v238, v238\n" \
  "v_cvt_f32_i32 v235, v235\n" \
  "v_cvt_f32_i32 v239, v239\n" \
  "v_mul_f32 %[t0], %[sS], v234\n" \
  "v_mul_f32 %[t1], %[sS], v238\n" \
  "v_mul_f32 %[t2], %[sS], v235\n" \
  "v_mul_f32 %[t3], %[sS], v239\n" \
  "v_max_f32 v234, %[t0], %[vZ]\n" \
  "v_max_f32 v238, %[t1], %[vZ]\n" \
  "v_max_f32 v235, %[t2], %[vZ]\n" \
  "v_max_f32 v239, %[t3], %[vZ]\n" \
  "v_max_f32_dpp v234, %[t1], %[vZ] row_ror:8 row_mask:0xf bank_mask:0xc\n" \
  "v_max_f32_dpp v238, %[t0], %[vZ] row_ror:8 row_mask:0xf bank_mask:0x3\n" \
  "v_max_f32_dpp v235, %[t3], %[vZ] row_ror:8 row_mask:0xf bank_mask:0xc\n" \
  "v_max_f32_dpp v239, %[t2], %[vZ] row_ror:8 row_mask:0xf bank_mask:0x3\n" \
  "s_mov_b32 %[sO], 0xa00000\n" \
  "buffer_store_dwordx4 v[224:227], %[cv], %[rc], %[sO] offen offset:0 sc1\n" \
  "s_mov_b32 %[sO], 0xa00000\n" \
  "buffer_store_dwordx4 v[232:235], %[cv], %[rc], %[sO] offen offset:128 sc1\n" \
  "s_mov_b32 %[sO], 0xa80000\n" \
  "buffer_store_dwordx4 v[228:231], %[cv], %[rc], %[sO] offen offset:0 sc1\n" \
  "s_mov_b32 %[sO], 0xa80000\n" \
  "buffer_store_dwordx4 v[236:239], %[cv], %[rc], %[sO] offen offset:128 sc1\n" \
  "v_cvt_f32_i32 v240, v240\n" \
  "v_cvt_f32_i32 v244, v244\n" \
  "v_cvt_f32_i32 v241, v241\n" \
  "v_cvt_f32_i32 v245, v245\n" \
  "v_mul_f32 %[t0], %[sS], v240\n" \
  "v_mul_f32 %[t1], %[sS], v244\n" \
  "v_mul_f32 %[t2], %[sS], v241\n" \
  "v_mul_f32 %[t3], %[sS], v245\n" \
  "v_max_f32 v240, %[t0], %[vZ]\n" \
  "v_max_f32 v244, %[t1], %[vZ]\n" \
  "v_max_f32 v241, %[t2], %[vZ]\n" \
  "v_max_f32 v245, %[t3], %[vZ]\n" \
  "v_max_f32_dpp v240, %[t1], %[vZ] row_ror:8 row_mask:0xf bank_mask:0xc\n" \
  "v_max_f32_dpp v244, %[t0], %[vZ] row_ror:8 row_mask:0xf bank_mask:0x3\n" \
  "v_max_f32_dpp v241, %[t3], %[vZ] row_ror:8 row_mask:0xf bank_mask:0xc\n" \
  "v_max_f32_dpp v245, %[t2], %[vZ] row_ror:8 row_mask:0xf bank_mask:0x3\n" \
  "v_cvt_f32_i32 v242, v242\n" \
  "v_cvt_f32_i32 v246, v246\n" \
  "v_cvt_f32_i32 v243, v243\n" \
  "v_cvt_f32_i32 v247, v247\n" \
  "v_mul_f32 %[t0], %[sS], v242\n" \
  "v_mul_f32 %[t1], %[sS], v246\n" \
  "v_mul_f32 %[t2], %[sS], v243\n" \
  "v_mul_f32 %[t3], %[sS], v247\n" \
  "v_max_f32 v242, %[t0], %[vZ]\n" \
  "v_max_f32 v246, %[t1], %[vZ]\n" \
  "v_max_f32 v243, %[t2], %[vZ]\n" \
  "v_max_f32 v247, %[t3], %[vZ]\n" \
  "v_max_f32_dpp v242, %[t1], %[vZ] row_ror:8 row_mask:0xf bank_mask:0xc\n" \
  "v_max_f32_dpp v246, %[t0], %[vZ] row_ror:8 row_mask:0xf bank_mask:0x3\n" \
  "v_max_f32_dpp v243, %[t3], %[vZ] row_ror:8 row_mask:0xf bank_mask:0xc\n" \
  "v_max_f32_dpp v247, %[t2], %[vZ] row_ror:8 row_mask:0xf bank_mask:0x3\n" \
  "v_cvt_f32_i32 v248, v248\n" \
  "v_cvt_f32_i32 v252, v252\n" \
  "v_cvt_f32_i32 v249, v249\n" \
  "v_cvt_f32_i32 v253, v253\n" \
  "v_mul_f32 %[t0], %[sS], v248\n" \
  "v_mul_f32 %[t1], %[sS], v252\n" \
  "v_mul_f32 %[t2], %[sS], v249\n" \
  "v_mul_f32 %[t3], %[sS], v253\n" \
  "v_max_f32 v248, %[t0], %[vZ]\n" \
  "v_max_f32 v252, %[t1], %[vZ]\n" \
  "v_max_f32 v249, %[t2], %[vZ]\n" \
  "v_max_f32 v253, %[t3], %[vZ]\n" \
  "v_max_f32_dpp v248, %[t1], %[vZ] row_ror:8 row_mask:0xf bank_mask:0xc\n" \
  "v_max_f32_dpp v252, %[t0], %[vZ] row_ror:8 row_mask:0xf bank_mask:0x3\n" \
  "v_max_f32_dpp v249, %[t3], %[vZ] row_ror:8 row_mask:0xf bank_mask:0xc\n" \
  "v_max_f32_dpp v253, %[t2], %[vZ] row_ror:8 row_mask:0xf bank_mask:0x3\n" \
  "v_cvt_f32_i32 v250, v250\n" \
  "v_cvt_f32_i32 v254, v254\n" \
  "v_cvt_f32_i32 v251, v251\n" \
  "v_cvt_f32_i32 v255, v255\n" \
  "v_mul_f32 %[t0], %[sS], v250\n" \
  "v_mul_f32 %[t1], %[sS], v254\n" \
  "v_mul_f32 %[t2], %[sS], v251\n" \
  "v_mul_f32 %[t3], %[sS], v255\n" \
  "v_max_f32 v250, %[t0], %[vZ]\n" \
  "v_max_f32 v254, %[t1], %[vZ]\n" \
  "v_max_f32 v251, %[t2], %[vZ]\n" \
  "v_max_f32 v255, %[t3], %[vZ]\n" \
  "v_max_f32_dpp v250, %[t1], %[vZ] row_ror:8 row_mask:0xf bank_mask:0xc\n" \
  "v_max_f32_dpp v254, %[t0], %[vZ] row_ror:8 row_mask:0xf bank_mask:0x3\n" \
  "v_max_f32_dpp v251, %[t3], %[vZ] row_ror:8 row_mask:0xf bank_mask:0xc\n" \
  "v_max_f32_dpp v255, %[t2], %[vZ] row_ror:8 row_mask:0xf bank_mask:0x3\n" \
  "s_mov_b32 %[sO], 0xb00000\n" \
  "buffer_store_dwordx4 v[240:243], %[cv], %[rc], %[sO] offen offset:0 sc1\n" \
  "s_mov_b32 %[sO], 0xb00000\n" \
  "buffer_store_dwordx4 v[248:251], %[cv], %[rc], %[sO] offen offset:128 sc1\n" \
  "s_mov_b32 %[sO], 0xb80000\n" \
  "buffer_store_dwordx4 v[244:247], %[cv], %[rc], %[sO] offen offset:0 sc1\n" \
  "s_mov_b32 %[sO], 0xb80000\n" \
  "buffer_store_dwordx4 v[252:255], %[cv], %[rc], %[sO] offen offset:128 sc1\n" \
  ""
#define ALOAD1_ASM \
  "s_nop 3\n" \
  "s_add_u32 m0, %[sM], 0\n" \
  "s_nop 0\n" \
  "buffer_load_dwordx4 %[va0], %[rs1], %[sO] offen offset:0 lds\n" \
  "buffer_load_dwordx4 %[va1], %[rs1], %[sO] offen offset:1024 lds\n" \
  "buffer_load_dwordx4 %[va2], %[rs1], %[sO] offen offset:2048 lds\n" \
  "buffer_load_dwordx4 %[va3], %[rs1], %[sO] offen offset:3072 lds\n" \
  "s_add_u32 m0, %[sM], 4096\n" \
  "s_add_u32 %[sO], %[sO], 0x4000\n" \
  "buffer_load_dwordx4 %[va0], %[rs1], %[sO] offen offset:0 lds\n" \
  "buffer_load_dwordx4 %[va1], %[rs1], %[sO] offen offset:1024 lds\n" \
  "buffer_load_dwordx4 %[va2], %[rs1], %[sO] offen offset:2048 lds\n" \
  "buffer_load_dwordx4 %[va3], %[rs1], %[sO] offen offset:3072 lds\n" \
  "s_add_u32 m0, %[sM], 8192\n" \
  "s_add_u32 %[sO], %[sO], 0x4000\n" \
  "buffer_load_dwordx4 %[va0], %[rs1], %[sO] offen offset:0 lds\n" \
  "buffer_load_dwordx4 %[va1], %[rs1], %[sO] offen offset:1024 lds\n" \
  "buffer_load_dwordx4 %[va2], %[rs1], %[sO] offen offset:2048 lds\n" \
  "buffer_load_dwordx4 %[va3], %[rs1], %[sO] offen offset:3072 lds\n" \
  "s_add_u32 m0, %[sM], 12288\n" \
  "s_add_u32 %[sO], %[sO], 0x4000\n" \
  "buffer_load_dwordx4 %[va0], %[rs1], %[sO] offen offset:0 lds\n" \
  "buffer_load_dwordx4 %[va1], %[rs1], %[sO] offen offset:1024 lds\n" \
  "buffer_load_dwordx4 %[va2], %[rs1], %[sO] offen offset:2048 lds\n" \
  "buffer_load_dwordx4 %[va3], %[rs1], %[sO] offen offset:3072 lds\n" \
  "s_add_u32 %[sM], %[sM], 0x8000\n" \
  "s_add_u32 %[sO], %[sO], 0x4000\n" \
  "s_add_u32 m0, %[sM], 0\n" \
  "s_nop 0\n" \
  "buffer_load_dwordx4 %[va0], %[rs1], %[sO] offen offset:0 lds\n" \
  "buffer_load_dwordx4 %[va1], %[rs1], %[sO] offen offset:1024 lds\n" \
  "buffer_load_dwordx4 %[va2], %[rs1], %[sO] offen offset:2048 lds\n" \
  "buffer_load_dwordx4 %[va3], %[rs1], %[sO] offen offset:3072 lds\n" \
  "s_add_u32 m0, %[sM], 4096\n" \
  "s_add_u32 %[sO], %[sO], 0x4000\n" \
  "buffer_load_dwordx4 %[va0], %[rs1], %[sO] offen offset:0 lds\n" \
  "buffer_load_dwordx4 %[va1], %[rs1], %[sO] offen offset:1024 lds\n" \
  "buffer_load_dwordx4 %[va2], %[rs1], %[sO] offen offset:2048 lds\n" \
  "buffer_load_dwordx4 %[va3], %[rs1], %[sO] offen offset:3072 lds\n" \
  "s_add_u32 m0, %[sM], 8192\n" \
  "s_add_u32 %[sO], %[sO], 0x4000\n" \
  "buffer_load_dwordx4 %[va0], %[rs1], %[sO] offen offset:0 lds\n" \
  "buffer_load_dwordx4 %[va1], %[rs1], %[sO] offen offset:1024 lds\n" \
  "buffer_load_dwordx4 %[va2], %[rs1], %[sO] offen offset:2048 lds\n" \
  "buffer_load_dwordx4 %[va3], %[rs1], %[sO] offen offset:3072 lds\n" \
  "s_add_u32 m0, %[sM], 12288\n" \
  "s_add_u32 %[sO], %[sO], 0x4000\n" \
  "buffer_load_dwordx4 %[va0], %[rs1], %[sO] offen offset:0 lds\n" \
  "buffer_load_dwordx4 %[va1], %[rs1], %[sO] offen offset:1024 lds\n" \
  "buffer_load_dwordx4 %[va2], %[rs1], %[sO] offen offset:2048 lds\n" \
  "buffer_load_dwordx4 %[va3], %[rs1], %[sO] offen offset:3072 lds\n" \
  ""
#define ALOAD2_ASM \
  "s_mov_b64 s[86:87], %[sWl]\n" \
  "s_mov_b32 %[sI], 2\n" \
  "s_waitcnt vmcnt(16)\n" \
  "s_barrier\n" \
  "s_add_u32 m0, %[sM], 0\n" \
  "s_add_u32 %[sI], %[sI], 1\n" \
  "buffer_load_dwordx4 %[va0], %[rs1], %[sO] offen offset:0 lds\n" \
  "buffer_load_dwordx4 %[va1], %[rs1], %[sO] offen offset:1024 lds\n" \
  "buffer_load_dwordx4 %[va2], %[rs1], %[sO] offen offset:2048 lds\n" \
  "buffer_load_dwordx4 %[va3], %[rs1], %[sO] offen offset:3072 lds\n" \
  "v_readlane_b32 %[sP], %[vtab], %[sI]\n" \
  "s_add_u32 m0, %[sM], 4096\n" \
  "s_add_u32 %[sO], %[sO], 0x4000\n" \
  "buffer_load_dwordx4 %[va0], %[rs1], %[sO] offen offset:0 lds\n" \
  "buffer_load_dwordx4 %[va1], %[rs1], %[sO] offen offset:1024 lds\n" \
  "buffer_load_dwordx4 %[va2], %[rs1], %[sO] offen offset:2048 lds\n" \
  "buffer_load_dwordx4 %[va3], %[rs1], %[sO] offen offset:3072 lds\n" \
  "s_add_u32 m0, %[sM], 8192\n" \
  "s_add_u32 %[sO], %[sO], 0x4000\n" \
  "buffer_load_dwordx4 %[va0], %[rs1], %[sO] offen offset:0 lds\n" \
  "buffer_load_dwordx4 %[va1], %[rs1], %[sO] offen offset:1024 lds\n" \
  "buffer_load_dwordx4 %[va2], %[rs1], %[sO] offen offset:2048 lds\n" \
  "buffer_load_dwordx4 %[va3], %[rs1], %[sO] offen offset:3072 lds\n" \
  "s_add_u32 %[sX], %[sM], 0x8000\n" \
  "s_sub_u32 %[sY], %[sM], 0x10000\n" \
  "s_cmp_ge_u32 %[sM], 0x20000\n" \
  "s_cselect_b32 %[sQ], %[sY], %[sX]\n" \
  "s_add_u32 m0, %[sM], 12288\n" \
  "s_add_u32 %[sO], %[sO], 0x4000\n" \
  "buffer_load_dwordx4 %[va0], %[rs1], %[sO] offen offset:0 lds\n" \
  "buffer_load_dwordx4 %[va1], %[rs1], %[sO] offen offset:1024 lds\n" \
  "buffer_load_dwordx4 %[va2], %[rs1], %[sO] offen offset:2048 lds\n" \
  "buffer_load_dwordx4 %[va3], %[rs1], %[sO] offen offset:3072 lds\n" \
  "s_mov_b32 %[sM], %[sQ]\n" \
  "s_mov_b32 %[sO], %[sP]\n" \
  "s_waitcnt vmcnt(16)\n" \
  "s_barrier\n" \
  "s_add_u32 m0, %[sM], 0\n" \
  "s_add_u32 %[sI], %[sI], 1\n" \
  "buffer_load_dwordx4 %[va0], %[rs1], %[sO] offen offset:0 lds\n" \
  "buffer_load_dwordx4 %[va1], %[rs1], %[sO] offen offset:1024 lds\n" \
  "buffer_load_dwordx4 %[va2], %[rs1], %[sO] offen offset:2048 lds\n" \
  "buffer_load_dwordx4 %[va3], %[rs1], %[sO] offen offset:3072 lds\n" \
  "v_readlane_b32 %[sP], %[vtab], %[sI]\n" \
  "s_add_u32 m0, %[sM], 4096\n" \
  "s_add_u32 %[sO], %[sO], 0x4000\n" \
  "buffer_load_dwordx4 %[va0], %[rs1], %[sO] offen offset:0 lds\n" \
  "buffer_load_dwordx4 %[va1], %[rs1], %[sO] offen offset:1024 lds\n" \
  "buffer_load_dwordx4 %[va2], %[rs1], %[sO] offen offset:2048 lds\n" \
  "buffer_load_dwordx4 %[va3], %[rs1], %[sO] offen offset:3072 lds\n" \
  "s_add_u32 m0, %[sM], 8192\n" \
  "s_add_u32 %[sO], %[sO], 0x4000\n" \
  "buffer_load_dwordx4 %[va0], %[rs1], %[sO] offen offset:0 lds\n" \
  "buffer_load_dwordx4 %[va1], %[rs1], %[sO] offen offset:1024 lds\n" \
  "buffer_load_dwordx4 %[va2], %[rs1], %[sO] offen offset:2048 lds\n" \
  "buffer_load_dwordx4 %[va3], %[rs1], %[sO] offen offset:3072 lds\n" \
  "s_add_u32 %[sX], %[sM], 0x8000\n" \
  "s_sub_u32 %[sY], %[sM], 0x10000\n" \
  "s_cmp_ge_u32 %[sM], 0x20000\n" \
  "s_cselect_b32 %[sQ], %[sY], %[sX]\n" \
  "s_add_u32 m0, %[sM], 12288\n" \
  "s_add_u32 %[sO], %[sO], 0x4000\n" \
  "buffer_load_dwordx4 %[va0], %[rs1], %[sO] offen offset:0 lds\n" \
  "buffer_load_dwordx4 %[va1], %[rs1], %[sO] offen offset:1024 lds\n" \
  "buffer_load_dwordx4 %[va2], %[rs1], %[sO] offen offset:2048 lds\n" \
  "buffer_load_dwordx4 %[va3], %[rs1], %[sO] offen offset:3072 lds\n" \
  "s_mov_b32 %[sM], %[sQ]\n" \
  "s_mov_b32 %[sO], %[sP]\n" \
  "s_waitcnt vmcnt(16)\n" \
  "s_barrier\n" \
  "s_memrealtime s[80:81]\n" \
  "s_mov_b64 exec, %[sEx]\n" \
  "Laspin_%=:\n" \
  "buffer_load_dword v248, %[vfo], %[rf], 0 offen sc0 sc1\n" \
  "s_memrealtime s[82:83]\n" \
  "s_waitcnt vmcnt(0) lgkmcnt(0)\n" \
  "v_cmpx_gt_u32 vcc, %[sCall], v248\n" \
  "s_cbranch_execz Laok_%=\n" \
  "s_sub_u32 %[sX], s82, s80\n" \
  "s_cmp_lt_u32 %[sX], 0x1c9c380\n" \
  "s_sleep 4\n" \
  "s_cbranch_scc1 Laspin_%=\n" \
  "v_mov_b32 v249, %[sCall]\n" \
  "s_mov_b32 %[sX], 0x20000\n" \
  "buffer_store_dword v249, off, %[rf], %[sX] sc1\n" \
  "s_waitcnt vmcnt(0)\n" \
  "Laok_%=:\n" \
  "s_mov_b64 exec, -1\n" \
  "Laloop_%=:\n" \
  "s_add_u32 m0, %[sM], 0\n" \
  "s_add_u32 %[sI], %[sI], 1\n" \
  "buffer_load_dwordx4 %[va0], %[rs2], %[sO] offen offset:0 sc0 sc1 lds\n" \
  "buffer_load_dwordx4 %[va1], %[rs2], %[sO] offen offset:1024 sc0 sc1 lds\n" \
  "buffer_load_dwordx4 %[va2], %[rs2], %[sO] offen offset:2048 sc0 sc1 lds\n" \
  "buffer_load_dwordx4 %[va3], %[rs2], %[sO] offen offset:3072 sc0 sc1 lds\n" \
  "v_readlane_b32 %[sP], %[vtab], %[sI]\n" \
  "s_add_u32 m0, %[sM], 4096\n" \
  "s_add_u32 %[sO], %[sO], 0x4000\n" \
  "buffer_load_dwordx4 %[va0], %[rs2], %[sO] offen offset:0 sc0 sc1 lds\n" \
  "buffer_load_dwordx4 %[va1], %[rs2], %[sO] offen offset:1024 sc0 sc1 lds\n" \
  "buffer_load_dwordx4 %[va2], %[rs2], %[sO] offen offset:2048 sc0 sc1 lds\n" \
  "buffer_load_dwordx4 %[va3], %[rs2], %[sO] offen offset:3072 sc0 sc1 lds\n" \
  "s_add_u32 m0, %[sM], 8192\n" \
  "s_add_u32 %[sO], %[sO], 0x4000\n" \
  "buffer_load_dwordx4 %[va0], %[rs2], %[sO] offen offset:0 sc0 sc1 lds\n" \
  "buffer_load_dwordx4 %[va1], %[rs2], %[sO] offen offset:1024 sc0 sc1 lds\n" \
  "buffer_load_dwordx4 %[va2], %[rs2], %[sO] offen offset:2048 sc0 sc1 lds\n" \
  "buffer_load_dwordx4 %[va3], %[rs2], %[sO] offen offset:3072 sc0 sc1 lds\n" \
  "s_add_u32 %[sX], %[sM], 0x8000\n" \
  "s_sub_u32 %[sY], %[sM], 0x10000\n" \
  "s_cmp_ge_u32 %[sM], 0x20000\n" \
  "s_cselect_b32 %[sQ], %[sY], %[sX]\n" \
  "s_add_u32 m0, %[sM], 12288\n" \
  "s_add_u32 %[sO], %[sO], 0x4000\n" \
  "buffer_load_dwordx4 %[va0], %[rs2], %[sO] offen offset:0 sc0 sc1 lds\n" \
  "buffer_load_dwordx4 %[va1], %[rs2], %[sO] offen offset:1024 sc0 sc1 lds\n" \
  "buffer_load_dwordx4 %[va2], %[rs2], %[sO] offen offset:2048 sc0 sc1 lds\n" \
  "buffer_load_dwordx4 %[va3], %[rs2], %[sO] offen offset:3072 sc0 sc1 lds\n" \
  "s_mov_b32 %[sM], %[sQ]\n" \
  "s_mov_b32 %[sO], %[sP]\n" \
  "s_cmp_lt_u32 %[sI], 32\n" \
  "s_waitcnt vmcnt(16)\n" \
  "s_barrier\n" \
  "s_cbranch_scc1 Laloop_%=\n" \
  "s_waitcnt vmcnt(0) lgkmcnt(0)\n" \
  "s_barrier\n" \
  ""
#define WLOAD1_ASM \
  "s_nop 3\n" \
  "s_mov_b32 m0, 0x0\n" \
  "s_add_u32 %[sO], %[sP0], 0x0\n" \
  "buffer_load_dwordx4 %[va0], %[rs], %[sO] offen offset:0 nt lds\n" \
  "buffer_load_dwordx4 %[va1], %[rs], %[sO] offen offset:1024 nt lds\n" \
  "buffer_load_dwordx4 %[va2], %[rs], %[sO] offen offset:2048 nt lds\n" \
  "buffer_load_dwordx4 %[va3], %[rs], %[sO] offen offset:3072 nt lds\n" \
  "s_mov_b32 m0, 0x1000\n" \
  "s_add_u32 %[sO], %[sP0], 0x80000\n" \
  "buffer_load_dwordx4 %[va0], %[rs], %[sO] offen offset:0 nt lds\n" \
  "buffer_load_dwordx4 %[va1], %[rs], %[sO] offen offset:1024 nt lds\n" \
  "buffer_load_dwordx4 %[va2], %[rs], %[sO] offen offset:2048 nt lds\n" \
  "buffer_load_dwordx4 %[va3], %[rs], %[sO] offen offset:3072 nt lds\n" \
  "s_mov_b32 m0, 0x2000\n" \
  "s_add_u32 %[sO], %[sP0], 0x100000\n" \
  "buffer_load_dwordx4 %[va0], %[rs], %[sO] offen offset:0 nt lds\n" \
  "buffer_load_dwordx4 %[va1], %[rs], %[sO] offen offset:1024 nt lds\n" \
  "buffer_load_dwordx4 %[va2], %[rs], %[sO] offen offset:2048 nt lds\n" \
  "buffer_load_dwordx4 %[va3], %[rs], %[sO] offen offset:3072 nt lds\n" \
  "s_mov_b32 m0, 0x3000\n" \
  "s_add_u32 %[sO], %[sP0], 0x180000\n" \
  "buffer_load_dwordx4 %[va0], %[rs], %[sO] offen offset:0 nt lds\n" \
  "buffer_load_dwordx4 %[va1], %[rs], %[sO] offen offset:1024 nt lds\n" \
  "buffer_load_dwordx4 %[va2], %[rs], %[sO] offen offset:2048 nt lds\n" \
  "buffer_load_dwordx4 %[va3], %[rs], %[sO] offen offset:3072 nt lds\n" \
  "s_cmp_eq_u32 %[sU], 0\n" \
  "s_cbranch_scc1 Lwnu_%=\n" \
  "s_barrier\n" \
  "s_mov_b32 m0, 0x4000\n" \
  "s_add_u32 %[sO], %[sP1], 0x0\n" \
  "buffer_load_dwordx4 %[va0], %[rs], %[sO] offen offset:0 nt lds\n" \
  "buffer_load_dwordx4 %[va1], %[rs], %[sO] offen offset:1024 nt lds\n" \
  "buffer_load_dwordx4 %[va2], %[rs], %[sO] offen offset:2048 nt lds\n" \
  "buffer_load_dwordx4 %[va3], %[rs], %[sO] offen offset:3072 nt lds\n" \
  "s_mov_b32 m0, 0x5000\n" \
  "s_add_u32 %[sO], %[sP1], 0x80000\n" \
  "buffer_load_dwordx4 %[va0], %[rs], %[sO] offen offset:0 nt lds\n" \
  "buffer_load_dwordx4 %[va1], %[rs], %[sO] offen offset:1024 nt lds\n" \
  "buffer_load_dwordx4 %[va2], %[rs], %[sO] offen offset:2048 nt lds\n" \
  "buffer_load_dwordx4 %[va3], %[rs], %[sO] offen offset:3072 nt lds\n" \
  "s_mov_b32 m0, 0x6000\n" \
  "s_add_u32 %[sO], %[sP1], 0x100000\n" \
  "buffer_load_dwordx4 %[va0], %[rs], %[sO] offen offset:0 nt lds\n" \
  "buffer_load_dwordx4 %[va1], %[rs], %[sO] offen offset:1024 nt lds\n" \
  "buffer_load_dwordx4 %[va2], %[rs], %[sO] offen offset:2048 nt lds\n" \
  "buffer_load_dwordx4 %[va3], %[rs], %[sO] offen offset:3072 nt lds\n" \
  "s_mov_b32 m0, 0x7000\n" \
  "s_add_u32 %[sO], %[sP1], 0x180000\n" \
  "buffer_load_dwordx4 %[va0], %[rs], %[sO] offen offset:0 nt lds\n" \
  "buffer_load_dwordx4 %[va1], %[rs], %[sO] offen offset:1024 nt lds\n" \
  "buffer_load_dwordx4 %[va2], %[rs], %[sO] offen offset:2048 nt lds\n" \
  "buffer_load_dwordx4 %[va3], %[rs], %[sO] offen offset:3072 nt lds\n" \
  "Lwnu_%=:\n" \
  "s_add_u32 %[sP0], %[sP0], 0x400000\n" \
  "s_mov_b32 m0, 0x8000\n" \
  "s_add_u32 %[sO], %[sP0], 0x0\n" \
  "buffer_load_dwordx4 %[va0], %[rs], %[sO] offen offset:0 nt lds\n" \
  "buffer_load_dwordx4 %[va1], %[rs], %[sO] offen offset:1024 nt lds\n" \
  "buffer_load_dwordx4 %[va2], %[rs], %[sO] offen offset:2048 nt lds\n" \
  "buffer_load_dwordx4 %[va3], %[rs], %[sO] offen offset:3072 nt lds\n" \
  "s_mov_b32 m0, 0x9000\n" \
  "s_add_u32 %[sO], %[sP0], 0x80000\n" \
  "buffer_load_dwordx4 %[va0], %[rs], %[sO] offen offset:0 nt lds\n" \
  "buffer_load_dwordx4 %[va1], %[rs], %[sO] offen offset:1024 nt lds\n" \
  "buffer_load_dwordx4 %[va2], %[rs], %[sO] offen offset:2048 nt lds\n" \
  "buffer_load_dwordx4 %[va3], %[rs], %[sO] offen offset:3072 nt lds\n" \
  "s_mov_b32 m0, 0xa000\n" \
  "s_add_u32 %[sO], %[sP0], 0x100000\n" \
  "buffer_load_dwordx4 %[va0], %[rs], %[sO] offen offset:0 nt lds\n" \
  "buffer_load_dwordx4 %[va1], %[rs], %[sO] offen offset:1024 nt lds\n" \
  "buffer_load_dwordx4 %[va2], %[rs], %[sO] offen offset:2048 nt lds\n" \
  "buffer_load_dwordx4 %[va3], %[rs], %[sO] offen offset:3072 nt lds\n" \
  "s_mov_b32 m0, 0xb000\n" \
  "s_add_u32 %[sO], %[sP0], 0x180000\n" \
  "buffer_load_dwordx4 %[va0], %[rs], %[sO] offen offset:0 nt lds\n" \
  "buffer_load_dwordx4 %[va1], %[rs], %[sO] offen offset:1024 nt lds\n" \
  "buffer_load_dwordx4 %[va2], %[rs], %[sO] offen offset:2048 nt lds\n" \
  "buffer_load_dwordx4 %[va3], %[rs], %[sO] offen offset:3072 nt lds\n" \
  ""
#define WLOAD2_ASM \
  "s_mov_b32 %[sL], -1\n" \
  "s_mov_b32 %[sC], 0\n" \
  "s_mov_b32 %[sK], 4\n" \
  "s_add_u32 %[sP1], %[sP1], 0x400000\n" \
  "s_mov_b32 %[sJ], 2\n" \
  "v_readlane_b32 %[sR], %[vtabp], 2\n" \
  "s_waitcnt vmcnt(16)\n" \
  "s_barrier\n" \
  "s_mov_b32 m0, 0xc000\n" \
  "s_add_u32 %[sO], %[sP1], 0x0\n" \
  "buffer_load_dwordx4 %[va0], %[rs], %[sO] offen offset:0 nt lds\n" \
  "buffer_load_dwordx4 %[va1], %[rs], %[sO] offen offset:1024 nt lds\n" \
  "buffer_load_dwordx4 %[va2], %[rs], %[sO] offen offset:2048 nt lds\n" \
  "buffer_load_dwordx4 %[va3], %[rs], %[sO] offen offset:3072 nt lds\n" \
  "s_mov_b32 m0, 0xd000\n" \
  "s_add_u32 %[sO], %[sP1], 0x80000\n" \
  "buffer_load_dwordx4 %[va0], %[rs], %[sO] offen offset:0 nt lds\n" \
  "buffer_load_dwordx4 %[va1], %[rs], %[sO] offen offset:1024 nt lds\n" \
  "buffer_load_dwordx4 %[va2], %[rs], %[sO] offen offset:2048 nt lds\n" \
  "buffer_load_dwordx4 %[va3], %[rs], %[sO] offen offset:3072 nt lds\n" \
  "s_mov_b32 m0, 0xe000\n" \
  "s_add_u32 %[sO], %[sP1], 0x100000\n" \
  "buffer_load_dwordx4 %[va0], %[rs], %[sO] offen offset:0 nt lds\n" \
  "buffer_load_dwordx4 %[va1], %[rs], %[sO] offen offset:1024 nt lds\n" \
  "buffer_load_dwordx4 %[va2], %[rs], %[sO] offen offset:2048 nt lds\n" \
  "buffer_load_dwordx4 %[va3], %[rs], %[sO] offen offset:3072 nt lds\n" \
  "s_mov_b32 m0, 0xf000\n" \
  "s_add_u32 %[sO], %[sP1], 0x180000\n" \
  "buffer_load_dwordx4 %[va0], %[rs], %[sO] offen offset:0 nt lds\n" \
  "buffer_load_dwordx4 %[va1], %[rs], %[sO] offen offset:1024 nt lds\n" \
  "buffer_load_dwordx4 %[va2], %[rs], %[sO] offen offset:2048 nt lds\n" \
  "buffer_load_dwordx4 %[va3], %[rs], %[sO] offen offset:3072 nt lds\n" \
  "s_add_u32 %[sX], %[sK], 1\n" \
  "v_readlane_b32 %[sP0], %[vtab], %[sK]\n" \
  "v_readlane_b32 %[sP1], %[vtab], %[sX]\n" \
  "s_add_u32 %[sK], %[sK], 2\n" \
  "buffer_load_dwordx4 v[244:247], %[vsp], %[rx], %[sR] offen nt\n" \
  "buffer_load_dwordx4 v[244:247], %[vsp], %[rx], %[sR] offen offset:1024 nt\n" \
  "buffer_load_dwordx4 v[244:247], %[vsp3], %[rx3], %[sR] offen nt\n" \
  "buffer_load_dwordx4 v[244:247], %[vsp], %[rx], %[sR] offen sc1\n" \
  "buffer_load_dwordx4 v[244:247], %[vsp], %[rx], %[sR] offen offset:1024 sc1\n" \
  "buffer_load_dwordx4 v[244:247], %[vsp3], %[rx3], %[sR] offen sc1\n" \
  "s_add_u32 %[sJ], %[sJ], 1\n" \
  "v_readlane_b32 %[sR], %[vtabp], %[sJ]\n" \
  "s_nop 4\n" \
  "Lwloop_%=:\n" \
  "s_barrier\n" \
  "buffer_load_dwordx4 v[244:247], %[vsp], %[rx], %[sR] offen nt\n" \
  "buffer_load_dwordx4 v[244:247], %[vsp], %[rx], %[sR] offen offset:1024 nt\n" \
  "buffer_load_dwordx4 v[244:247], %[vsp3], %[rx3], %[sR] offen nt\n" \
  "buffer_load_dwordx4 v[244:247], %[vsp], %[rx], %[sR] offen sc1\n" \
  "buffer_load_dwordx4 v[244:247], %[vsp], %[rx], %[sR] offen offset:1024 sc1\n" \
  "buffer_load_dwordx4 v[244:247], %[vsp3], %[rx3], %[sR] offen sc1\n" \
  "s_add_u32 %[sJ], %[sJ], 1\n" \
  "v_readlane_b32 %[sR], %[vtabp], %[sJ]\n" \
  "s_waitcnt vmcnt(6)\n" \
  "s_barrier\n" \
  "s_sleep 7\n" \
  "s_mov_b32 m0, 0x0\n" \
  "s_add_u32 %[sO], %[sP0], 0x0\n" \
  "buffer_load_dwordx4 %[va0], %[rs], %[sO] offen offset:0 nt lds\n" \
  "buffer_load_dwordx4 %[va1], %[rs], %[sO] offen offset:1024 nt lds\n" \
  "buffer_load_dwordx4 %[va2], %[rs], %[sO] offen offset:2048 nt lds\n" \
  "buffer_load_dwordx4 %[va3], %[rs], %[sO] offen offset:3072 nt lds\n" \
  "s_mov_b32 m0, 0x1000\n" \
  "s_add_u32 %[sO], %[sP0], 0x80000\n" \
  "buffer_load_dwordx4 %[va0], %[rs], %[sO] offen offset:0 nt lds\n" \
  "buffer_load_dwordx4 %[va1], %[rs], %[sO] offen offset:1024 nt lds\n" \
  "buffer_load_dwordx4 %[va2], %[rs], %[sO] offen offset:2048 nt lds\n" \
  "buffer_load_dwordx4 %[va3], %[rs], %[sO] offen offset:3072 nt lds\n" \
  "s_mov_b32 m0, 0x2000\n" \
  "s_add_u32 %[sO], %[sP0], 0x100000\n" \
  "buffer_load_dwordx4 %[va0], %[rs], %[sO] offen offset:0 nt lds\n" \
  "buffer_load_dwordx4 %[va1], %[rs], %[sO] offen offset:1024 nt lds\n" \
  "buffer_load_dwordx4 %[va2], %[rs], %[sO] offen offset:2048 nt lds\n" \
  "buffer_load_dwordx4 %[va3], %[rs], %[sO] offen offset:3072 nt lds\n" \
  "s_mov_b32 m0, 0x3000\n" \
  "s_add_u32 %[sO], %[sP0], 0x180000\n" \
  "buffer_load_dwordx4 %[va0], %[rs], %[sO] offen offset:0 nt lds\n" \
  "buffer_load_dwordx4 %[va1], %[rs], %[sO] offen offset:1024 nt lds\n" \
  "buffer_load_dwordx4 %[va2], %[rs], %[sO] offen offset:2048 nt lds\n" \
  "buffer_load_dwordx4 %[va3], %[rs], %[sO] offen offset:3072 nt lds\n" \
  "s_mov_b32 m0, 0x4000\n" \
  "s_add_u32 %[sO], %[sP1], 0x0\n" \
  "buffer_load_dwordx4 %[va0], %[rs], %[sO] offen offset:0 nt lds\n" \
  "buffer_load_dwordx4 %[va1], %[rs], %[sO] offen offset:1024 nt lds\n" \
  "buffer_load_dwordx4 %[va2], %[rs], %[sO] offen offset:2048 nt lds\n" \
  "buffer_load_dwordx4 %[va3], %[rs], %[sO] offen offset:3072 nt lds\n" \
  "s_mov_b32 m0, 0x5000\n" \
  "s_add_u32 %[sO], %[sP1], 0x80000\n" \
  "buffer_load_dwordx4 %[va0], %[rs], %[sO] offen offset:0 nt lds\n" \
  "buffer_load_dwordx4 %[va1], %[rs], %[sO] offen offset:1024 nt lds\n" \
  "buffer_load_dwordx4 %[va2], %[rs], %[sO] offen offset:2048 nt lds\n" \
  "buffer_load_dwordx4 %[va3], %[rs], %[sO] offen offset:3072 nt lds\n" \
  "s_mov_b32 m0, 0x6000\n" \
  "s_add_u32 %[sO], %[sP1], 0x100000\n" \
  "buffer_load_dwordx4 %[va0], %[rs], %[sO] offen offset:0 nt lds\n" \
  "buffer_load_dwordx4 %[va1], %[rs], %[sO] offen offset:1024 nt lds\n" \
  "buffer_load_dwordx4 %[va2], %[rs], %[sO] offen offset:2048 nt lds\n" \
  "buffer_load_dwordx4 %[va3], %[rs], %[sO] offen offset:3072 nt lds\n" \
  "s_mov_b32 m0, 0x7000\n" \
  "s_add_u32 %[sO], %[sP1], 0x180000\n" \
  "buffer_load_dwordx4 %[va0], %[rs], %[sO] offen offset:0 nt lds\n" \
  "buffer_load_dwordx4 %[va1], %[rs], %[sO] offen offset:1024 nt lds\n" \
  "buffer_load_dwordx4 %[va2], %[rs], %[sO] offen offset:2048 nt lds\n" \
  "buffer_load_dwordx4 %[va3], %[rs], %[sO] offen offset:3072 nt lds\n" \
  "s_add_u32 %[sX], %[sK], 1\n" \
  "v_readlane_b32 %[sP0], %[vtab], %[sK]\n" \
  "v_readlane_b32 %[sP1], %[vtab], %[sX]\n" \
  "s_add_u32 %[sK], %[sK], 2\n" \
  "v_readlane_b32 %[sL], %[vtabl], %[sC]\n" \
  "s_add_u32 %[sC], %[sC], 1\n" \
  "s_min_u32 %[sC], %[sC], 63\n" \
  "s_nop 1\n" \
  "s_cmp_lt_i32 %[sL], 0\n" \
  "s_cbranch_scc1 Lwa110_%=\n" \
  "buffer_load_dwordx4 v[244:247], %[va0], %[rs], %[sL] offen offset:0 nt\n" \
  "buffer_load_dwordx4 v[244:247], %[va1], %[rs], %[sL] offen offset:1024 nt\n" \
  "buffer_load_dwordx4 v[244:247], %[va2], %[rs], %[sL] offen offset:2048 nt\n" \
  "buffer_load_dwordx4 v[244:247], %[va3], %[rs], %[sL] offen offset:3072 nt\n" \
  "s_add_u32 %[sL], %[sL], 0x80000\n" \
  "buffer_load_dwordx4 v[244:247], %[va0], %[rs], %[sL] offen offset:0 nt\n" \
  "buffer_load_dwordx4 v[244:247], %[va1], %[rs], %[sL] offen offset:1024 nt\n" \
  "buffer_load_dwordx4 v[244:247], %[va2], %[rs], %[sL] offen offset:2048 nt\n" \
  "buffer_load_dwordx4 v[244:247], %[va3], %[rs], %[sL] offen offset:3072 nt\n" \
  "s_sub_u32 %[sL], %[sL], 0x80000\n" \
  "buffer_load_dword v244, %[vg], %[rs], %[sL] offen sc1\n" \
  "Lwa110_%=:\n" \
  "s_barrier\n" \
  "buffer_load_dwordx4 v[244:247], %[vsp], %[rx], %[sR] offen nt\n" \
  "buffer_load_dwordx4 v[244:247], %[vsp], %[rx], %[sR] offen offset:1024 nt\n" \
  "buffer_load_dwordx4 v[244:247], %[vsp3], %[rx3], %[sR] offen nt\n" \
  "buffer_load_dwordx4 v[244:247], %[vsp], %[rx], %[sR] offen sc1\n" \
  "buffer_load_dwordx4 v[244:247], %[vsp], %[rx], %[sR] offen offset:1024 sc1\n" \
  "buffer_load_dwordx4 v[244:247], %[vsp3], %[rx3], %[sR] offen sc1\n" \
  "s_add_u32 %[sJ], %[sJ], 1\n" \
  "v_readlane_b32 %[sR], %[vtabp], %[sJ]\n" \
  "s_waitcnt vmcnt(6)\n" \
  "s_barrier\n" \
  "s_sleep 7\n" \
  "s_mov_b32 m0, 0x8000\n" \
  "s_add_u32 %[sO], %[sP0], 0x0\n" \
  "buffer_load_dwordx4 %[va0], %[rs], %[sO] offen offset:0 nt lds\n" \
  "buffer_load_dwordx4 %[va1], %[rs], %[sO] offen offset:1024 nt lds\n" \
  "buffer_load_dwordx4 %[va2], %[rs], %[sO] offen offset:2048 nt lds\n" \
  "buffer_load_dwordx4 %[va3], %[rs], %[sO] offen offset:3072 nt lds\n" \
  "s_mov_b32 m0, 0x9000\n" \
  "s_add_u32 %[sO], %[sP0], 0x80000\n" \
  "buffer_load_dwordx4 %[va0], %[rs], %[sO] offen offset:0 nt lds\n" \
  "buffer_load_dwordx4 %[va1], %[rs], %[sO] offen offset:1024 nt lds\n" \
  "buffer_load_dwordx4 %[va2], %[rs], %[sO] offen offset:2048 nt lds\n" \
  "buffer_load_dwordx4 %[va3], %[rs], %[sO] offen offset:3072 nt lds\n" \
  "s_mov_b32 m0, 0xa000\n" \
  "s_add_u32 %[sO], %[sP0], 0x100000\n" \
  "buffer_load_dwordx4 %[va0], %[rs], %[sO] offen offset:0 nt lds\n" \
  "buffer_load_dwordx4 %[va1], %[rs], %[sO] offen offset:1024 nt lds\n" \
  "buffer_load_dwordx4 %[va2], %[rs], %[sO] offen offset:2048 nt lds\n" \
  "buffer_load_dwordx4 %[va3], %[rs], %[sO] offen offset:3072 nt lds\n" \
  "s_mov_b32 m0, 0xb000\n" \
  "s_add_u32 %[sO], %[sP0], 0x180000\n" \
  "buffer_load_dwordx4 %[va0], %[rs], %[sO] offen offset:0 nt lds\n" \
  "buffer_load_dwordx4 %[va1], %[rs], %[sO] offen offset:1024 nt lds\n" \
  "buffer_load_dwordx4 %[va2], %[rs], %[sO] offen offset:2048 nt lds\n" \
  "buffer_load_dwordx4 %[va3], %[rs], %[sO] offen offset:3072 nt lds\n" \
  "s_mov_b32 m0, 0xc000\n" \
  "s_add_u32 %[sO], %[sP1], 0x0\n" \
  "buffer_load_dwordx4 %[va0], %[rs], %[sO] offen offset:0 nt lds\n" \
  "buffer_load_dwordx4 %[va1], %[rs], %[sO] offen offset:1024 nt lds\n" \
  "buffer_load_dwordx4 %[va2], %[rs], %[sO] offen offset:2048 nt lds\n" \
  "buffer_load_dwordx4 %[va3], %[rs], %[sO] offen offset:3072 nt lds\n" \
  "s_mov_b32 m0, 0xd000\n" \
  "s_add_u32 %[sO], %[sP1], 0x80000\n" \
  "buffer_load_dwordx4 %[va0], %[rs], %[sO] offen offset:0 nt lds\n" \
  "buffer_load_dwordx4 %[va1], %[rs], %[sO] offen offset:1024 nt lds\n" \
  "buffer_load_dwordx4 %[va2], %[rs], %[sO] offen offset:2048 nt lds\n" \
  "buffer_load_dwordx4 %[va3], %[rs], %[sO] offen offset:3072 nt lds\n" \
  "s_mov_b32 m0, 0xe000\n" \
  "s_add_u32 %[sO], %[sP1], 0x100000\n" \
  "buffer_load_dwordx4 %[va0], %[rs], %[sO] offen offset:0 nt lds\n" \
  "buffer_load_dwordx4 %[va1], %[rs], %[sO] offen offset:1024 nt lds\n" \
  "buffer_load_dwordx4 %[va2], %[rs], %[sO] offen offset:2048 nt lds\n" \
  "buffer_load_dwordx4 %[va3], %[rs], %[sO] offen offset:3072 nt lds\n" \
  "s_mov_b32 m0, 0xf000\n" \
  "s_add_u32 %[sO], %[sP1], 0x180000\n" \
  "buffer_load_dwordx4 %[va0], %[rs], %[sO] offen offset:0 nt lds\n" \
  "buffer_load_dwordx4 %[va1], %[rs], %[sO] offen offset:1024 nt lds\n" \
  "buffer_load_dwordx4 %[va2], %[rs], %[sO] offen offset:2048 nt lds\n" \
  "buffer_load_dwordx4 %[va3], %[rs], %[sO] offen offset:3072 nt lds\n" \
  "s_add_u32 %[sX], %[sK], 1\n" \
  "v_readlane_b32 %[sP0], %[vtab], %[sK]\n" \
  "v_readlane_b32 %[sP1], %[vtab], %[sX]\n" \
  "s_add_u32 %[sK], %[sK], 2\n" \
  "v_readlane_b32 %[sL], %[vtabl], %[sC]\n" \
  "s_add_u32 %[sC], %[sC], 1\n" \
  "s_min_u32 %[sC], %[sC], 63\n" \
  "s_nop 1\n" \
  "s_cmp_lt_i32 %[sL], 0\n" \
  "s_cbranch_scc1 Lwa192_%=\n" \
  "buffer_load_dwordx4 v[244:247], %[va0], %[rs], %[sL] offen offset:0 nt\n" \
  "buffer_load_dwordx4 v[244:247], %[va1], %[rs], %[sL] offen offset:1024 nt\n" \
  "buffer_load_dwordx4 v[244:247], %[va2], %[rs], %[sL] offen offset:2048 nt\n" \
  "buffer_load_dwordx4 v[244:247], %[va3], %[rs], %[sL] offen offset:3072 nt\n" \
  "s_add_u32 %[sL], %[sL], 0x80000\n" \
  "buffer_load_dwordx4 v[244:247], %[va0], %[rs], %[sL] offen offset:0 nt\n" \
  "buffer_load_dwordx4 v[244:247], %[va1], %[rs], %[sL] offen offset:1024 nt\n" \
  "buffer_load_dwordx4 v[244:247], %[va2], %[rs], %[sL] offen offset:2048 nt\n" \
  "buffer_load_dwordx4 v[244:247], %[va3], %[rs], %[sL] offen offset:3072 nt\n" \
  "s_sub_u32 %[sL], %[sL], 0x80000\n" \
  "buffer_load_dword v244, %[vg], %[rs], %[sL] offen sc1\n" \
  "Lwa192_%=:\n" \
  "s_cmp_lt_u32 %[sK], 30\n" \
  "s_cbranch_scc1 Lwloop_%=\n" \
  "s_barrier\n" \
  "buffer_load_dwordx4 v[244:247], %[vsp], %[rx], %[sR] offen nt\n" \
  "buffer_load_dwordx4 v[244:247], %[vsp], %[rx], %[sR] offen offset:1024 nt\n" \
  "buffer_load_dwordx4 v[244:247], %[vsp3], %[rx3], %[sR] offen nt\n" \
  "buffer_load_dwordx4 v[244:247], %[vsp], %[rx], %[sR] offen sc1\n" \
  "buffer_load_dwordx4 v[244:247], %[vsp], %[rx], %[sR] offen offset:1024 sc1\n" \
  "buffer_load_dwordx4 v[244:247], %[vsp3], %[rx3], %[sR] offen sc1\n" \
  "s_add_u32 %[sJ], %[sJ], 1\n" \
  "v_readlane_b32 %[sR], %[vtabp], %[sJ]\n" \
  "s_waitcnt vmcnt(6)\n" \
  "s_barrier\n" \
  "s_sleep 7\n" \
  "s_mov_b32 m0, 0x0\n" \
  "s_add_u32 %[sO], %[sP0], 0x0\n" \
  "buffer_load_dwordx4 %[va0], %[rs], %[sO] offen offset:0 lds\n" \
  "buffer_load_dwordx4 %[va1], %[rs], %[sO] offen offset:1024 lds\n" \
  "buffer_load_dwordx4 %[va2], %[rs], %[sO] offen offset:2048 lds\n" \
  "buffer_load_dwordx4 %[va3], %[rs], %[sO] offen offset:3072 lds\n" \
  "s_mov_b32 m0, 0x1000\n" \
  "s_add_u32 %[sO], %[sP0], 0x80000\n" \
  "buffer_load_dwordx4 %[va0], %[rs], %[sO] offen offset:0 lds\n" \
  "buffer_load_dwordx4 %[va1], %[rs], %[sO] offen offset:1024 lds\n" \
  "buffer_load_dwordx4 %[va2], %[rs], %[sO] offen offset:2048 lds\n" \
  "buffer_load_dwordx4 %[va3], %[rs], %[sO] offen offset:3072 lds\n" \
  "s_mov_b32 m0, 0x2000\n" \
  "s_add_u32 %[sO], %[sP0], 0x100000\n" \
  "buffer_load_dwordx4 %[va0], %[rs], %[sO] offen offset:0 lds\n" \
  "buffer_load_dwordx4 %[va1], %[rs], %[sO] offen offset:1024 lds\n" \
  "buffer_load_dwordx4 %[va2], %[rs], %[sO] offen offset:2048 lds\n" \
  "buffer_load_dwordx4 %[va3], %[rs], %[sO] offen offset:3072 lds\n" \
  "s_mov_b32 m0, 0x3000\n" \
  "s_add_u32 %[sO], %[sP0], 0x180000\n" \
  "buffer_load_dwordx4 %[va0], %[rs], %[sO] offen offset:0 lds\n" \
  "buffer_load_dwordx4 %[va1], %[rs], %[sO] offen offset:1024 lds\n" \
  "buffer_load_dwordx4 %[va2], %[rs], %[sO] offen offset:2048 lds\n" \
  "buffer_load_dwordx4 %[va3], %[rs], %[sO] offen offset:3072 lds\n" \
  "s_mov_b32 m0, 0x4000\n" \
  "s_add_u32 %[sO], %[sP1], 0x0\n" \
  "buffer_load_dwordx4 %[va0], %[rs], %[sO] offen offset:0 lds\n" \
  "buffer_load_dwordx4 %[va1], %[rs], %[sO] offen offset:1024 lds\n" \
  "buffer_load_dwordx4 %[va2], %[rs], %[sO] offen offset:2048 lds\n" \
  "buffer_load_dwordx4 %[va3], %[rs], %[sO] offen offset:3072 lds\n" \
  "s_mov_b32 m0, 0x5000\n" \
  "s_add_u32 %[sO], %[sP1], 0x80000\n" \
  "buffer_load_dwordx4 %[va0], %[rs], %[sO] offen offset:0 lds\n" \
  "buffer_load_dwordx4 %[va1], %[rs], %[sO] offen offset:1024 lds\n" \
  "buffer_load_dwordx4 %[va2], %[rs], %[sO] offen offset:2048 lds\n" \
  "buffer_load_dwordx4 %[va3], %[rs], %[sO] offen offset:3072 lds\n" \
  "s_mov_b32 m0, 0x6000\n" \
  "s_add_u32 %[sO], %[sP1], 0x100000\n" \
  "buffer_load_dwordx4 %[va0], %[rs], %[sO] offen offset:0 lds\n" \
  "buffer_load_dwordx4 %[va1], %[rs], %[sO] offen offset:1024 lds\n" \
  "buffer_load_dwordx4 %[va2], %[rs], %[sO] offen offset:2048 lds\n" \
  "buffer_load_dwordx4 %[va3], %[rs], %[sO] offen offset:3072 lds\n" \
  "s_mov_b32 m0, 0x7000\n" \
  "s_add_u32 %[sO], %[sP1], 0x180000\n" \
  "buffer_load_dwordx4 %[va0], %[rs], %[sO] offen offset:0 lds\n" \
  "buffer_load_dwordx4 %[va1], %[rs], %[sO] offen offset:1024 lds\n" \
  "buffer_load_dwordx4 %[va2], %[rs], %[sO] offen offset:2048 lds\n" \
  "buffer_load_dwordx4 %[va3], %[rs], %[sO] offen offset:3072 lds\n" \
  "s_add_u32 %[sX], %[sK], 1\n" \
  "v_readlane_b32 %[sP0], %[vtab], %[sK]\n" \
  "v_readlane_b32 %[sP1], %[vtab], %[sX]\n" \
  "s_add_u32 %[sK], %[sK], 2\n" \
  "v_readlane_b32 %[sL], %[vtabl], %[sC]\n" \
  "s_add_u32 %[sC], %[sC], 1\n" \
  "s_min_u32 %[sC], %[sC], 63\n" \
  "s_nop 1\n" \
  "s_cmp_lt_i32 %[sL], 0\n" \
  "s_cbranch_scc1 Lwa276_%=\n" \
  "buffer_load_dwordx4 v[244:247], %[va0], %[rs], %[sL] offen offset:0 nt\n" \
  "buffer_load_dwordx4 v[244:247], %[va1], %[rs], %[sL] offen offset:1024 nt\n" \
  "buffer_load_dwordx4 v[244:247], %[va2], %[rs], %[sL] offen offset:2048 nt\n" \
  "buffer_load_dwordx4 v[244:247], %[va3], %[rs], %[sL] offen offset:3072 nt\n" \
  "s_add_u32 %[sL], %[sL], 0x80000\n" \
  "buffer_load_dwordx4 v[244:247], %[va0], %[rs], %[sL] offen offset:0 nt\n" \
  "buffer_load_dwordx4 v[244:247], %[va1], %[rs], %[sL] offen offset:1024 nt\n" \
  "buffer_load_dwordx4 v[244:247], %[va2], %[rs], %[sL] offen offset:2048 nt\n" \
  "buffer_load_dwordx4 v[244:247], %[va3], %[rs], %[sL] offen offset:3072 nt\n" \
  "s_sub_u32 %[sL], %[sL], 0x80000\n" \
  "buffer_load_dword v244, %[vg], %[rs], %[sL] offen sc1\n" \
  "Lwa276_%=:\n" \
  "s_barrier\n" \
  "buffer_load_dwordx4 v[244:247], %[vsp], %[rx], %[sR] offen nt\n" \
  "buffer_load_dwordx4 v[244:247], %[vsp], %[rx], %[sR] offen offset:1024 nt\n" \
  "buffer_load_dwordx4 v[244:247], %[vsp3], %[rx3], %[sR] offen nt\n" \
  "buffer_load_dwordx4 v[244:247], %[vsp], %[rx], %[sR] offen sc1\n" \
  "buffer_load_dwordx4 v[244:247], %[vsp], %[rx], %[sR] offen offset:1024 sc1\n" \
  "buffer_load_dwordx4 v[244:247], %[vsp3], %[rx3], %[sR] offen sc1\n" \
  "s_add_u32 %[sJ], %[sJ], 1\n" \
  "v_readlane_b32 %[sR], %[vtabp], %[sJ]\n" \
  "s_waitcnt vmcnt(6)\n" \
  "s_barrier\n" \
  "s_sleep 7\n" \
  "s_mov_b32 m0, 0x8000\n" \
  "s_add_u32 %[sO], %[sP0], 0x0\n" \
  "buffer_load_dwordx4 %[va0], %[rs], %[sO] offen offset:0 lds\n" \
  "buffer_load_dwordx4 %[va1], %[rs], %[sO] offen offset:1024 lds\n" \
  "buffer_load_dwordx4 %[va2], %[rs], %[sO] offen offset:2048 lds\n" \
  "buffer_load_dwordx4 %[va3], %[rs], %[sO] offen offset:3072 lds\n" \
  "s_mov_b32 m0, 0x9000\n" \
  "s_add_u32 %[sO], %[sP0], 0x80000\n" \
  "buffer_load_dwordx4 %[va0], %[rs], %[sO] offen offset:0 lds\n" \
  "buffer_load_dwordx4 %[va1], %[rs], %[sO] offen offset:1024 lds\n" \
  "buffer_load_dwordx4 %[va2], %[rs], %[sO] offen offset:2048 lds\n" \
  "buffer_load_dwordx4 %[va3], %[rs], %[sO] offen offset:3072 lds\n" \
  "s_mov_b32 m0, 0xa000\n" \
  "s_add_u32 %[sO], %[sP0], 0x100000\n" \
  "buffer_load_dwordx4 %[va0], %[rs], %[sO] offen offset:0 lds\n" \
  "buffer_load_dwordx4 %[va1], %[rs], %[sO] offen offset:1024 lds\n" \
  "buffer_load_dwordx4 %[va2], %[rs], %[sO] offen offset:2048 lds\n" \
  "buffer_load_dwordx4 %[va3], %[rs], %[sO] offen offset:3072 lds\n" \
  "s_mov_b32 m0, 0xb000\n" \
  "s_add_u32 %[sO], %[sP0], 0x180000\n" \
  "buffer_load_dwordx4 %[va0], %[rs], %[sO] offen offset:0 lds\n" \
  "buffer_load_dwordx4 %[va1], %[rs], %[sO] offen offset:1024 lds\n" \
  "buffer_load_dwordx4 %[va2], %[rs], %[sO] offen offset:2048 lds\n" \
  "buffer_load_dwordx4 %[va3], %[rs], %[sO] offen offset:3072 lds\n" \
  "s_mov_b32 m0, 0xc000\n" \
  "s_add_u32 %[sO], %[sP1], 0x0\n" \
  "buffer_load_dwordx4 %[va0], %[rs], %[sO] offen offset:0 lds\n" \
  "buffer_load_dwordx4 %[va1], %[rs], %[sO] offen offset:1024 lds\n" \
  "buffer_load_dwordx4 %[va2], %[rs], %[sO] offen offset:2048 lds\n" \
  "buffer_load_dwordx4 %[va3], %[rs], %[sO] offen offset:3072 lds\n" \
  "s_mov_b32 m0, 0xd000\n" \
  "s_add_u32 %[sO], %[sP1], 0x80000\n" \
  "buffer_load_dwordx4 %[va0], %[rs], %[sO] offen offset:0 lds\n" \
  "buffer_load_dwordx4 %[va1], %[rs], %[sO] offen offset:1024 lds\n" \
  "buffer_load_dwordx4 %[va2], %[rs], %[sO] offen offset:2048 lds\n" \
  "buffer_load_dwordx4 %[va3], %[rs], %[sO] offen offset:3072 lds\n" \
  "s_mov_b32 m0, 0xe000\n" \
  "s_add_u32 %[sO], %[sP1], 0x100000\n" \
  "buffer_load_dwordx4 %[va0], %[rs], %[sO] offen offset:0 lds\n" \
  "buffer_load_dwordx4 %[va1], %[rs], %[sO] offen offset:1024 lds\n" \
  "buffer_load_dwordx4 %[va2], %[rs], %[sO] offen offset:2048 lds\n" \
  "buffer_load_dwordx4 %[va3], %[rs], %[sO] offen offset:3072 lds\n" \
  "s_mov_b32 m0, 0xf000\n" \
  "s_add_u32 %[sO], %[sP1], 0x180000\n" \
  "buffer_load_dwordx4 %[va0], %[rs], %[sO] offen offset:0 lds\n" \
  "buffer_load_dwordx4 %[va1], %[rs], %[sO] offen offset:1024 lds\n" \
  "buffer_load_dwordx4 %[va2], %[rs], %[sO] offen offset:2048 lds\n" \
  "buffer_load_dwordx4 %[va3], %[rs], %[sO] offen offset:3072 lds\n" \
  "s_add_u32 %[sX], %[sK], 1\n" \
  "v_readlane_b32 %[sP0], %[vtab], %[sK]\n" \
  "v_readlane_b32 %[sP1], %[vtab], %[sX]\n" \
  "s_add_u32 %[sK], %[sK], 2\n" \
  "v_readlane_b32 %[sL], %[vtabl], %[sC]\n" \
  "s_add_u32 %[sC], %[sC], 1\n" \
  "s_min_u32 %[sC], %[sC], 63\n" \
  "s_nop 1\n" \
  "s_cmp_lt_i32 %[sL], 0\n" \
  "s_cbranch_scc1 Lwa358_%=\n" \
  "buffer_load_dwordx4 v[244:247], %[va0], %[rs], %[sL] offen offset:0 nt\n" \
  "buffer_load_dwordx4 v[244:247], %[va1], %[rs], %[sL] offen offset:1024 nt\n" \
  "buffer_load_dwordx4 v[244:247], %[va2], %[rs], %[sL] offen offset:2048 nt\n" \
  "buffer_load_dwordx4 v[244:247], %[va3], %[rs], %[sL] offen offset:3072 nt\n" \
  "s_add_u32 %[sL], %[sL], 0x80000\n" \
  "buffer_load_dwordx4 v[244:247], %[va0], %[rs], %[sL] offen offset:0 nt\n" \
  "buffer_load_dwordx4 v[244:247], %[va1], %[rs], %[sL] offen offset:1024 nt\n" \
  "buffer_load_dwordx4 v[244:247], %[va2], %[rs], %[sL] offen offset:2048 nt\n" \
  "buffer_load_dwordx4 v[244:247], %[va3], %[rs], %[sL] offen offset:3072 nt\n" \
  "s_sub_u32 %[sL], %[sL], 0x80000\n" \
  "buffer_load_dword v244, %[vg], %[rs], %[sL] offen sc1\n" \
  "Lwa358_%=:\n" \
  "s_waitcnt vmcnt(0)\n" \
  "s_barrier\n" \
  ""
#define HLOAD_ASM \
  "s_nop 3\n" \
  "s_mov_b32 m0, 0x4000\n" \
  "s_add_u32 %[sO], %[sP1], 0x0\n" \
  "buffer_load_dwordx4 %[va0], %[rs], %[sO] offen offset:0 nt lds\n" \
  "buffer_load_dwordx4 %[va1], %[rs], %[sO] offen offset:1024 nt lds\n" \
  "buffer_load_dwordx4 %[va2], %[rs], %[sO] offen offset:2048 nt lds\n" \
  "buffer_load_dwordx4 %[va3], %[rs], %[sO] offen offset:3072 nt lds\n" \
  "s_mov_b32 m0, 0x5000\n" \
  "s_add_u32 %[sO], %[sP1], 0x80000\n" \
  "buffer_load_dwordx4 %[va0], %[rs], %[sO] offen offset:0 nt lds\n" \
  "buffer_load_dwordx4 %[va1], %[rs], %[sO] offen offset:1024 nt lds\n" \
  "buffer_load_dwordx4 %[va2], %[rs], %[sO] offen offset:2048 nt lds\n" \
  "buffer_load_dwordx4 %[va3], %[rs], %[sO] offen offset:3072 nt lds\n" \
  "s_mov_b32 m0, 0x6000\n" \
  "s_add_u32 %[sO], %[sP1], 0x100000\n" \
  "buffer_load_dwordx4 %[va0], %[rs], %[sO] offen offset:0 nt lds\n" \
  "buffer_load_dwordx4 %[va1], %[rs], %[sO] offen offset:1024 nt lds\n" \
  "buffer_load_dwordx4 %[va2], %[rs], %[sO] offen offset:2048 nt lds\n" \
  "buffer_load_dwordx4 %[va3], %[rs], %[sO] offen offset:3072 nt lds\n" \
  "s_mov_b32 m0, 0x7000\n" \
  "s_add_u32 %[sO], %[sP1], 0x180000\n" \
  "buffer_load_dwordx4 %[va0], %[rs], %[sO] offen offset:0 nt lds\n" \
  "buffer_load_dwordx4 %[va1], %[rs], %[sO] offen offset:1024 nt lds\n" \
  "buffer_load_dwordx4 %[va2], %[rs], %[sO] offen offset:2048 nt lds\n" \
  "buffer_load_dwordx4 %[va3], %[rs], %[sO] offen offset:3072 nt lds\n" \
  ""
#define TAIL_G 2
#define HALF_SWAP 0
#define LATE_SETS 2
#define LATE_FIRST 6
#define XPF 0
#define XLATE 0
#define XLDIST 6


namespace {
#define DI __device__ __forceinline__
constexpr int NCOL = 16384;
constexpr int A_OFF = 64 * 1024;
constexpr int LDS_BYTES = 160 * 1024;
constexpr unsigned CHUNK = 512u * 512u;
constexpr unsigned FLAG_OFF = 16u * CHUNK;
constexpr unsigned CNT_OFF = 0x10000u;
constexpr unsigned TMO_OFF = 0x20000u;
constexpr unsigned CTRL_BYTES = 0x20100u;

struct Args {
  const signed char* x;
  const signed char* w;
  const float* sx;
  const float* sw;
  float* out;
  unsigned char* myws;
  unsigned rank, pad[3];
  unsigned char* ws[8];
};

static_assert(offsetof(Args, rank) == 48 && offsetof(Args, ws) == 64, "the kernel reads its arguments by offset");
DI unsigned rfl(unsigned v) { return __builtin_amdgcn_readfirstlane(v); }
#define V10(b) "v" #b "0", "v" #b "1", "v" #b "2", "v" #b "3", "v" #b "4", "v" #b "5", "v" #b "6", "v" #b "7", "v" #b "8", "v" #b "9"
#define VCLOB "v16", "v17", "v18", "v19", V10(2), V10(3), V10(4), V10(5), V10(6), V10(7), V10(8), V10(9), V10(10), V10(11), V10(12), V10(13), \
  V10(14), V10(15), V10(16), V10(17), V10(18), V10(19), V10(20), V10(21), V10(22), V10(23), V10(24), "v250", "v251", "v252", "v253", "v254", "v255"

DI unsigned phys_kt(unsigned kt, unsigned rank) {
  const unsigned j = kt - 4, sub = (j * 37u) >> 8, cr = 1 + j - sub * 7u;
  return kt < 4 ? rank * 4 + kt : ((rank + cr) & 7u) * 4 + sub;
}
DI void unit_of(unsigned u, unsigned& ss, unsigned& half) {
  constexpr unsigned NF = TAIL_G >= 2 ? 16 - TAIL_G : 16;
  ss = u < 2 * NF ? u >> 1 : (u < NF + 16 ? u - NF : 31 + NF - u);
  half = u < 2 * NF ? (u & 1) : (u >= NF + 16 ? 1u : 0u);
}
constexpr unsigned GATE1 = 8 - XPF, GATE2 = 16 - XPF, GATE3 = 20;
static_assert(TAIL_G <= 4, "the gates assume the regular unit order up to unit 24");

__global__ void __launch_bounds__(512, 2) a2a_gemm_kernel(const Args args) {
  typedef unsigned v16u __attribute__((ext_vector_type(16)));
  v16u ka;
  unsigned long long peer_ws, pc;
  unsigned code_lim;
  {
    unsigned long long ex;
    unsigned off;
    asm volatile("s_load_dwordx16 %[ka], %[kp], 0x0\n"
                 "s_load_dwordx2 %[pw], %[kp], %[qo] offset:0x40\n"
                 "s_getpc_b64 %[pc]\n"
                 "La2a_code_begin:\n"
                 "s_mov_b32 %[lim], La2a_code_end-La2a_code_begin-16\n"
                 "v_lshlrev_b32 %[off], 4, %[tid]\n"
                 "v_add_u32 %[off], %[blk], %[off]\n"
                 "v_min_u32 %[off], %[lim], %[off]\n"
                 "v_cmp_gt_u32 vcc, 0x100, %[tid]\n"
                 "s_and_saveexec_b64 %[ex], vcc\n"
                 "s_cbranch_execz Lcw_%=\n"
                 "global_load_dwordx4 v[16:19], %[off], %[pc]\n"
                 "Lcw_%=:\n"
                 "s_mov_b64 exec, %[ex]\n"
                 "s_waitcnt lgkmcnt(0)\n"
                 : [ka] "=&s"(ka), [pw] "=&s"(peer_ws), [pc] "=&s"(pc), [ex] "=&s"(ex), [off] "=&v"(off), [lim] "=&s"(code_lim)
                 : [kp] "s"(__builtin_amdgcn_kernarg_segment_ptr()), [qo] "s"(blockIdx.x & 56u), [blk] "s"(((blockIdx.x >> 3) % 5u) * 4096u),
                   [tid] "v"(threadIdx.x)
                 : "memory", "vcc", "scc", "v16", "v17", "v18", "v19");
  }
#define KARG(T, i) ((T)(((unsigned long long)ka[(i) + 1] << 32) | ka[i]))
  struct {
    const signed char* x;
    const signed char* w;
    const float* sx;
    const float* sw;
    float* out;
    unsigned char* myws;
    unsigned rank;
  } const a = {KARG(const signed char*, 0), KARG(const signed char*, 2), KARG(const float*, 4), KARG(const float*, 6), KARG(float*, 8), KARG(unsigned char*, 10), ka[12]};
  const int lane = threadIdx.x & 63;
  const int wave = rfl(threadIdx.x >> 6);
  const unsigned b = blockIdx.x;
  const unsigned mh = (b >> 3) & 1;
  const unsigned line0 = (unsigned)((size_t)a.w >> 7);
  const unsigned n0 = ((b & 7) * 16 + (((b >> 4) + 4u - line0) & 15u)) * 128, m0 = mh * 256;
  const unsigned rank = a.rank & 7;
  unsigned char* const myws = a.myws;
  const unsigned* const cntp = (const unsigned*)(myws + FLAG_OFF + CNT_OFF + b * 64);

  unsigned cnt0;
  asm volatile("s_load_dword %0, %1, 0x0" : "=s"(cnt0) : "s"(cntp));

  if (wave >= 4) {
    const unsigned r = lane >> 3, cp = lane & 7;
    const bool urgent = b < 64 && ((b >> 3) & 7) != 0;
    if (wave == 5 || wave == 6) {
      const __amdgpu_buffer_rsrc_t rw = __builtin_amdgcn_make_buffer_rsrc((void*)(a.x + rank * CHUNK + (b >> 3) * 8192 + (wave - 5) * 4096), 0, 4096, 0x00020000);
      asm volatile("s_nop 3\n"
                   "buffer_load_dwordx4 v[244:247], %[v], %[r], 0 offen\n"
                   "buffer_load_dwordx4 v[244:247], %[v], %[r], 0 offen offset:1024\n"
                   "buffer_load_dwordx4 v[244:247], %[v], %[r], 0 offen offset:2048\n"
                   "buffer_load_dwordx4 v[244:247], %[v], %[r], 0 offen offset:3072\n"
                   "s_waitcnt vmcnt(0)\n" ::[v] "v"(lane * 16), [r] "s"(rw)
                   : "memory", "v244", "v245", "v246", "v247");
    }
    if (urgent && (wave == 5 || wave == 6)) __builtin_amdgcn_s_barrier();
    unsigned sX, sY, sO;
    if (wave == 4 || wave == 7) {
      const __amdgpu_buffer_rsrc_t rs = __builtin_amdgcn_make_buffer_rsrc((void*)(a.w + n0), 0, 4096u * NCOL - n0, 0x00020000);
      const unsigned v0 = r * NCOL + ((cp ^ (r >> 1)) << 4), st = 8 * NCOL - 1024;
      unsigned sP0 = rank * 4 * (128u * NCOL), sP1 = sP0 + 128u * NCOL, sK;
      if (wave == 4) {
        unsigned tw0 = 0;
        {
          const unsigned char* dw = (const unsigned char*)peer_ws;
          asm volatile("s_load_dword %0, %1, 0x0\n s_load_dword %0, %2, 0x0\n s_load_dword %0, %3, 0x0"
                       : "+s"(tw0)
                       : "s"(dw + rank * CHUNK), "s"(dw + (8 + rank) * CHUNK), "s"(dw + FLAG_OFF + 0x8000));
        }
        {
          const unsigned c = b >> 3;
          asm volatile("s_load_dword %0, %1, 0x0\n s_load_dword %0, %2, 0x0\n s_load_dword %0, %3, 0x0"
                       : "+s"(tw0)
                       : "s"(a.w + (size_t)c * (128u * NCOL) + n0), "s"((const char*)a.out + (size_t)c * (16u * NCOL * 4) + n0 * 4), "s"(myws + c * (CHUNK / 2)));
        }
        const unsigned c = b >> 3, g0 = (b & 7) * 2048;
        const unsigned s1 = (3u - line0) & 7u, s3 = (63u - line0 - (b & 7) * 16) & 63u;
        const __amdgpu_buffer_rsrc_t rx = __builtin_amdgcn_make_buffer_rsrc((void*)(a.w + g0), 0, 4096u * NCOL - g0, 0x00020000);
        const __amdgpu_buffer_rsrc_t rx3 = __builtin_amdgcn_make_buffer_rsrc((void*)(a.w + g0), 0, s3 < 16 ? 4096u * NCOL - g0 : 0u, 0x00020000);
        const unsigned vsb = ((c & 15) * 8 + r) * NCOL + cp * 16;
        asm volatile("s_nop 3\n"
                       "buffer_load_dwordx4 v[244:247], %[vsp], %[rx], %[o0] offen nt\n"
                       "buffer_load_dwordx4 v[244:247], %[vsp], %[rx], %[o0] offen offset:1024 nt\n"
                       "buffer_load_dwordx4 v[244:247], %[vsp3], %[rx3], %[o0] offen nt\n"
                       "buffer_load_dwordx4 v[244:247], %[vsp], %[rx], %[o1] offen nt\n"
                       "buffer_load_dwordx4 v[244:247], %[vsp], %[rx], %[o1] offen offset:1024 nt\n"
                       "buffer_load_dwordx4 v[244:247], %[vsp3], %[rx3], %[o1] offen nt\n"
                       "buffer_load_dwordx4 v[244:247], %[vsp], %[rx], %[o1] offen sc1\n"
                       "buffer_load_dwordx4 v[244:247], %[vsp], %[rx], %[o1] offen offset:1024 sc1\n"
                       "buffer_load_dwordx4 v[244:247], %[vsp3], %[rx3], %[o1] offen sc1\n" ::[vsp] "v"(vsb + s1 * 128),
                       [vsp3] "v"(vsb + (s3 & 15) * 128), [rx] "s"(rx), [rx3] "s"(rx3), [o0] "s"((rank * 4 + (c >> 4)) * (128u * NCOL)),
                       [o1] "s"((rank * 4 + 2 + (c >> 4)) * (128u * NCOL))
                       : "memory", "v244", "v245", "v246", "v247");
        asm volatile(WLOAD1_ASM
                     : [sO] "=&s"(sO), [sP0] "+s"(sP0)
                     : [va0] "v"(v0), [va1] "v"(v0 + st), [va2] "v"((v0 ^ 64) + 2 * st), [va3] "v"((v0 ^ 64) + 3 * st), [rs] "s"(rs),
                       [sU] "s"(b < 64 ? b & 56u : 0u), [sP1] "s"(sP1)
                     : "memory", "scc");
        const unsigned vtab = phys_kt((lane & 31) ^ (HALF_SWAP ? mh : 0u), rank) * (128u * NCOL);
        const unsigned vtabp = lane < 16 ? phys_kt(2 * lane + (c >> 4), rank) * (128u * NCOL) : 0x7f000000u;
        const unsigned lb = lane - LATE_FIRST;
        const unsigned vtabl = lb < 2 * LATE_SETS ? phys_kt(2 * (16 - LATE_SETS + (lb >> 1)) + mh, rank) * (128u * NCOL) + (lb & 1) * (64u * NCOL) : 0xffffffffu;
        unsigned sJ, sR, sL, sC;
        asm volatile(WLOAD2_ASM
                     : [sO] "=&s"(sO), [sX] "=&s"(sX), [sP0] "+s"(sP0), [sP1] "+s"(sP1), [sK] "=&s"(sK), [sJ] "=&s"(sJ), [sR] "=&s"(sR), [sL] "=&s"(sL), [sC] "=&s"(sC)
                     : [va0] "v"(v0), [va1] "v"(v0 + st), [va2] "v"((v0 ^ 64) + 2 * st), [va3] "v"((v0 ^ 64) + 3 * st), [vtab] "v"(vtab), [rs] "s"(rs),
                       [vsp] "v"(vsb + s1 * 128), [vsp3] "v"(vsb + (s3 & 15) * 128), [vtabp] "v"(vtabp), [vtabl] "v"(vtabl), [vg] "v"(lane * NCOL), [sMh] "s"(mh * 16384), [rx] "s"(rx), [rx3] "s"(rx3)
                     : "memory", "scc", "v244", "v245", "v246", "v247");
        asm volatile("s_waitcnt lgkmcnt(0)\n La2a_code_end:" ::"s"(cnt0), "s"(tw0));
        return;
      }
      const unsigned q = (b >> 3) & 7, sub = b >> 6, grp = b & 7, dest = (rank + q) & 7;
      const unsigned vx = r * 512 + cp * 16, poff = grp * 32768 + sub * 128;
      const __amdgpu_buffer_rsrc_t rx = __builtin_amdgcn_make_buffer_rsrc((void*)(a.x + dest * CHUNK + poff), 0, q ? 32768 - sub * 128 : 0, 0x00020000);
      unsigned char* const dws = (unsigned char*)peer_ws;
#define A2A_XLOADS                                                \
  "s_mov_b32 %[sO], 0x1000\n"                                     \
  "buffer_load_dwordx4 v[208:211], %[vx], %[rx], 0 offen nt\n"            \
  "buffer_load_dwordx4 v[212:215], %[vx], %[rx], %[sO] offen nt\n"        \
  "s_mov_b32 %[sO], 0x2000\n"                                     \
  "buffer_load_dwordx4 v[216:219], %[vx], %[rx], %[sO] offen nt\n"        \
  "s_mov_b32 %[sO], 0x3000\n"                                     \
  "buffer_load_dwordx4 v[220:223], %[vx], %[rx], %[sO] offen nt\n"        \
  "s_mov_b32 %[sO], 0x4000\n"                                     \
  "buffer_load_dwordx4 v[224:227], %[vx], %[rx], %[sO] offen nt\n"        \
  "s_mov_b32 %[sO], 0x5000\n"                                     \
  "buffer_load_dwordx4 v[228:231], %[vx], %[rx], %[sO] offen nt\n"        \
  "s_mov_b32 %[sO], 0x6000\n"                                     \
  "buffer_load_dwordx4 v[232:235], %[vx], %[rx], %[sO] offen nt\n"        \
  "s_mov_b32 %[sO], 0x7000\n"                                     \
  "buffer_load_dwordx4 v[236:239], %[vx], %[rx], %[sO] offen nt\n"
#define A2A_XSTORES                                                       \
  "s_mov_b32 %[sO], 0x1000\n"                                             \
  "buffer_store_dwordx4 v[208:211], %[vx], %[rd], 0 offen sc0 sc1\n"           \
  "buffer_store_dwordx4 v[212:215], %[vx], %[rd], %[sO] offen sc0 sc1\n"       \
  "s_mov_b32 %[sO], 0x2000\n"                                             \
  "buffer_store_dwordx4 v[216:219], %[vx], %[rd], %[sO] offen sc0 sc1\n"       \
  "s_mov_b32 %[sO], 0x3000\n"                                             \
  "buffer_store_dwordx4 v[220:223], %[vx], %[rd], %[sO] offen sc0 sc1\n"       \
  "s_mov_b32 %[sO], 0x4000\n"                                             \
  "buffer_store_dwordx4 v[224:227], %[vx], %[rd], %[sO] offen sc0 sc1\n"       \
  "s_mov_b32 %[sO], 0x5000\n"                                             \
  "buffer_store_dwordx4 v[228:231], %[vx], %[rd], %[sO] offen sc0 sc1\n"       \
  "s_mov_b32 %[sO], 0x6000\n"                                             \
  "buffer_store_dwordx4 v[232:235], %[vx], %[rd], %[sO] offen sc0 sc1\n"       \
  "s_mov_b32 %[sO], 0x7000\n"                                             \
  "buffer_store_dwordx4 v[236:239], %[vx], %[rd], %[sO] offen sc0 sc1\n"
#define A2A_DCLOB "v208", "v209", V10(21), V10(22), V10(23)
#define A2A_WIN [va0] "v"(v0), [va1] "v"(v0 + st), [va2] "v"((v0 ^ 64) + 2 * st), [va3] "v"((v0 ^ 64) + 3 * st), [rs] "s"(rs), [sP1] "s"(sP1)
      if (urgent) asm volatile("s_nop 3\n" A2A_XLOADS : [sO] "=&s"(sO) : [vx] "v"(vx), [rx] "s"(rx) : "memory", A2A_DCLOB);
      else asm volatile(HLOAD_ASM A2A_XLOADS : [sO] "=&s"(sO) : A2A_WIN, [vx] "v"(vx), [rx] "s"(rx) : "memory", "scc", A2A_DCLOB);
      unsigned call;
      asm volatile("s_waitcnt lgkmcnt(0)\n s_add_u32 %0, %1, 1" : "=s"(call) : "s"(cnt0) : "scc");
      const __amdgpu_buffer_rsrc_t rd =
          __builtin_amdgcn_make_buffer_rsrc((void*)(dws + ((call & 1) * 8 + rank) * CHUNK + poff), 0, q ? 32768 - sub * 128 : 0, 0x00020000);
      const __amdgpu_buffer_rsrc_t rfl_ =
          __builtin_amdgcn_make_buffer_rsrc((void*)(dws + FLAG_OFF + (sub ? 8192 + sub * 256 + rank * 32 + grp * 4 : rank * 1024 + grp * 128)), 0, q ? 4 : 0, 0x00020000);
      if (urgent) {
        asm volatile("s_waitcnt vmcnt(0)\n" A2A_XSTORES "s_barrier\n s_waitcnt vmcnt(0)\n buffer_store_dword %[f], off, %[rf], 0 sc0 sc1\n"
                     : [sO] "=&s"(sO)
                     : [vx] "v"(vx), [rd] "s"(rd), [f] "v"(call), [rf] "s"(rfl_)
                     : "memory");
      } else {
        asm volatile("s_waitcnt vmcnt(8)" ::: "memory");
      }
      __builtin_amdgcn_s_barrier();
      const __amdgpu_buffer_rsrc_t rcn = __builtin_amdgcn_make_buffer_rsrc((void*)cntp, 0, 4, 0x00020000);
      asm volatile("buffer_store_dword %0, off, %1, 0 sc1" ::"v"(call), "s"(rcn) : "memory");
      const __amdgpu_buffer_rsrc_t rf = __builtin_amdgcn_make_buffer_rsrc((void*)(myws + FLAG_OFF), 0, CTRL_BYTES, 0x00020000);
#define A2A_SAMPLE() asm volatile("buffer_load_dword v200, %0, %1, %2 offen sc0 sc1" ::"v"(lane * 4), "s"(rf), "s"(lo) : "memory", "v200")
      bool dead = false;
      for (unsigned k = 1; k <= GATE3; ++k) {
        if (sub != 0 && k == 2 * sub) {
          asm volatile("s_waitcnt vmcnt(0)\n" A2A_XSTORES "s_nop 1\n" : [sO] "=&s"(sO) : [vx] "v"(vx), [rd] "s"(rd) : "memory");
        }
        if (sub != 0 && k == 2 * sub + 2) {
          asm volatile("s_waitcnt vmcnt(0)\n buffer_store_dword %0, off, %1, 0 sc0 sc1" ::"v"(call), "s"(rfl_) : "memory");
        }
        const unsigned lo = k <= GATE1 ? 8448u : k <= GATE2 ? 8704u : 8960u;
        if (k == GATE1 - 1 || k == GATE2 - 1 || k == GATE3 - 1)
          A2A_SAMPLE();
        if (k == GATE1 || k == GATE2 || k == GATE3) {
          const unsigned long long tstart = __builtin_amdgcn_s_memrealtime();
          for (;;) {
            unsigned long long miss;
            asm volatile("s_waitcnt vmcnt(0)\n v_cmp_gt_u32 %0, %1, v200\n s_nop 3" : "=s"(miss) : "s"(call) : "memory");
            if (dead || (miss & ~(0xffull << (8 * rank))) == 0) break;
            if (__builtin_amdgcn_s_memrealtime() - tstart > 30000000ull) {
              asm volatile("buffer_store_dword %0, off, %1, %2 sc1\n s_waitcnt vmcnt(0)" ::"v"(call), "s"(rf), "s"(TMO_OFF) : "memory");
              dead = true;
              break;
            }
            __builtin_amdgcn_s_sleep(4);
            A2A_SAMPLE();
          }
        }
        if (k == GATE3)
          asm volatile("global_load_dwordx4 v[244:247], %0, %1" ::"v"(min((b >> 3) * 1024 + lane * 16, code_lim)), "s"(pc) : "memory", "v244", "v245", "v246", "v247");
        __builtin_amdgcn_s_barrier();
      }
      return;
    }
    const unsigned o = wave - 5;
    const __amdgpu_buffer_rsrc_t rs1 = __builtin_amdgcn_make_buffer_rsrc((void*)(a.x + m0 * 512), 0, 8 * CHUNK - m0 * 512, 0x00020000);
    const unsigned v0 = r * 512 + ((cp ^ (r >> 1)) << 4), st = 4096 - 1024;
    unsigned sM = A_OFF + o * 16384;
    sO = rank * CHUNK + o * 128u;
    asm volatile(ALOAD1_ASM
                 : [sO] "+s"(sO), [sM] "+s"(sM)
                 : [va0] "v"(v0), [va1] "v"((v0 ^ 64) + st), [va2] "v"(v0 + 2 * st), [va3] "v"((v0 ^ 64) + 3 * st), [rs1] "s"(rs1)
                 : "memory", "scc");
    unsigned call;
    asm volatile("s_waitcnt lgkmcnt(0)\n s_add_u32 %0, %1, 1" : "=s"(call) : "s"(cnt0) : "scc");
    const __amdgpu_buffer_rsrc_t rs2 = __builtin_amdgcn_make_buffer_rsrc((void*)(myws + (call & 1) * 8 * CHUNK + m0 * 512), 0, 8 * CHUNK - m0 * 512, 0x00020000);
    const __amdgpu_buffer_rsrc_t rf = __builtin_amdgcn_make_buffer_rsrc((void*)(myws + FLAG_OFF), 0, CTRL_BYTES, 0x00020000);
    unsigned ss, half;
    const unsigned uu = lane < 32 ? lane : lane - 32 + XLDIST;
    unit_of(uu & 31, ss, half);
    const unsigned pk = phys_kt(2 * ss + o, rank);
    const unsigned vtab = (lane < 32 || (uu < 32 && uu >= 32 - XLATE)) ? (pk >> 2) * CHUNK + (pk & 3) * 128u + half * (128u * 512u) : 0x7f000000u;
    unsigned sP, sQ, sI, sPf;
    sM = A_OFF + 65536 + o * 16384;
    sO = rank * CHUNK + (2 + o) * 128u;
    asm volatile(ALOAD2_ASM
                 : [sX] "=&s"(sX), [sY] "=&s"(sY), [sO] "+s"(sO), [sM] "+s"(sM), [sP] "=&s"(sP), [sQ] "=&s"(sQ), [sI] "=&s"(sI), [sPf] "=&s"(sPf)

                 : [va0] "v"(v0), [va1] "v"((v0 ^ 64) + st), [va2] "v"(v0 + 2 * st), [va3] "v"((v0 ^ 64) + 3 * st), [vtab] "v"(vtab),
                   [vpf] "v"(r * 512 + cp * 16 + (b >> 4) * 4096), [vfo] "v"(lane * 128), [sEx] "s"(~(0xffull << (8 * rank))), [sWl] "s"(a.w + n0 + (size_t)phys_kt(28 + 2 * o + mh, rank) * (128u * NCOL)), [sCall] "s"(call),
                   [rs1] "s"(rs1), [rs2] "s"(rs2), [rf] "s"(rf)
                 : "memory", "scc", "vcc", "s80", "s81", "s82", "s83", "s84", "s85", "s86", "s87", "v248", "v249", "v244", "v245", "v246", "v247");
    return;
  }

  if (b < 64 && ((b >> 3) & 7) != 0) __builtin_amdgcn_s_barrier();
  const unsigned wm = wave >> 1, wn = wave & 1;
  const unsigned g = lane >> 4, i16 = lane & 15;
  const unsigned xa0 = A_OFF + (wm * 64 + i16) * 128 + ((g ^ ((i16 >> 1) & 7)) << 4);
  const unsigned q = i16 >> 1, p = i16 & 1;
  const unsigned wa = (16 * g + q) * 128 + (((wn * 4) ^ ((q >> 1) | ((g & 1) << 2))) << 4) + 8 * p;
  const unsigned coff = ((m0 + wm * 64) * NCOL + n0 + wn * 64) * 4;
  const __amdgpu_buffer_rsrc_t rc = __builtin_amdgcn_make_buffer_rsrc((void*)((char*)a.out + coff), 0, 512u * NCOL * 4 - coff, 0x00020000);
  const __amdgpu_buffer_rsrc_t rt = __builtin_amdgcn_make_buffer_rsrc((void*)(myws + FLAG_OFF + TMO_OFF), 0, 4, 0x00020000);
  const unsigned cv = ((i16 & 7) * NCOL + 4 * g + (i16 >> 3) * 16) * 4;
  unsigned call;
  asm volatile("s_waitcnt lgkmcnt(0)\n s_add_u32 %0, %1, 1" : "=s"(call) : "s"(cnt0) : "scc");
  typedef const __attribute__((address_space(1))) float* gfp;
  unsigned sS = rfl(__builtin_bit_cast(unsigned, *(gfp)a.sx * *(gfp)a.sw));
  unsigned xc0, xc1, vtmp, vZ, t0, t1, t2, t3, sA, sI, sZ, sO;
  asm volatile("s_waitcnt vmcnt(0)\n" COMPUTE_ASM
               : [xc0] "=&v"(xc0), [xc1] "=&v"(xc1), [vtmp] "=&v"(vtmp), [vZ] "=&v"(vZ), [t0] "=&v"(t0), [t1] "=&v"(t1), [t2] "=&v"(t2), [t3] "=&v"(t3), [sS] "+s"(sS), [sA] "=&s"(sA), [sI] "=&s"(sI), [sZ] "=&s"(sZ), [sO] "=&s"(sO)
               : [xa0] "v"(xa0), [wr0] "v"(wa), [wr1] "v"(wa ^ 16), [wr2] "v"(wa ^ 32), [wr3] "v"(wa ^ 48), [cv] "v"(cv), [rc] "s"(rc), [rt] "s"(rt), [sCall] "s"(call)
               : "memory", "scc", VCLOB);
}
}


extern "C" void kernel_launch(int rank, int nranks, ncclComm_t comm, hipStream_t stream, void* const* d_in, const int* in_sizes, int n_in,
                              void* d_out, int out_size, const int* in_dtypes, int out_dtype, void* const* d_out_all, void* const* d_ws_all,
                              size_t ws_bytes) {
  (void)comm; (void)in_sizes; (void)n_in; (void)out_size; (void)in_dtypes; (void)out_dtype; (void)d_out_all;
  if (nranks != 8 || (unsigned)rank >= 8u || ws_bytes < (size_t)FLAG_OFF + CTRL_BYTES) {
    fprintf(stderr, "unsupported configuration\n");
    return;
  }
  hipError_t e = hipFuncSetAttribute((const void*)a2a_gemm_kernel, hipFuncAttributeMaxDynamicSharedMemorySize, LDS_BYTES);
  if (e != hipSuccess) { fprintf(stderr, "hipFuncSetAttribute: %s\n", hipGetErrorString(e)); return; }
  Args a;
  a.x = (const signed char*)d_in[0];
  a.w = (const signed char*)d_in[1];
  a.sx = (const float*)d_in[2];
  a.sw = (const float*)d_in[3];
  a.out = (float*)d_out;
  for (int p = 0; p < 8; ++p) a.ws[p] = (unsigned char*)d_ws_all[(rank + p) & 7];
  a.myws = (unsigned char*)d_ws_all[rank];
  a.rank = (unsigned)rank;
  a2a_gemm_kernel<<<256, 512, LDS_BYTES, stream>>>(a);
  e = hipGetLastError();
  if (e != hipSuccess) fprintf(stderr, "launch: %s\n", hipGetErrorString(e));
}
```
